```python
import numpy as np

kernel_name = "gpu_gemm_ar_m8192_k8192_n8192_bf16_none"


ABSMAX_RTOL = 0.008

COMPUTE_DTYPE = "bfloat16"
EPILOGUE = "none"
IN_QUANT = None
OUT_QUANT = None
IN_DTYPES = {"x": "bfloat16", "w_mat": "bfloat16"}
WORLD_SIZE = 8

def _to_bf16(x):
    u = np.ascontiguousarray(x, dtype=np.float32).view(np.uint32)
    return ((u + 0x7FFF + ((u >> 16) & 1)) & 0xFFFF0000).view(np.float32)

def setup_inputs():

    x = _to_bf16(np.random.randn(8192, 8192).astype(np.float32))
    w_mat = _to_bf16((np.random.randn(8192, 8192) / np.sqrt(8192)).astype(np.float32))
    return {"x": x, "w_mat": w_mat}

def _epilogue(y):
    return y

def reference(x, w_mat):
    y = (x @ w_mat).astype(np.float32)
    return _epilogue(y)
```

```cpp
#include <hip/hip_runtime.h>
#include <rccl/rccl.h>
#include <cstdio>

#define HIP_CHECK(expr) do { hipError_t _e = (expr); if (_e != hipSuccess) { \
    fprintf(stderr, "HIP error %s at %s:%d\n", hipGetErrorString(_e), __FILE__, __LINE__); \
    return; } } while (0)

#define LAS __attribute__((address_space(3)))
#define GAS __attribute__((address_space(1)))
typedef short bf16x8 __attribute__((ext_vector_type(8)));
typedef short s16x4 __attribute__((ext_vector_type(4)));
typedef float f32x4 __attribute__((ext_vector_type(4)));
typedef unsigned u32x4 __attribute__((ext_vector_type(4)));
typedef unsigned u32x2 __attribute__((ext_vector_type(2)));

namespace {
constexpr int NR = 8;
constexpr int DIM = 8192, KL = 1024;
constexpr unsigned PANEL_B = 4u << 20;
constexpr unsigned KT_B = 32768, HT_B = 16384;
constexpr unsigned CHUNK_B = 4 * KT_B;
constexpr unsigned XS_OFF = 0, WS_OFF = 16 * PANEL_B, STAGE_B = 24 * PANEL_B;
constexpr unsigned CNT_OFF = STAGE_B;
constexpr unsigned DONE_OFF = STAGE_B + 4096;
constexpr unsigned CALL_OFF = STAGE_B + 8192;
constexpr unsigned TMO_OFF = STAGE_B + 8192 + 128;
constexpr size_t WS_NEED = (size_t)STAGE_B + 16384;
constexpr unsigned OUT_B = (unsigned)DIM * DIM * 2;
constexpr int NPUSH = 128, NGEMM = 128;
constexpr int LDS_B = 131072;
constexpr unsigned long long DEADLINE = 100000000ull;
constexpr int AUX_SYS = 17;

#define PUSH_WAVES 128
#define PUSH_INFL 2
#define OUT_INFL 3

struct Params {
    const char* x; const char* w;
    char* out[NR]; char* ws[NR];
    int rank;
};

#define STR2(x) #x
#define STR(x) STR2(x)
#define WAIT_V(n) asm volatile("s_waitcnt vmcnt(" STR(n) ")" ::: "memory")
#define WAIT_L(n) asm volatile("s_waitcnt lgkmcnt(" STR(n) ")" ::: "memory")
#define BAR __builtin_amdgcn_s_barrier()
#define SCHED __builtin_amdgcn_sched_barrier(0)

__device__ __forceinline__ unsigned ld_sys(const char* p) {
    return __hip_atomic_load((const GAS unsigned*)p, __ATOMIC_RELAXED, __HIP_MEMORY_SCOPE_SYSTEM);
}
__device__ __forceinline__ void st_sys(char* p, unsigned v) {
    __hip_atomic_store((GAS unsigned*)p, v, __ATOMIC_RELAXED, __HIP_MEMORY_SCOPE_SYSTEM);
}
__device__ __forceinline__ void publish_add(char* ctr, int lane) {
    asm volatile("s_waitcnt vmcnt(0)" ::: "memory");
    if (lane == 0) {
        __builtin_amdgcn_fence(__ATOMIC_RELEASE, "");
        asm volatile("s_waitcnt vmcnt(0)" ::: "memory");
        __hip_atomic_fetch_add((GAS unsigned*)ctr, 1u, __ATOMIC_RELAXED, __HIP_MEMORY_SCOPE_SYSTEM);
    }
}
__device__ __forceinline__ unsigned cvt_pk_bf16(float lo, float hi) {
    unsigned r; asm volatile("v_cvt_pk_bf16_f32 %0, %1, %2" : "=v"(r) : "v"(lo), "v"(hi)); return r;
}

struct Piece { const char* src; unsigned sA, sB, dst, ctr; int isw; };

__device__ __forceinline__ Piece decode_piece(const Params& p, int q, int di, int dj) {
    int c, panel, pc, isw;
    if (q < 512) { c = q >> 7; const int r = q & 127, pi = r >> 3; isw = (pi & 1) ^ 1; panel = pi >> 1; pc = r & 7; }
    else { const int qq = q - 512; c = qq >> 6; const int r = qq & 63; isw = 0; panel = 8 + (r >> 3); pc = r & 7; }
    const int kt = pc >> 1, h = pc & 1, k0 = c * 256 + kt * 64;
    Piece o; o.isw = isw;
    const unsigned koff = (unsigned)((p.rank * 16 + c * 4 + kt) * 2 + h) * HT_B;
    if (isw) {
        o.src = p.w + ((size_t)k0 * DIM + dj * 2048 + panel * 256 + h * 128) * 2;
        o.sA = 8u * DIM * 2; o.sB = 4u * DIM * 2;
        o.dst = WS_OFF + (unsigned)panel * PANEL_B + koff;
        o.ctr = CNT_OFF + (unsigned)(16 + panel) * 128 + (unsigned)(p.rank * 4 + c) * 4;
    } else {
        o.src = p.x + ((size_t)(di * 4096 + panel * 256 + h * 128) * KL + k0) * 2;
        o.sA = 16u * KL * 2; o.sB = 64;
        o.dst = XS_OFF + (unsigned)panel * PANEL_B + koff;
        o.ctr = CNT_OFF + (unsigned)panel * 128 + (unsigned)(p.rank * 4 + c) * 4;
    }
    return o;
}

__device__ __forceinline__ void piece_dma(const Piece& pc, LAS char* buf, unsigned xl, unsigned wl) {
    const char* s = pc.src + (pc.isw ? wl : xl);
#pragma unroll
    for (int f = 0; f < 16; ++f)
        __builtin_amdgcn_global_load_lds((const GAS unsigned*)(s + (size_t)(f >> 1) * pc.sA + (size_t)(f & 1) * pc.sB),
                                         (LAS unsigned*)(buf + f * 1024), 16, 0, 0);
}

__device__ __forceinline__ void pusher(const Params& p, LAS char* lds, int pb) {
    const int wid = __builtin_amdgcn_readfirstlane(threadIdx.x >> 6), lane = threadIdx.x & 63;
    const int g = pb >> 3, dsel = (pb + g) & 7;
    const int d = (p.rank + dsel) & 7;
    const bool local = dsel == 0;
    const int nwv = local ? 128 : PUSH_WAVES;
    const int wv = wid * 16 + g;
    if (wv >= nwv) return;
    char* dws = p.ws[d];
    const __amdgpu_buffer_rsrc_t rs = __builtin_amdgcn_make_buffer_rsrc(dws, 0, STAGE_B, 0x00020000);
    const int di = d >> 2, dj = d & 3;
    LAS char* buf = lds + wid * HT_B;
    const unsigned xl = (unsigned)(lane & 15) * (KL * 2) + (unsigned)(lane >> 4) * 16;
    const unsigned wl = (unsigned)(lane >> 4) * (DIM * 2) + (unsigned)(lane & 15) * 16;
    const unsigned trl = (unsigned)(8 * (lane >> 4) + ((lane >> 2) & 3)) * 256 + (unsigned)(lane & 3) * 16;
    int q = wv;
    Piece cur = decode_piece(p, q, di, dj);
    piece_dma(cur, buf, xl, wl);
    WAIT_V(0);
    for (;;) {
        u32x4 v[16];
        if (cur.isw) {
#pragma unroll
            for (int f = 0; f < 16; ++f) {
                const int nb = f >> 1, kb = f & 1;
                LAS char* a = buf + trl + kb * 8192 + (nb >> 1) * 64 + (nb & 1) * 8;
                const u32x2 lo = __builtin_bit_cast(u32x2, __builtin_amdgcn_ds_read_tr16_b64_v4i16((LAS s16x4*)a));
                const u32x2 hi = __builtin_bit_cast(u32x2, __builtin_amdgcn_ds_read_tr16_b64_v4i16((LAS s16x4*)(a + 1024)));
                v[f] = (u32x4){lo.x, lo.y, hi.x, hi.y};
            }
        } else {
#pragma unroll
            for (int f = 0; f < 16; ++f) v[f] = *(const LAS u32x4*)(buf + f * 1024 + lane * 16);
        }
        WAIT_L(0);
#pragma unroll
        for (int f = 0; f < 16; ++f) asm volatile("" : "+v"(v[f]));
        const int qn = q + nwv;
        const bool more = qn < 768;
        Piece nxt = cur;
        if (more) { nxt = decode_piece(p, qn, di, dj); piece_dma(nxt, buf, xl, wl); }
        if (local) {
#pragma unroll
            for (int f = 0; f < 16; ++f) __builtin_amdgcn_raw_buffer_store_b128(v[f], rs, cur.dst + lane * 16 + f * 1024, 0, AUX_SYS);
        } else {
#pragma unroll
            for (int f = 0; f < 16; ++f) {
                __builtin_amdgcn_raw_buffer_store_b128(v[f], rs, cur.dst + lane * 16 + f * 1024, 0, AUX_SYS);
                WAIT_V(PUSH_INFL - 1);
            }
        }
        publish_add(dws + cur.ctr, lane);
        if (!more) break;
        cur = nxt; q = qn;
    }
}

__device__ __forceinline__ int chunk_slot(int s, int rank) {
    if (s < 4) return rank * 4 + s;
    const int r = (s - 4) / 7, j = (s - 4) - r * 7;
    return ((rank + 1 + j) & 7) * 4 + r;
}
__device__ __forceinline__ int ready_prefix(unsigned v, unsigned target) {
    const unsigned long long m = __ballot((int)(v - target) >= 0);
    const unsigned both = (unsigned)m & (unsigned)(m >> 32);
    return both == 0xffffffffu ? 32 : __builtin_ctz(~both);
}
__device__ __noinline__ int wait_chunk(const char* cptr, char* tmo, unsigned call, int need) {
    const unsigned long long t0 = __builtin_amdgcn_s_memrealtime();
    int known;
    for (;;) {
        known = ready_prefix(ld_sys(cptr), call * 8u);
        if (known > need) break;
        if (__builtin_amdgcn_readfirstlane(__hip_atomic_load((const GAS unsigned*)tmo, __ATOMIC_RELAXED, __HIP_MEMORY_SCOPE_AGENT)) == call
            || __builtin_amdgcn_s_memrealtime() - t0 > DEADLINE) {
            __hip_atomic_store((GAS unsigned*)tmo, call, __ATOMIC_RELAXED, __HIP_MEMORY_SCOPE_AGENT);
            known = 32; break;
        }
        __builtin_amdgcn_s_sleep(8);
    }
    __builtin_amdgcn_fence(__ATOMIC_ACQUIRE, "");
    asm volatile("s_waitcnt vmcnt(0)" ::: "memory");
    return known;
}

__device__ __forceinline__ void gemm_tile(const Params& p, LAS char* lds, int gb) {
    const int tid = threadIdx.x, wid = __builtin_amdgcn_readfirstlane(tid >> 6), lane = tid & 63, wr = wid >> 2, wc = wid & 3;
    const int xcd = gb & 7, sl = gb >> 3;
    const int pm = (sl >> 3) * 8 + (xcd >> 1) * 2 + ((sl >> 2) & 1), pn = (xcd & 1) * 4 + (sl & 3);
    char* ws = p.ws[p.rank];
    const unsigned call = __builtin_amdgcn_readfirstlane(ld_sys(ws + CALL_OFF)) + 1u;
    const unsigned target = call * 8u;
    const char* Ab = ws + XS_OFF + (size_t)pm * PANEL_B;
    const char* Bb = ws + WS_OFF + (size_t)pn * PANEL_B;
    const unsigned coff = CNT_OFF + (unsigned)(lane < 32 ? pm : 16 + pn) * 128 + (unsigned)chunk_slot(lane & 31, p.rank) * 4;
    const unsigned voff = (unsigned)tid * 16;
    const unsigned ldsw = (unsigned)wid * 1024u;
    const int aoff = wr * 8192 + lane * 16, boff = wc * 4096 + lane * 16;
#define SA(b, h) (((b) * 2 + (h)) * HT_B)
#define SB(b, h) ((4 + (b) * 2 + (h)) * HT_B)
#define STAGE(bufoff, gbase) do { _Pragma("unroll") for (int _i = 0; _i < 2; ++_i) \
        __builtin_amdgcn_global_load_lds((const GAS unsigned*)((const char*)(gbase) + voff + _i * 8192), (LAS unsigned*)(lds + (bufoff) + ldsw + _i * 8192), 16, 0, 0); } while (0)
#define LDA(dst, b, h) do { _Pragma("unroll") for (int m = 0; m < 4; ++m) _Pragma("unroll") for (int k = 0; k < 2; ++k) dst[m][k] = *(const LAS bf16x8*)(lds + SA(b, h) + aoff + m * 2048 + k * 1024); } while (0)
#define LDB(dst, b, h) do { _Pragma("unroll") for (int n = 0; n < 2; ++n) _Pragma("unroll") for (int k = 0; k < 2; ++k) dst[n][k] = *(const LAS bf16x8*)(lds + SB(b, h) + boff + n * 2048 + k * 1024); } while (0)
#define MMA(ai, bj, At, Bt) do { __builtin_amdgcn_s_setprio(1); _Pragma("unroll") for (int m = 0; m < 4; ++m) _Pragma("unroll") for (int n = 0; n < 2; ++n) _Pragma("unroll") for (int k = 0; k < 2; ++k) \
        acc[ai][bj][m][n] = __builtin_amdgcn_mfma_f32_16x16x32_bf16(Bt[n][k], At[m][k], acc[ai][bj][m][n], 0, 0, 0); __builtin_amdgcn_s_setprio(0); } while (0)
    f32x4 acc[2][2][4][2];
#pragma unroll
    for (int a = 0; a < 2; ++a)
#pragma unroll
        for (int b = 0; b < 2; ++b)
#pragma unroll
            for (int m = 0; m < 4; ++m)
#pragma unroll
                for (int n = 0; n < 2; ++n) acc[a][b][m][n] = (f32x4){0.f, 0.f, 0.f, 0.f};
    bf16x8 At[4][2], B0[2][2], B1[2][2];
    int known = wait_chunk(ws + coff, ws + TMO_OFF, call, 0);
    unsigned cur = (unsigned)chunk_slot(0, p.rank) * CHUNK_B;
    {
        const char* cA = Ab + cur; const char* cB = Bb + cur;
        STAGE(SB(0, 0), cB); STAGE(SA(0, 0), cA); STAGE(SB(0, 1), cB + HT_B); STAGE(SA(0, 1), cA + HT_B);
        if (wr == 1) BAR;
        WAIT_V(4); BAR;
        STAGE(SB(1, 0), cB + KT_B); STAGE(SA(1, 0), cA + KT_B); STAGE(SB(1, 1), cB + KT_B + HT_B);
        WAIT_V(6); BAR;
    }
    for (int T = 0; T < 64; ++T) {
        const int s = T >> 1;
        const bool last = T & 1;
        unsigned o1 = cur + (last ? 3 * KT_B : KT_B), o2 = cur + 2 * KT_B;
        if (last) {
            if (s + 1 < 32) {
                if (known <= s + 1) known = wait_chunk(ws + coff, ws + TMO_OFF, call, s + 1);
                cur = (unsigned)chunk_slot(s + 1, p.rank) * CHUNK_B;
            }
            o2 = cur;
        }
        const char* a1 = Ab + o1; const char* a2 = Ab + o2; const char* b2 = Bb + o2;
        const char* a3 = a2 + KT_B; const char* b3 = b2 + KT_B;
        unsigned pv;
        LDB(B0, 0, 0); SCHED; LDA(At, 0, 0);
        asm volatile("global_load_dword %0, %1, %2 sc0 sc1" : "=v"(pv) : "v"(coff), "s"(ws) : "memory");
        STAGE(SA(1, 1), a1 + HT_B);
        WAIT_L(8); BAR; WAIT_L(0); MMA(0, 0, At, B0); BAR; SCHED;
        LDB(B1, 0, 1); STAGE(SB(0, 0), b2);
        BAR; WAIT_L(0); MMA(0, 1, At, B1); BAR;
        LDA(At, 0, 1); STAGE(SA(0, 0), a2);
        BAR; WAIT_L(0); MMA(1, 0, At, B0); BAR; SCHED;
        STAGE(SB(0, 1), b2 + HT_B);
        WAIT_V(6);
        asm volatile("" : "+v"(pv));
        { const int k2 = ready_prefix(pv, target); known = k2 > known ? k2 : known; }
        BAR; MMA(1, 1, At, B1); BAR;
        LDB(B0, 1, 0); SCHED; LDA(At, 1, 0); STAGE(SA(0, 1), a2 + HT_B);
        WAIT_L(8); BAR; WAIT_L(0); MMA(0, 0, At, B0); BAR; SCHED;
        LDB(B1, 1, 1); STAGE(SB(1, 0), b3);
        BAR; WAIT_L(0); MMA(0, 1, At, B1); BAR;
        LDA(At, 1, 1); STAGE(SA(1, 0), a3);
        BAR; WAIT_L(0); MMA(1, 0, At, B0); BAR; SCHED;
        STAGE(SB(1, 1), b3 + HT_B);
        WAIT_V(6); BAR; MMA(1, 1, At, B1); BAR;
    }
    WAIT_V(0);
    if (wr == 0) BAR;
    BAR;
    const bool bad = __builtin_amdgcn_readfirstlane(__hip_atomic_load((const GAS unsigned*)(ws + TMO_OFF), __ATOMIC_RELAXED, __HIP_MEMORY_SCOPE_AGENT)) == call;
    {
        int tz = threadIdx.x; asm volatile("" : "+v"(tz));
        const int fr = tz & 15, fq = (tz >> 4) & 3;
#pragma unroll
        for (int ai = 0; ai < 2; ++ai)
#pragma unroll
            for (int m = 0; m < 4; ++m) {
                const int row = ai * 128 + wr * 64 + m * 16 + fr;
#pragma unroll
                for (int bj = 0; bj < 2; ++bj) {
                    const f32x4 v0 = acc[ai][bj][m][0], v1 = acc[ai][bj][m][1];
                    u32x4 o; o.x = cvt_pk_bf16(v0[0], v0[1]); o.y = cvt_pk_bf16(v0[2], v0[3]); o.z = cvt_pk_bf16(v1[0], v1[1]); o.w = cvt_pk_bf16(v1[2], v1[3]);
                    if (bad) o = (u32x4){0x7fc07fc0u, 0x7fc07fc0u, 0x7fc07fc0u, 0x7fc07fc0u};
                    const int ch = (bj * 16 + wc * 4 + fq) ^ (fr & 7);
                    *(LAS u32x4*)(lds + row * 512 + ch * 16) = o;
                }
            }
    }
    WAIT_L(0); BAR; asm volatile("" ::: "memory");
    {
        int tz = threadIdx.x; asm volatile("" : "+v"(tz));
        const int ln = tz & 63;
        const int d = (p.rank + wid) & 7;
        const __amdgpu_buffer_rsrc_t ro = __builtin_amdgcn_make_buffer_rsrc(p.out[d], 0, OUT_B, 0x00020000);
        const int di = p.rank >> 2, dj = p.rank & 3;
        const unsigned base = (unsigned)(di * 4096 + pm * 256) * (DIM * 2) + (unsigned)(dj * 2048 + pn * 256) * 2;
        const unsigned vo = (unsigned)(ln >> 5) * (DIM * 2) + (unsigned)(ln & 31) * 16;
        const int hi = ln >> 5, chn = ln & 31;
        for (int i0 = 0; i0 < 128; i0 += 4) {
            u32x4 v[4];
#pragma unroll
            for (int u = 0; u < 4; ++u) { const int row = 2 * (i0 + u) + hi; v[u] = *(const LAS u32x4*)(lds + row * 512 + ((chn ^ ((2 * u + hi) & 7)) * 16)); }
#pragma unroll
            for (int u = 0; u < 4; ++u) {
                __builtin_amdgcn_raw_buffer_store_b128(v[u], ro, vo + base + (unsigned)(2 * (i0 + u)) * (DIM * 2), 0, AUX_SYS);
                if (wid != 0) WAIT_V(OUT_INFL - 1);
            }
        }
        publish_add(p.ws[d] + DONE_OFF + p.rank * 128, ln);
    }
#undef SA
#undef SB
#undef STAGE
#undef LDA
#undef LDB
#undef MMA
}

__global__ __launch_bounds__(512, 2) void gemm_ar_kernel(Params p) {
    extern __shared__ __attribute__((aligned(16))) char smem[];
    LAS char* lds = (LAS char*)smem;
    const int b = blockIdx.x;
    if (b >= NPUSH) { gemm_tile(p, lds, b - NPUSH); return; }
    pusher(p, lds, b);
    if (b == 0 && threadIdx.x < 64) {
        char* ws = p.ws[p.rank];
        const int lane = threadIdx.x;
        const unsigned call = __builtin_amdgcn_readfirstlane(ld_sys(ws + CALL_OFF)) + 1u;
        const unsigned long long t0 = __builtin_amdgcn_s_memrealtime();
        bool dead = false;
        for (;;) {
            const unsigned v = ld_sys(ws + DONE_OFF + (lane & 7) * 128);
            if (__all((int)(v - call * (unsigned)NGEMM) >= 0)) break;
            if (__builtin_amdgcn_readfirstlane(__hip_atomic_load((const GAS unsigned*)(ws + TMO_OFF), __ATOMIC_RELAXED, __HIP_MEMORY_SCOPE_AGENT)) == call
                || __builtin_amdgcn_s_memrealtime() - t0 > DEADLINE) {
                __hip_atomic_store((GAS unsigned*)(ws + TMO_OFF), call, __ATOMIC_RELAXED, __HIP_MEMORY_SCOPE_AGENT);
                dead = true; break;
            }
            __builtin_amdgcn_s_sleep(4);
        }
        __builtin_amdgcn_fence(__ATOMIC_ACQUIRE, "");
        if (lane == 0) {
            if (dead) st_sys(p.out[p.rank] + ((size_t)(p.rank >> 2) * 4096 * DIM + (size_t)(p.rank & 3) * 2048) * 2, 0x7fc07fc0u);
            st_sys(ws + CALL_OFF, call);
        }
    }
}
}

extern "C" void kernel_launch(int rank, int nranks, ncclComm_t comm,
                              hipStream_t stream,
                              void* const* d_in, const int* in_sizes, int n_in,
                              void* d_out, int out_size,
                              const int* in_dtypes, int out_dtype,
                              void* const* d_out_all,
                              void* const* d_ws_all, size_t ws_bytes) {
    (void)comm; (void)in_dtypes; (void)out_dtype; (void)d_out;
    if (nranks != NR || (unsigned)rank >= (unsigned)NR || n_in < 2 || in_sizes[0] != DIM * KL || in_sizes[1] != KL * DIM ||
        out_size != DIM * DIM || ws_bytes < WS_NEED) {
        fprintf(stderr, "kernel_launch: unexpected problem shape\n");
        return;
    }
    Params p;
    p.x = (const char*)d_in[0]; p.w = (const char*)d_in[1]; p.rank = rank;
    for (int r = 0; r < NR; ++r) { p.out[r] = (char*)d_out_all[r]; p.ws[r] = (char*)d_ws_all[r]; }
    HIP_CHECK(hipFuncSetAttribute((const void*)gemm_ar_kernel, hipFuncAttributeMaxDynamicSharedMemorySize, LDS_B));
    hipLaunchKernelGGL(gemm_ar_kernel, dim3(NPUSH + NGEMM), dim3(512), LDS_B, stream, p);
    HIP_CHECK(hipGetLastError());
}
```

```cpp
#include <hip/hip_runtime.h>
#include <rccl/rccl.h>
#include <cstdio>

#define HIP_CHECK(expr) do { hipError_t _e = (expr); if (_e != hipSuccess) { \
    fprintf(stderr, "HIP error %s at %s:%d\n", hipGetErrorString(_e), __FILE__, __LINE__); \
    return; } } while (0)

#define LAS __attribute__((address_space(3)))
#define GAS __attribute__((address_space(1)))
typedef short bf16x8 __attribute__((ext_vector_type(8)));
typedef short s16x4 __attribute__((ext_vector_type(4)));
typedef float f32x4 __attribute__((ext_vector_type(4)));
typedef unsigned u32x4 __attribute__((ext_vector_type(4)));
typedef unsigned u32x2 __attribute__((ext_vector_type(2)));

namespace {
constexpr int NR = 8;
constexpr int DIM = 8192, KL = 1024;
constexpr unsigned PANEL_B = 4u << 20;
constexpr unsigned KT_B = 32768, HT_B = 16384;
constexpr unsigned CHUNK_B = 4 * KT_B;
constexpr unsigned XS_OFF = 0, WS_OFF = 16 * PANEL_B, STAGE_B = 24 * PANEL_B;
constexpr unsigned CNT_OFF = STAGE_B;
constexpr unsigned DONE_OFF = STAGE_B + 4096;
constexpr unsigned CALL_OFF = STAGE_B + 8192;
constexpr unsigned TMO_OFF = STAGE_B + 8192 + 128;
constexpr size_t WS_NEED = (size_t)STAGE_B + 16384;
constexpr unsigned OUT_B = (unsigned)DIM * DIM * 2;
constexpr int NPUSH = 128, NGEMM = 128;
constexpr int LDS_B = 131072;
constexpr unsigned long long DEADLINE = 100000000ull;
constexpr int AUX_SYS = 17;

#define PUSH_WAVES 128
#define PUSH_INFL 2
#define POLL_SLEEP 32
#define OUT_INFL 3

struct Params {
    const char* x; const char* w;
    char* out[NR]; char* ws[NR];
    int rank;
};

#define STR2(x) #x
#define STR(x) STR2(x)
#define WAIT_V(n) asm volatile("s_waitcnt vmcnt(" STR(n) ")" ::: "memory")
#define WAIT_L(n) asm volatile("s_waitcnt lgkmcnt(" STR(n) ")" ::: "memory")
#define BAR __builtin_amdgcn_s_barrier()
#define SCHED __builtin_amdgcn_sched_barrier(0)

#define STAMP(i) do {} while (0)
__device__ __forceinline__ unsigned ld_sys(const char* p) {
    return __hip_atomic_load((const GAS unsigned*)p, __ATOMIC_RELAXED, __HIP_MEMORY_SCOPE_SYSTEM);
}
__device__ __forceinline__ void st_sys(char* p, unsigned v) {
    __hip_atomic_store((GAS unsigned*)p, v, __ATOMIC_RELAXED, __HIP_MEMORY_SCOPE_SYSTEM);
}
__device__ __forceinline__ void publish_add(char* ctr, int lane) {
    asm volatile("s_waitcnt vmcnt(0)" ::: "memory");
    if (lane == 0) {
        __hip_atomic_fetch_add((GAS unsigned*)ctr, 1u, __ATOMIC_RELAXED, __HIP_MEMORY_SCOPE_SYSTEM);
    }
}
__device__ __forceinline__ unsigned cvt_pk_bf16(float lo, float hi) {
    unsigned r; asm volatile("v_cvt_pk_bf16_f32 %0, %1, %2" : "=v"(r) : "v"(lo), "v"(hi)); return r;
}

struct Piece { const char* src; unsigned sA, sB, dst, ctr; int isw; };

__device__ __forceinline__ Piece decode_piece(const Params& p, int q, int di, int dj) {
    int c, panel, pc, isw;
    if (q < 512) { c = q >> 7; const int r = q & 127, pi = r >> 3; isw = (pi & 1) ^ 1; panel = pi >> 1; pc = r & 7; }
    else { const int qq = q - 512; c = qq >> 6; const int r = qq & 63; isw = 0; panel = 8 + (r >> 3); pc = r & 7; }
    const int kt = pc >> 1, h = pc & 1, k0 = c * 256 + kt * 64;
    Piece o; o.isw = isw;
    const unsigned koff = (unsigned)((p.rank * 16 + c * 4 + kt) * 2 + h) * HT_B;
    if (isw) {
        o.src = p.w + ((size_t)k0 * DIM + dj * 2048 + panel * 256 + h * 128) * 2;
        o.sA = 8u * DIM * 2; o.sB = 4u * DIM * 2;
        o.dst = WS_OFF + (unsigned)panel * PANEL_B + koff;
        o.ctr = CNT_OFF + (unsigned)(16 + panel) * 128 + (unsigned)(p.rank * 4 + c) * 4;
    } else {
        o.src = p.x + ((size_t)(di * 4096 + panel * 256 + h * 128) * KL + k0) * 2;
        o.sA = 16u * KL * 2; o.sB = 64;
        o.dst = XS_OFF + (unsigned)panel * PANEL_B + koff;
        o.ctr = CNT_OFF + (unsigned)panel * 128 + (unsigned)(p.rank * 4 + c) * 4;
    }
    return o;
}

__device__ __forceinline__ void piece_dma(const Piece& pc, LAS char* buf, unsigned xl, unsigned wl) {
    const char* s = pc.src + (pc.isw ? wl : xl);
#pragma unroll
    for (int f = 0; f < 16; ++f)
        __builtin_amdgcn_global_load_lds((const GAS unsigned*)(s + (size_t)(f >> 1) * pc.sA + (size_t)(f & 1) * pc.sB),
                                         (LAS unsigned*)(buf + f * 1024), 16, 0, 0);
}

__device__ __forceinline__ void pusher(const Params& p, LAS char* lds, int pb) {
    const int wid = __builtin_amdgcn_readfirstlane(threadIdx.x >> 6), lane = threadIdx.x & 63;
    const int g = pb >> 3, dsel = (pb + g) & 7;
    const int d = (p.rank + dsel) & 7;
    const bool local = dsel == 0;
    const int nwv = local ? 128 : PUSH_WAVES;
    const int wv = wid * 16 + g;
    if (wv >= nwv) return;
    char* dws = p.ws[d];
    const __amdgpu_buffer_rsrc_t rs = __builtin_amdgcn_make_buffer_rsrc(dws, 0, STAGE_B, 0x00020000);
    const int di = d >> 2, dj = d & 3;
    LAS char* buf = lds + wid * HT_B;
    const unsigned xl = (unsigned)(lane & 15) * (KL * 2) + (unsigned)(lane >> 4) * 16;
    const unsigned wl = (unsigned)(lane >> 4) * (DIM * 2) + (unsigned)(lane & 15) * 16;
    const unsigned trl = (unsigned)(8 * (lane >> 4) + ((lane >> 2) & 3)) * 256 + (unsigned)(lane & 3) * 16;
    int q = wv;
    STAMP(0);
    int si = 1; (void)si;
    Piece cur = decode_piece(p, q, di, dj);
    piece_dma(cur, buf, xl, wl);
    WAIT_V(0);
    for (;;) {
        u32x4 v[16];
        if (cur.isw) {
#pragma unroll
            for (int f = 0; f < 16; ++f) {
                const int nb = f >> 1, kb = f & 1;
                LAS char* a = buf + trl + kb * 8192 + (nb >> 1) * 64 + (nb & 1) * 8;
                const u32x2 lo = __builtin_bit_cast(u32x2, __builtin_amdgcn_ds_read_tr16_b64_v4i16((LAS s16x4*)a));
                const u32x2 hi = __builtin_bit_cast(u32x2, __builtin_amdgcn_ds_read_tr16_b64_v4i16((LAS s16x4*)(a + 1024)));
                v[f] = (u32x4){lo.x, lo.y, hi.x, hi.y};
            }
        } else {
#pragma unroll
            for (int f = 0; f < 16; ++f) v[f] = *(const LAS u32x4*)(buf + f * 1024 + lane * 16);
        }
        WAIT_L(0);
#pragma unroll
        for (int f = 0; f < 16; ++f) asm volatile("" : "+v"(v[f]));
        const int qn = q + nwv;
        const bool more = qn < 768;
        Piece nxt = cur;
        if (more) { nxt = decode_piece(p, qn, di, dj); piece_dma(nxt, buf, xl, wl); }
        if (local) {
#pragma unroll
            for (int f = 0; f < 16; ++f) __builtin_amdgcn_raw_buffer_store_b128(v[f], rs, cur.dst + lane * 16 + f * 1024, 0, AUX_SYS);
        } else {
#pragma unroll
            for (int f = 0; f < 16; ++f) {
                __builtin_amdgcn_raw_buffer_store_b128(v[f], rs, cur.dst + lane * 16 + f * 1024, 0, AUX_SYS);
                WAIT_V(PUSH_INFL - 1);
            }
        }
        publish_add(dws + cur.ctr, lane);
        if (si < 16) { STAMP(si); } ++si;
        if (!more) break;
        cur = nxt; q = qn;
    }
}

__device__ __forceinline__ int chunk_slot(int s, int rank) {
    if (s < 4) return rank * 4 + s;
    const int r = (s - 4) / 7, j = (s - 4) - r * 7;
    return ((rank + 1 + j) & 7) * 4 + r;
}
__device__ __forceinline__ int ready_prefix(unsigned v, unsigned target) {
    const unsigned long long m = __ballot((int)(v - target) >= 0);
    const unsigned both = (unsigned)m & (unsigned)(m >> 32);
    return both == 0xffffffffu ? 32 : __builtin_ctz(~both);
}
__device__ __noinline__ int wait_chunk(const char* cptr, char* tmo, unsigned call, int need) {
    const unsigned long long t0 = __builtin_amdgcn_s_memrealtime();
    int known;
    for (unsigned polls = 1;; ++polls) {
        known = ready_prefix(ld_sys(cptr), call * 8u);
        if (known > need) break;
        if ((polls & 63u) == 0u) {
            if (__builtin_amdgcn_readfirstlane(__hip_atomic_load((const GAS unsigned*)tmo, __ATOMIC_RELAXED, __HIP_MEMORY_SCOPE_AGENT)) == call
                || __builtin_amdgcn_s_memrealtime() - t0 > DEADLINE) {
                __hip_atomic_store((GAS unsigned*)tmo, call, __ATOMIC_RELAXED, __HIP_MEMORY_SCOPE_AGENT);
                known = 32; break;
            }
        }
        __builtin_amdgcn_s_sleep(POLL_SLEEP);
    }
    __builtin_amdgcn_fence(__ATOMIC_ACQUIRE, "");
    asm volatile("s_waitcnt vmcnt(0)" ::: "memory");
    return known;
}

__device__ __forceinline__ void gemm_tile(const Params& p, LAS char* lds, int gb) {
    const int tid = threadIdx.x, wid = __builtin_amdgcn_readfirstlane(tid >> 6), lane = tid & 63, wr = wid >> 2, wc = wid & 3;
    const int xcd = gb & 7, sl = gb >> 3;
    const int pm = (sl >> 3) * 8 + (xcd >> 1) * 2 + ((sl >> 2) & 1), pn = (xcd & 1) * 4 + (sl & 3);
    char* ws = p.ws[p.rank];
    const unsigned call = __builtin_amdgcn_readfirstlane(ld_sys(ws + CALL_OFF)) + 1u;
    const unsigned target = call * 8u;
    const char* Ab = ws + XS_OFF + (size_t)pm * PANEL_B;
    const char* Bb = ws + WS_OFF + (size_t)pn * PANEL_B;
    const unsigned coff = CNT_OFF + (unsigned)(lane < 32 ? pm : 16 + pn) * 128 + (unsigned)chunk_slot(lane & 31, p.rank) * 4;
    const unsigned voff = (unsigned)tid * 16;
    const unsigned ldsw = (unsigned)wid * 1024u;
    const int aoff = wr * 8192 + lane * 16, boff = wc * 4096 + lane * 16;
#define SA(b, h) (((b) * 2 + (h)) * HT_B)
#define SB(b, h) ((4 + (b) * 2 + (h)) * HT_B)
#define STAGE(bufoff, gbase) do { _Pragma("unroll") for (int _i = 0; _i < 2; ++_i) \
        __builtin_amdgcn_global_load_lds((const GAS unsigned*)((const char*)(gbase) + voff + _i * 8192), (LAS unsigned*)(lds + (bufoff) + ldsw + _i * 8192), 16, 0, 0); } while (0)
#define LDA(dst, b, h) do { _Pragma("unroll") for (int m = 0; m < 4; ++m) _Pragma("unroll") for (int k = 0; k < 2; ++k) dst[m][k] = *(const LAS bf16x8*)(lds + SA(b, h) + aoff + m * 2048 + k * 1024); } while (0)
#define LDB(dst, b, h) do { _Pragma("unroll") for (int n = 0; n < 2; ++n) _Pragma("unroll") for (int k = 0; k < 2; ++k) dst[n][k] = *(const LAS bf16x8*)(lds + SB(b, h) + boff + n * 2048 + k * 1024); } while (0)
#define MMA(ai, bj, At, Bt) do { __builtin_amdgcn_s_setprio(1); _Pragma("unroll") for (int m = 0; m < 4; ++m) _Pragma("unroll") for (int n = 0; n < 2; ++n) _Pragma("unroll") for (int k = 0; k < 2; ++k) \
        acc[ai][bj][m][n] = __builtin_amdgcn_mfma_f32_16x16x32_bf16(Bt[n][k], At[m][k], acc[ai][bj][m][n], 0, 0, 0); __builtin_amdgcn_s_setprio(0); } while (0)
    f32x4 acc[2][2][4][2];
#pragma unroll
    for (int a = 0; a < 2; ++a)
#pragma unroll
        for (int b = 0; b < 2; ++b)
#pragma unroll
            for (int m = 0; m < 4; ++m)
#pragma unroll
                for (int n = 0; n < 2; ++n) acc[a][b][m][n] = (f32x4){0.f, 0.f, 0.f, 0.f};
    bf16x8 At[4][2], B0[2][2], B1[2][2];
    STAMP(0);
    int known = wait_chunk(ws + coff, ws + TMO_OFF, call, 0);
    STAMP(1);
    unsigned cur = (unsigned)chunk_slot(0, p.rank) * CHUNK_B;
    {
        const char* cA = Ab + cur; const char* cB = Bb + cur;
        STAGE(SB(0, 0), cB); STAGE(SA(0, 0), cA); STAGE(SB(0, 1), cB + HT_B); STAGE(SA(0, 1), cA + HT_B);
        if (wr == 1) BAR;
        WAIT_V(4); BAR;
        STAGE(SB(1, 0), cB + KT_B); STAGE(SA(1, 0), cA + KT_B); STAGE(SB(1, 1), cB + KT_B + HT_B);
        WAIT_V(6); BAR;
    }
    for (int T = 0; T < 64; ++T) {
        const int s = T >> 1;
        const bool last = T & 1;
        unsigned o1 = cur + (last ? 3 * KT_B : KT_B), o2 = cur + 2 * KT_B;
        if (last) {
            if (s + 1 < 32) {
                if (known <= s + 1) known = wait_chunk(ws + coff, ws + TMO_OFF, call, s + 1);
                cur = (unsigned)chunk_slot(s + 1, p.rank) * CHUNK_B;
            }
            o2 = cur;
        }
        const char* a1 = Ab + o1; const char* a2 = Ab + o2; const char* b2 = Bb + o2;
        const char* a3 = a2 + KT_B; const char* b3 = b2 + KT_B;
        unsigned pv;
        LDB(B0, 0, 0); SCHED; LDA(At, 0, 0);
        asm volatile("global_load_dword %0, %1, %2 sc0 sc1" : "=v"(pv) : "v"(coff), "s"(ws) : "memory");
        STAGE(SA(1, 1), a1 + HT_B);
        WAIT_L(8); BAR; WAIT_L(0); MMA(0, 0, At, B0); BAR; SCHED;
        LDB(B1, 0, 1); STAGE(SB(0, 0), b2);
        BAR; WAIT_L(0); MMA(0, 1, At, B1); BAR;
        LDA(At, 0, 1); STAGE(SA(0, 0), a2);
        BAR; WAIT_L(0); MMA(1, 0, At, B0); BAR; SCHED;
        STAGE(SB(0, 1), b2 + HT_B);
        WAIT_V(6);
        asm volatile("" : "+v"(pv));
        { const int k2 = ready_prefix(pv, target); known = k2 > known ? k2 : known; }
        BAR; MMA(1, 1, At, B1); BAR;
        LDB(B0, 1, 0); SCHED; LDA(At, 1, 0); STAGE(SA(0, 1), a2 + HT_B);
        WAIT_L(8); BAR; WAIT_L(0); MMA(0, 0, At, B0); BAR; SCHED;
        LDB(B1, 1, 1); STAGE(SB(1, 0), b3);
        BAR; WAIT_L(0); MMA(0, 1, At, B1); BAR;
        LDA(At, 1, 1); STAGE(SA(1, 0), a3);
        BAR; WAIT_L(0); MMA(1, 0, At, B0); BAR; SCHED;
        STAGE(SB(1, 1), b3 + HT_B);
        WAIT_V(6); BAR; MMA(1, 1, At, B1); BAR;
    }
    WAIT_V(0);
    if (wr == 0) BAR;
    BAR;
    STAMP(10);
    const bool bad = __builtin_amdgcn_readfirstlane(__hip_atomic_load((const GAS unsigned*)(ws + TMO_OFF), __ATOMIC_RELAXED, __HIP_MEMORY_SCOPE_AGENT)) == call;
    {
        int tz = threadIdx.x; asm volatile("" : "+v"(tz));
        const int fr = tz & 15, fq = (tz >> 4) & 3;
#pragma unroll
        for (int ai = 0; ai < 2; ++ai)
#pragma unroll
            for (int m = 0; m < 4; ++m) {
                const int row = ai * 128 + wr * 64 + m * 16 + fr;
#pragma unroll
                for (int bj = 0; bj < 2; ++bj) {
                    const f32x4 v0 = acc[ai][bj][m][0], v1 = acc[ai][bj][m][1];
                    u32x4 o; o.x = cvt_pk_bf16(v0[0], v0[1]); o.y = cvt_pk_bf16(v0[2], v0[3]); o.z = cvt_pk_bf16(v1[0], v1[1]); o.w = cvt_pk_bf16(v1[2], v1[3]);
                    if (bad) o = (u32x4){0x7fc07fc0u, 0x7fc07fc0u, 0x7fc07fc0u, 0x7fc07fc0u};
                    const int ch = (bj * 16 + wc * 4 + fq) ^ (fr & 7);
                    *(LAS u32x4*)(lds + row * 512 + ch * 16) = o;
                }
            }
    }
    WAIT_L(0); BAR; asm volatile("" ::: "memory");
    {
        int tz = threadIdx.x; asm volatile("" : "+v"(tz));
        const int ln = tz & 63;
        const int d = (p.rank + wid) & 7;
        const __amdgpu_buffer_rsrc_t ro = __builtin_amdgcn_make_buffer_rsrc(p.out[d], 0, OUT_B, 0x00020000);
        const int di = p.rank >> 2, dj = p.rank & 3;
        const unsigned base = (unsigned)(di * 4096 + pm * 256) * (DIM * 2) + (unsigned)(dj * 2048 + pn * 256) * 2;
        const unsigned vo = (unsigned)(ln >> 5) * (DIM * 2) + (unsigned)(ln & 31) * 16;
        const int hi = ln >> 5, chn = ln & 31;
        for (int i0 = 0; i0 < 128; i0 += 4) {
            u32x4 v[4];
#pragma unroll
            for (int u = 0; u < 4; ++u) { const int row = 2 * (i0 + u) + hi; v[u] = *(const LAS u32x4*)(lds + row * 512 + ((chn ^ ((2 * u + hi) & 7)) * 16)); }
#pragma unroll
            for (int u = 0; u < 4; ++u) {
                __builtin_amdgcn_raw_buffer_store_b128(v[u], ro, vo + base + (unsigned)(2 * (i0 + u)) * (DIM * 2), 0, AUX_SYS);
                if (wid != 0) WAIT_V(OUT_INFL - 1);
            }
        }
        publish_add(p.ws[d] + DONE_OFF + p.rank * 128, ln);
        STAMP(11);
    }
#undef SA
#undef SB
#undef STAGE
#undef LDA
#undef LDB
#undef MMA
}

__global__ __launch_bounds__(512, 2) void gemm_ar_kernel(Params p) {
    extern __shared__ __attribute__((aligned(16))) char smem[];
    LAS char* lds = (LAS char*)smem;
    const int b = blockIdx.x;
    if (b >= NPUSH) { gemm_tile(p, lds, b - NPUSH); return; }
    pusher(p, lds, b);
    if (b == 0 && threadIdx.x < 64) {
        char* ws = p.ws[p.rank];
        const int lane = threadIdx.x;
        const unsigned call = __builtin_amdgcn_readfirstlane(ld_sys(ws + CALL_OFF)) + 1u;
        const unsigned long long t0 = __builtin_amdgcn_s_memrealtime();
        bool dead = false;
        for (;;) {
            const unsigned v = ld_sys(ws + DONE_OFF + (lane & 7) * 128);
            if (__all((int)(v - call * (unsigned)NGEMM) >= 0)) break;
            if (__builtin_amdgcn_readfirstlane(__hip_atomic_load((const GAS unsigned*)(ws + TMO_OFF), __ATOMIC_RELAXED, __HIP_MEMORY_SCOPE_AGENT)) == call
                || __builtin_amdgcn_s_memrealtime() - t0 > DEADLINE) {
                __hip_atomic_store((GAS unsigned*)(ws + TMO_OFF), call, __ATOMIC_RELAXED, __HIP_MEMORY_SCOPE_AGENT);
                dead = true; break;
            }
            __builtin_amdgcn_s_sleep(4);
        }
        __builtin_amdgcn_fence(__ATOMIC_ACQUIRE, "");
        if (lane == 0) {
            if (dead) st_sys(p.out[p.rank] + ((size_t)(p.rank >> 2) * 4096 * DIM + (size_t)(p.rank & 3) * 2048) * 2, 0x7fc07fc0u);
            st_sys(ws + CALL_OFF, call);
            STAMP(15);
        }
    }
}
}

extern "C" void kernel_launch(int rank, int nranks, ncclComm_t comm,
                              hipStream_t stream,
                              void* const* d_in, const int* in_sizes, int n_in,
                              void* d_out, int out_size,
                              const int* in_dtypes, int out_dtype,
                              void* const* d_out_all,
                              void* const* d_ws_all, size_t ws_bytes) {
    (void)comm; (void)in_dtypes; (void)out_dtype; (void)d_out;
    if (nranks != NR || (unsigned)rank >= (unsigned)NR || n_in < 2 || in_sizes[0] != DIM * KL || in_sizes[1] != KL * DIM ||
        out_size != DIM * DIM || ws_bytes < WS_NEED) {
        fprintf(stderr, "kernel_launch: unexpected problem shape\n");
        return;
    }
    Params p;
    p.x = (const char*)d_in[0]; p.w = (const char*)d_in[1]; p.rank = rank;
    for (int r = 0; r < NR; ++r) { p.out[r] = (char*)d_out_all[r]; p.ws[r] = (char*)d_ws_all[r]; }
    HIP_CHECK(hipFuncSetAttribute((const void*)gemm_ar_kernel, hipFuncAttributeMaxDynamicSharedMemorySize, LDS_B));
    hipLaunchKernelGGL(gemm_ar_kernel, dim3(NPUSH + NGEMM), dim3(512), LDS_B, stream, p);
    HIP_CHECK(hipGetLastError());
}
```

```cpp
#include <hip/hip_runtime.h>
#include <rccl/rccl.h>
#include <cstdio>

#define HIP_CHECK(expr) do { hipError_t _e = (expr); if (_e != hipSuccess) { \
    fprintf(stderr, "HIP error %s at %s:%d\n", hipGetErrorString(_e), __FILE__, __LINE__); \
    return; } } while (0)

#define LAS __attribute__((address_space(3)))
#define GAS __attribute__((address_space(1)))
typedef short bf16x8 __attribute__((ext_vector_type(8)));
typedef short s16x4 __attribute__((ext_vector_type(4)));
typedef float f32x4 __attribute__((ext_vector_type(4)));
typedef unsigned u32x4 __attribute__((ext_vector_type(4)));
typedef unsigned u32x2 __attribute__((ext_vector_type(2)));

namespace {
constexpr int NR = 8;
constexpr int DIM = 8192, KL = 1024;
constexpr int NI = 5, NX = 3;
constexpr unsigned KT_B = 32768, HT_B = 16384;
constexpr unsigned SLOT_B = 2 * KT_B;
constexpr unsigned PANEL_B = 32 * SLOT_B;
constexpr unsigned STAGE_B = 32 * PANEL_B;
constexpr unsigned TILE_B = 131072;
constexpr unsigned PIN_OFF = STAGE_B;
constexpr unsigned POUT_OFF = PIN_OFF + 128 * TILE_B;
constexpr unsigned DATA_B = POUT_OFF + 128 * TILE_B;
constexpr unsigned CNT_OFF = DATA_B;
constexpr unsigned PF_OFF = DATA_B + 4096;
constexpr unsigned FL_OFF = DATA_B + 8192;
constexpr unsigned PL_OFF = DATA_B + 12288;
constexpr unsigned DONE_OFF = DATA_B + 16384;
constexpr unsigned TAIL_OFF = DONE_OFF + 1024;
constexpr unsigned HEAD_OFF = DONE_OFF + 4096;
constexpr unsigned CALL_OFF = DONE_OFF + 2048;
constexpr unsigned TMO_OFF = CALL_OFF + 128;
constexpr size_t WS_NEED = (size_t)DATA_B + 32768;
constexpr unsigned OUT_B = (unsigned)DIM * DIM * 2;
#define LINK_WGS 8
#define SENDERS 2
constexpr int NE = LINK_WGS;
constexpr int NS = SENDERS;
constexpr int NLINK = 8 * NE, NGEMM = 256 - NLINK;
constexpr int LDS_B = 131072;
constexpr unsigned long long DEADLINE = 100000000ull;
constexpr int AUX_SYS = 17;

#define WIN 3
#define STAGE_AUX 16
#define POLL_SLEEP 32

struct Params {
    const char* x; const char* w;
    char* out[NR]; char* ws[NR];
    int rank;
    unsigned units[8 * (NI + NX)];
    unsigned tiles[NGEMM];
};

#define STR2(x) #x
#define STR(x) STR2(x)
#define WAIT_V(n) asm volatile("s_waitcnt vmcnt(" STR(n) ")" ::: "memory")
#define WAIT_L(n) asm volatile("s_waitcnt lgkmcnt(" STR(n) ")" ::: "memory")
#define BAR __builtin_amdgcn_s_barrier()
#define SCHED __builtin_amdgcn_sched_barrier(0)

#define STAMPV(i, v) do {} while (0)
#define STAMP(i) do {} while (0)

__device__ __forceinline__ unsigned ld_sys(const char* p) {
    return __hip_atomic_load((const GAS unsigned*)p, __ATOMIC_RELAXED, __HIP_MEMORY_SCOPE_SYSTEM);
}
__device__ __forceinline__ void st_sys(char* p, unsigned v) {
    __hip_atomic_store((GAS unsigned*)p, v, __ATOMIC_RELAXED, __HIP_MEMORY_SCOPE_SYSTEM);
}
__device__ __forceinline__ void add_sys(char* p, unsigned v) {
    __hip_atomic_fetch_add((GAS unsigned*)p, v, __ATOMIC_RELAXED, __HIP_MEMORY_SCOPE_SYSTEM);
}
__device__ __forceinline__ unsigned cvt_pk_bf16(float lo, float hi) {
    unsigned r; asm volatile("v_cvt_pk_bf16_f32 %0, %1, %2" : "=v"(r) : "v"(lo), "v"(hi)); return r;
}
__device__ __forceinline__ int panel_row(int d, int P) {
    const int g = d >> 2, ai = (d >> 1) & 1, k = P & 7, u = k & 1, i = ((P >> 3) ? g : 1 - g) * 4 + (k >> 1);
    return (g == 0 ? ai * 2 + u : u * 2 + ai) * 2048 + i * 256;
}
__device__ __forceinline__ int panel_col(int d, int k) {
    const int g = d >> 2, aj = d & 1, v = k & 1;
    return (g == 0 ? aj * 2 + v : v * 2 + aj) * 2048 + (k >> 1) * 256;
}
__device__ __forceinline__ bool give_up(char* tmo, unsigned call, unsigned long long t0) {
    if (__builtin_amdgcn_readfirstlane(__hip_atomic_load((const GAS unsigned*)tmo, __ATOMIC_RELAXED, __HIP_MEMORY_SCOPE_AGENT)) == call
        || __builtin_amdgcn_s_memrealtime() - t0 > DEADLINE) {
        __hip_atomic_store((GAS unsigned*)tmo, call, __ATOMIC_RELAXED, __HIP_MEMORY_SCOPE_AGENT);
        return true;
    }
    return false;
}

struct Piece { const char* src; unsigned sA, sB, dst, pub; int kind; };

__device__ __forceinline__ Piece input_piece(const Params& p, int q, int dsel) {
    const int d = p.rank ^ dsel;
    const int ui = (q >> 2) + (dsel >= 4 ? 32 * NI : 0);
    const int u = (p.units[ui >> 2] >> ((ui & 3) * 8)) & 255;
    const int panel = u & 31, cc = u >> 5, pc = q & 3;
    const int c = dsel >= 4 ? NI + cc : cc;
    const int slot = dsel == 0 ? cc : (dsel >= 4 ? NI + cc * 4 + (dsel & 3) : NI + 4 * NX + cc * 3 + dsel - 1);
    const int k0 = c * 128 + (pc >> 1) * 64, h = pc & 1;
    Piece o; o.kind = panel >= 16;
    o.dst = (unsigned)panel * PANEL_B + (unsigned)slot * SLOT_B + (unsigned)pc * HT_B;
    o.pub = CNT_OFF + (unsigned)panel * 128 + (unsigned)slot * 4;
    if (o.kind) {
        o.src = p.w + ((size_t)k0 * DIM + panel_col(d, panel - 16) + h * 128) * 2;
        o.sA = 8u * DIM * 2; o.sB = 4u * DIM * 2;
    } else {
        o.src = p.x + ((size_t)(panel_row(d, panel) + h * 128) * KL + k0) * 2;
        o.sA = 16u * KL * 2; o.sB = 64;
    }
    return o;
}
__device__ __forceinline__ Piece partial_piece(const Params& p, int id) {
    const int pm = (id >> 7) & 7, pn = (id >> 3) & 15, w = id & 7;
    const int ot = ((pm & 6) | ((p.rank >> 1) & 1)) * 16 + ((pn & 14) | (p.rank & 1));
    Piece o; o.kind = 2;
    o.src = p.ws[p.rank] + POUT_OFF + (size_t)(id & 1023) * HT_B;
    o.sA = 2048; o.sB = 1024;
    o.dst = PIN_OFF + (unsigned)(ot * 8 + w) * HT_B;
    o.pub = PF_OFF + (unsigned)(ot * 8 + w) * 4;
    return o;
}
__device__ __forceinline__ Piece output_piece(const Params& p, int id) {
    const int pm = (id >> 7) & 7, pn = (id >> 3) & 15, w = id & 7;
    Piece o; o.kind = 3;
    o.dst = (unsigned)(panel_row(p.rank, 8 + pm) + w * 32) * (DIM * 2) + (unsigned)panel_col(p.rank, pn) * 2;
    o.src = p.out[p.rank] + o.dst;
    o.sA = 4u * DIM * 2; o.sB = 2u * DIM * 2;
    o.pub = 0;
    return o;
}
__device__ __forceinline__ const char* piece_src(const Piece& pc, int lane) {
    const unsigned xl = (unsigned)(lane & 15) * (KL * 2) + (unsigned)(lane >> 4) * 16;
    const unsigned wl = (unsigned)(lane >> 4) * (DIM * 2) + (unsigned)(lane & 15) * 16;
    const unsigned ol = (unsigned)(lane >> 5) * (DIM * 2) + (unsigned)(lane & 31) * 16;
    return pc.src + (pc.kind == 0 ? xl : pc.kind == 1 ? wl : pc.kind == 2 ? (unsigned)lane * 16 : ol);
}
__device__ __forceinline__ void piece_dma2(const Piece& pc, const char* s, LAS char* buf, int i) {
    __builtin_amdgcn_global_load_lds((const GAS unsigned*)(s + (size_t)i * pc.sA), (LAS unsigned*)(buf + i * 2048), 16, 0, 16);
    __builtin_amdgcn_global_load_lds((const GAS unsigned*)(s + (size_t)i * pc.sA + pc.sB), (LAS unsigned*)(buf + i * 2048 + 1024), 16, 0, 16);
}
__device__ __forceinline__ void piece_dma(const Piece& pc, LAS char* buf, int lane) {
    const char* s = piece_src(pc, lane);
#pragma unroll
    for (int i = 0; i < 8; ++i) piece_dma2(pc, s, buf, i);
}
__device__ __forceinline__ void piece_read(u32x4 (&v)[16], int kind, LAS char* buf, int lane) {
    if (kind == 1) {
        const unsigned trl = (unsigned)(8 * (lane >> 4) + ((lane >> 2) & 3)) * 256 + (unsigned)(lane & 3) * 16;
#pragma unroll
        for (int f = 0; f < 16; ++f) {
            const int nb = f >> 1, kb = f & 1;
            LAS char* a = buf + trl + kb * 8192 + (nb >> 1) * 64 + (nb & 1) * 8;
            u32x2 lo, hi;
            asm volatile("ds_read_b64_tr_b16 %0, %1" : "=v"(lo) : "v"(a) : "memory");
            asm volatile("ds_read_b64_tr_b16 %0, %1 offset:1024" : "=v"(hi) : "v"(a) : "memory");
            asm volatile("s_waitcnt lgkmcnt(0)" : "+v"(lo), "+v"(hi) :: "memory");
            v[f] = (u32x4){lo.x, lo.y, hi.x, hi.y};
        }
    } else {
#pragma unroll
        for (int f = 0; f < 16; ++f) asm volatile("ds_read_b128 %0, %1 offset:%2" : "=v"(v[f]) : "v"(buf + lane * 16), "i"(f * 1024) : "memory");
        asm volatile("s_waitcnt lgkmcnt(0)" : "+v"(v[0]), "+v"(v[1]), "+v"(v[2]), "+v"(v[3]), "+v"(v[4]), "+v"(v[5]), "+v"(v[6]), "+v"(v[7]),
                     "+v"(v[8]), "+v"(v[9]), "+v"(v[10]), "+v"(v[11]), "+v"(v[12]), "+v"(v[13]), "+v"(v[14]), "+v"(v[15]) :: "memory");
    }
}

constexpr int KSL = 3;
constexpr int NMB = 3 * KSL;

__device__ __forceinline__ int next_task(LAS unsigned* mb, int lane) {
    for (;;) {
        unsigned v;
        asm volatile("ds_read_b32 %0, %1\n\ts_waitcnt lgkmcnt(0)" : "=&v"(v) : "v"(mb + (lane & 15)) : "memory");
        unsigned best = 0xffffffffu; int bl = 0;
#pragma unroll
        for (int l = 0; l < NMB; ++l) { const unsigned k = (unsigned)__builtin_amdgcn_readlane(v, l) - 1u; if (k < best) { best = k; bl = l; } }
        if (best == 0xffffffffu) return __builtin_amdgcn_readlane(v, NMB) ? -2 : -1;
        unsigned old = 0;
        if (lane == 0) asm volatile("ds_cmpst_rtn_b32 %0, %1, %2, %3\n\ts_waitcnt lgkmcnt(0)" : "=&v"(old) : "v"(mb + bl), "v"(best + 1u), "v"(0u) : "memory");
        if ((unsigned)__builtin_amdgcn_readfirstlane(old) == best + 1u) {
            const int cls = best >> 21;
            return cls << 12 | (cls == 1 ? (int)(best >> 10) & 2047 : (int)best & 1023);
        }
    }
}
__device__ __forceinline__ Piece task_piece(const Params& p, int code, int dsel) {
    const int cls = code >> 12, id = code & 4095;
    return cls == 1 ? input_piece(p, id, dsel) : cls == 0 ? partial_piece(p, id) : output_piece(p, id);
}

constexpr int younger(int f) {
    int n = 0;
    for (int i = f + 1 - WIN; i <= f; ++i) n += (i >= 0 && i < 8) ? 3 : 1;
    return n - 1;
}
static_assert(WIN >= 1 && WIN <= 8, "window");

__device__ __forceinline__ void link_wg(const Params& p, LAS char* lds, int dsel, int e) {
    const int wid = __builtin_amdgcn_readfirstlane(threadIdx.x >> 6), lane = threadIdx.x & 63;
    char* ws = p.ws[p.rank];
    LAS unsigned* mb = (LAS unsigned*)(lds + 7 * HT_B);
    LAS char* buf = lds + wid * HT_B;
    STAMPV(0, dsel);
    Piece cur;
    if (wid < NS) { cur = input_piece(p, wid * NE + e, dsel); piece_dma(cur, buf, lane); }
    else if (wid == 7) { if (lane < 16) mb[lane] = 0u; WAIT_L(0); }
    const unsigned call = __builtin_amdgcn_readfirstlane(ld_sys(ws + CALL_OFF)) + 1u;
    BAR; asm volatile("" ::: "memory");
    if (wid >= NS) {
        if (wid != 7) return;
        const int cls = lane < KSL ? 0 : lane < 2 * KSL ? 1 : 2;
        const unsigned first = cls == 1 ? NS * NE : 0u;
        const unsigned n = cls == 0 ? 256u : cls == 2 ? 1024u : dsel >= 4 ? 32u * NX * 4 : 32u * NI * 4;
        const unsigned base = (call - 1u) * (n - first + NE * KSL) - first;
        char* head = ws + HEAD_OFF + (dsel * 4 + cls) * 128;
        const char* list = ws + (cls == 0 ? PL_OFF + (dsel & 3) * 1024 : FL_OFF);
        const unsigned tag = call & 0xfffffu;
        int st = (lane >= NMB || (cls == 0 && dsel < 4)) ? 3 : 0;
        unsigned pos = 0;
        const unsigned long long t0 = __builtin_amdgcn_s_memrealtime();
        for (unsigned polls = 1;; ++polls) {
            if (st == 2 && __hip_atomic_load(mb + lane, __ATOMIC_RELAXED, __HIP_MEMORY_SCOPE_WORKGROUP) == 0u) st = 0;
            unsigned t = 0, en = 0;
            if (st == 0) t = __hip_atomic_fetch_add((GAS unsigned*)head, 1u, __ATOMIC_RELAXED, __HIP_MEMORY_SCOPE_AGENT);
            if (st == 1) en = ld_sys(list + pos * 4);
            bool post = false;
            if (st == 0) { pos = t - base; st = pos >= n ? 3 : 1; post = st == 1 && cls == 1; }
            else if (st == 1) post = (en >> 12) == tag;
            if (post) {
                __hip_atomic_store(mb + lane, ((((unsigned)cls << 11 | pos) << 10) | (en & 1023u)) + 1u, __ATOMIC_RELAXED, __HIP_MEMORY_SCOPE_WORKGROUP);
                st = 2;
            }
            if (__all(st == 3) || ((polls & 63u) == 0u && give_up(ws + TMO_OFF, call, t0))) break;
            __builtin_amdgcn_s_sleep(2);
        }
        if (lane == 0) __hip_atomic_store(mb + NMB, 1u, __ATOMIC_RELAXED, __HIP_MEMORY_SCOPE_WORKGROUP);
        return;
    }
    WAIT_V(0);
    const int d = p.rank ^ dsel;
    char* dws = p.ws[d];
    const __amdgpu_buffer_rsrc_t rsw = __builtin_amdgcn_make_buffer_rsrc(dws, 0, POUT_OFF, 0x00020000);
    const __amdgpu_buffer_rsrc_t rso = __builtin_amdgcn_make_buffer_rsrc(p.out[d], 0, OUT_B, 0x00020000);
    const unsigned ol = (unsigned)(lane >> 5) * (DIM * 2) + (unsigned)(lane & 31) * 16;
    unsigned nout = 0, pend = 0; int pendkind = 3;
    int si = 1; (void)si;
    for (;;) {
        u32x4 v[16];
        piece_read(v, cur.kind, buf, lane);
        int code = next_task(mb, lane);
        Piece nxt = cur;
        if (code >= 0) nxt = task_piece(p, code, dsel);
        const char* ns = piece_src(nxt, lane);
        const bool o = cur.kind == 3;
        const __amdgpu_buffer_rsrc_t rs = o ? rso : rsw;
        const unsigned ds = o ? 2u * DIM * 2 : 1024u;
        unsigned vo = cur.dst + (o ? ol : (unsigned)lane * 16);
#pragma unroll
        for (int f = 0; f < 16; ++f) {
            __builtin_amdgcn_raw_buffer_store_b128(v[f], rs, vo, 0, AUX_SYS);
            vo += ds;
            asm volatile("" ::: "memory");
            if (f < 8) piece_dma2(nxt, ns, buf, f);
            asm volatile("s_waitcnt vmcnt(%0)" :: "i"(younger(f)) : "memory");
            if (f == (WIN > 2 ? WIN - 2 : 0) && pendkind != 3 && lane == 0) {
                if (pendkind == 2) st_sys(dws + pend, call); else add_sys(dws + pend, 1u);
            }
        }
        pend = cur.pub; pendkind = cur.kind; nout += o;
        if (si < 128) { STAMPV(si, cur.kind + 1); } ++si;
        if (code < 0) {
            WAIT_V(0);
            if (pendkind != 3 && lane == 0) { if (pendkind == 2) st_sys(dws + pend, call); else add_sys(dws + pend, 1u); }
            pendkind = 3;
            while (code == -1) { __builtin_amdgcn_s_sleep(4); code = next_task(mb, lane); }
            if (code < 0) break;
            nxt = task_piece(p, code, dsel); piece_dma(nxt, buf, lane);
            WAIT_V(0);
        }
        cur = nxt;
    }
    if (lane == 0 && nout) add_sys(dws + DONE_OFF + p.rank * 128, nout);
}

__device__ __forceinline__ int ready_prefix(unsigned v, unsigned target) {
    const unsigned long long m = __ballot((int)(v - target) >= 0);
    const unsigned both = (unsigned)m & (unsigned)(m >> 32);
    return both == 0xffffffffu ? 32 : __builtin_ctz(~both);
}
__device__ __noinline__ int wait_slot(const char* cptr, char* tmo, unsigned call, int need) {
    const unsigned long long t0 = __builtin_amdgcn_s_memrealtime();
    int known;
    for (unsigned polls = 1;; ++polls) {
        known = ready_prefix(ld_sys(cptr), call * 4u);
        if (known > need) break;
        if ((polls & 63u) == 0u && give_up(tmo, call, t0)) { known = 32; break; }
        __builtin_amdgcn_s_sleep(POLL_SLEEP);
    }
    return known;
}

__device__ __forceinline__ void gemm_wg(const Params& p, LAS char* lds, int gb) {
    const int tid = threadIdx.x, wid = __builtin_amdgcn_readfirstlane(tid >> 6), lane = tid & 63, wr = wid >> 2, wc = wid & 3;
    char* ws = p.ws[p.rank];
    STAMP(0);
    {
        const int q = wid * NGEMM + gb;
        if (q < 32 * NI * 4) {
            const Piece pc = input_piece(p, q, 0);
            LAS char* buf = lds + wid * HT_B;
            piece_dma(pc, buf, lane);
            WAIT_V(0);
            u32x4 v[16];
            piece_read(v, pc.kind, buf, lane);
            const __amdgpu_buffer_rsrc_t rs = __builtin_amdgcn_make_buffer_rsrc(ws, 0, STAGE_B, 0x00020000);
#pragma unroll
            for (int f = 0; f < 16; ++f) __builtin_amdgcn_raw_buffer_store_b128(v[f], rs, pc.dst + lane * 16 + f * 1024, 0, AUX_SYS);
            WAIT_V(0);
            if (lane == 0) add_sys(ws + pc.pub, 1u);
        }
        BAR;
    }
    STAMP(1);
    const unsigned call = __builtin_amdgcn_readfirstlane(ld_sys(ws + CALL_OFF)) + 1u;
    const unsigned target = call * 4u;
    const unsigned voff = (unsigned)tid * 16;
    const unsigned ldsw = (unsigned)wid * 1024u;
    const int aoff = wr * 8192 + lane * 16, boff = wc * 4096 + lane * 16;
#define SA(b, h) (((b) * 2 + (h)) * HT_B)
#define SB(b, h) ((4 + (b) * 2 + (h)) * HT_B)
#define STAGE(bufoff, gbase) do { _Pragma("unroll") for (int _i = 0; _i < 2; ++_i) \
        __builtin_amdgcn_global_load_lds((const GAS unsigned*)((const char*)(gbase) + voff + _i * 8192), (LAS unsigned*)(lds + (bufoff) + ldsw + _i * 8192), 16, 0, STAGE_AUX); } while (0)
#define LDA(dst, b, h) do { _Pragma("unroll") for (int m = 0; m < 4; ++m) _Pragma("unroll") for (int k = 0; k < 2; ++k) dst[m][k] = *(const LAS bf16x8*)(lds + SA(b, h) + aoff + m * 2048 + k * 1024); } while (0)
#define LDB(dst, b, h) do { _Pragma("unroll") for (int n = 0; n < 2; ++n) _Pragma("unroll") for (int k = 0; k < 2; ++k) dst[n][k] = *(const LAS bf16x8*)(lds + SB(b, h) + boff + n * 2048 + k * 1024); } while (0)
#define MMA(ai, bj, At, Bt) do { __builtin_amdgcn_s_setprio(1); _Pragma("unroll") for (int m = 0; m < 4; ++m) _Pragma("unroll") for (int n = 0; n < 2; ++n) _Pragma("unroll") for (int k = 0; k < 2; ++k) \
        acc[ai][bj][m][n] = __builtin_amdgcn_mfma_f32_16x16x32_bf16(Bt[n][k], At[m][k], acc[ai][bj][m][n], 0, 0, 0); __builtin_amdgcn_s_setprio(0); } while (0)
    for (int rnd = 0; rnd < 2; ++rnd) {
        const int code = (p.tiles[gb] >> (16 * rnd)) & 0xffff;
        if (code == 0xffff) break;
        const int pm = (code >> 4) & 15, pn = code & 15;
        const char* Ab = ws + (size_t)pm * PANEL_B;
        const char* Bb = ws + (size_t)(16 + pn) * PANEL_B;
        const unsigned coff = CNT_OFF + (unsigned)(lane < 32 ? pm : 16 + pn) * 128 + (unsigned)(lane & 31) * 4;
        f32x4 acc[2][2][4][2];
#pragma unroll
        for (int a = 0; a < 2; ++a)
#pragma unroll
            for (int b = 0; b < 2; ++b)
#pragma unroll
                for (int m = 0; m < 4; ++m)
#pragma unroll
                    for (int n = 0; n < 2; ++n) acc[a][b][m][n] = (f32x4){0.f, 0.f, 0.f, 0.f};
        bf16x8 At[4][2], B0[2][2], B1[2][2];
        int known = wait_slot(ws + coff, ws + TMO_OFF, call, 0);
        STAMP(2 + rnd * 12);
        STAGE(SB(0, 0), Bb); STAGE(SA(0, 0), Ab); STAGE(SB(0, 1), Bb + HT_B); STAGE(SA(0, 1), Ab + HT_B);
        if (wr == 1) BAR;
        WAIT_V(4); BAR;
        STAGE(SB(1, 0), Bb + KT_B); STAGE(SA(1, 0), Ab + KT_B); STAGE(SB(1, 1), Bb + KT_B + HT_B);
        WAIT_V(6); BAR;
        for (int T = 0; T < 32; ++T) {
            if (T < 31 && known <= T + 1) known = wait_slot(ws + coff, ws + TMO_OFF, call, T + 1);
            const unsigned o1 = (unsigned)T * SLOT_B, o2 = (unsigned)(T < 31 ? T + 1 : T) * SLOT_B;
            const char* a1 = Ab + o1 + KT_B; const char* a2 = Ab + o2; const char* b2 = Bb + o2;
            const char* a3 = a2 + KT_B; const char* b3 = b2 + KT_B;
            unsigned pv;
            LDB(B0, 0, 0); SCHED; LDA(At, 0, 0);
            asm volatile("global_load_dword %0, %1, %2 sc0 sc1" : "=v"(pv) : "v"(coff), "s"(ws) : "memory");
            STAGE(SA(1, 1), a1 + HT_B);
            WAIT_L(8); BAR; WAIT_L(0); MMA(0, 0, At, B0); BAR; SCHED;
            LDB(B1, 0, 1); STAGE(SB(0, 0), b2);
            BAR; WAIT_L(0); MMA(0, 1, At, B1); BAR;
            LDA(At, 0, 1); STAGE(SA(0, 0), a2);
            BAR; WAIT_L(0); MMA(1, 0, At, B0); BAR; SCHED;
            STAGE(SB(0, 1), b2 + HT_B);
            WAIT_V(6);
            asm volatile("" : "+v"(pv));
            { const int k2 = ready_prefix(pv, target); known = k2 > known ? k2 : known; }
            BAR; MMA(1, 1, At, B1); BAR;
            LDB(B0, 1, 0); SCHED; LDA(At, 1, 0); STAGE(SA(0, 1), a2 + HT_B);
            WAIT_L(8); BAR; WAIT_L(0); MMA(0, 0, At, B0); BAR; SCHED;
            LDB(B1, 1, 1); STAGE(SB(1, 0), b3);
            BAR; WAIT_L(0); MMA(0, 1, At, B1); BAR;
            LDA(At, 1, 1); STAGE(SA(1, 0), a3);
            BAR; WAIT_L(0); MMA(1, 0, At, B0); BAR; SCHED;
            STAGE(SB(1, 1), b3 + HT_B);
            WAIT_V(6); BAR; MMA(1, 1, At, B1); BAR;
        }
        WAIT_V(0);
        if (wr == 0) BAR;
        BAR;
        STAMP(11 + rnd * 12);
        int tz = threadIdx.x; asm volatile("" : "+v"(tz));
        const int ln = tz & 63;
        const int id = ((pm & 7) * 16 + pn) * 8 + wid;
        if (pm < 8) {
            const int j = (((pm & 1) * 2 + (pn & 1)) ^ p.rank) & 3;
            unsigned pos = 0;
            if (ln == 0) pos = __hip_atomic_fetch_add((GAS unsigned*)(ws + TAIL_OFF + j * 128), 1u, __ATOMIC_RELAXED, __HIP_MEMORY_SCOPE_AGENT);
            const bool bad = __builtin_amdgcn_readfirstlane(__hip_atomic_load((const GAS unsigned*)(ws + TMO_OFF), __ATOMIC_RELAXED, __HIP_MEMORY_SCOPE_AGENT)) == call;
            const __amdgpu_buffer_rsrc_t rp = __builtin_amdgcn_make_buffer_rsrc(ws + POUT_OFF, 0, 128 * TILE_B, 0x00020000);
            const unsigned base = (unsigned)id * HT_B + (unsigned)ln * 16;
#pragma unroll
            for (int ai = 0; ai < 2; ++ai)
#pragma unroll
                for (int m = 0; m < 4; ++m)
#pragma unroll
                    for (int bj = 0; bj < 2; ++bj) {
                        const f32x4 v0 = acc[ai][bj][m][0], v1 = acc[ai][bj][m][1];
                        u32x4 w; w.x = cvt_pk_bf16(v0[0], v0[1]); w.y = cvt_pk_bf16(v0[2], v0[3]); w.z = cvt_pk_bf16(v1[0], v1[1]); w.w = cvt_pk_bf16(v1[2], v1[3]);
                        if (bad) w = (u32x4){0x7fc07fc0u, 0x7fc07fc0u, 0x7fc07fc0u, 0x7fc07fc0u};
                        __builtin_amdgcn_raw_buffer_store_b128(w, rp, base + (unsigned)((ai * 4 + m) * 2 + bj) * 1024, 0, AUX_SYS);
                    }
            WAIT_V(0);
            pos &= 255u;
            if (ln == 0) st_sys(ws + PL_OFF + (j * 256 + pos) * 4, (call & 0xfffffu) << 12 | (unsigned)id);
            STAMP(12 + rnd * 12);
            continue;
        }
        {
            const unsigned long long t0 = __builtin_amdgcn_s_memrealtime();
            for (unsigned polls = 1;; ++polls) {
                if ((int)(__builtin_amdgcn_readfirstlane(ld_sys(ws + PF_OFF + id * 4)) - call) >= 0) break;
                if ((polls & 63u) == 0u && give_up(ws + TMO_OFF, call, t0)) break;
                __builtin_amdgcn_s_sleep(16);
            }
            asm volatile("" ::: "memory");
            STAMP(26);
            const bool bad = __builtin_amdgcn_readfirstlane(__hip_atomic_load((const GAS unsigned*)(ws + TMO_OFF), __ATOMIC_RELAXED, __HIP_MEMORY_SCOPE_AGENT)) == call;
            const __amdgpu_buffer_rsrc_t rp = __builtin_amdgcn_make_buffer_rsrc(ws + PIN_OFF, 0, 128 * TILE_B, 0x00020000);
            const unsigned base = (unsigned)id * HT_B + (unsigned)ln * 16;
            u32x4 pr[16];
#pragma unroll
            for (int f = 0; f < 16; ++f) pr[f] = __builtin_amdgcn_raw_buffer_load_b128(rp, base + f * 1024, 0, AUX_SYS);
            const int fr = ln & 15, fq = ln >> 4;
#pragma unroll
            for (int ai = 0; ai < 2; ++ai)
#pragma unroll
                for (int m = 0; m < 4; ++m) {
                    const int row = ai * 128 + wr * 64 + m * 16 + fr;
#pragma unroll
                    for (int bj = 0; bj < 2; ++bj) {
                        const u32x4 q = pr[(ai * 4 + m) * 2 + bj];
                        const f32x4 v0 = acc[ai][bj][m][0], v1 = acc[ai][bj][m][1];
                        u32x4 w;
                        w.x = cvt_pk_bf16(v0[0] + __uint_as_float(q.x << 16), v0[1] + __uint_as_float(q.x & 0xffff0000u));
                        w.y = cvt_pk_bf16(v0[2] + __uint_as_float(q.y << 16), v0[3] + __uint_as_float(q.y & 0xffff0000u));
                        w.z = cvt_pk_bf16(v1[0] + __uint_as_float(q.z << 16), v1[1] + __uint_as_float(q.z & 0xffff0000u));
                        w.w = cvt_pk_bf16(v1[2] + __uint_as_float(q.w << 16), v1[3] + __uint_as_float(q.w & 0xffff0000u));
                        if (bad) w = (u32x4){0x7fc07fc0u, 0x7fc07fc0u, 0x7fc07fc0u, 0x7fc07fc0u};
                        const int ch = (bj * 16 + wc * 4 + fq) ^ (fr & 7);
                        *(LAS u32x4*)(lds + row * 512 + ch * 16) = w;
                    }
                }
        }
        WAIT_L(0); BAR; asm volatile("" ::: "memory");
        {
            unsigned pos = 0;
            if (ln == 0) pos = __hip_atomic_fetch_add((GAS unsigned*)(ws + TAIL_OFF + 4 * 128), 1u, __ATOMIC_RELAXED, __HIP_MEMORY_SCOPE_AGENT);
            const __amdgpu_buffer_rsrc_t ro = __builtin_amdgcn_make_buffer_rsrc(p.out[p.rank], 0, OUT_B, 0x00020000);
            const unsigned vo = (unsigned)(panel_row(p.rank, pm) + wid * 32 + (ln >> 5)) * (DIM * 2) + (unsigned)panel_col(p.rank, pn) * 2 + (unsigned)(ln & 31) * 16;
            const int hi = ln >> 5, chn = ln & 31;
#pragma unroll
            for (int f = 0; f < 16; ++f) {
                const u32x4 v = *(const LAS u32x4*)(lds + (wid * 32 + 2 * f + hi) * 512 + ((chn ^ ((2 * f + hi) & 7)) * 16));
                __builtin_amdgcn_raw_buffer_store_b128(v, ro, vo + (unsigned)(2 * f) * (DIM * 2), 0, AUX_SYS);
            }
            WAIT_V(0);
            pos &= 1023u;
            if (ln == 0) {
                st_sys(ws + FL_OFF + pos * 4, (call & 0xfffffu) << 12 | (unsigned)id);
                add_sys(ws + DONE_OFF + p.rank * 128, 1u);
            }
            STAMP(27);
        }
        break;
    }
#undef SA
#undef SB
#undef STAGE
#undef LDA
#undef LDB
#undef MMA
}

__global__ __launch_bounds__(512, 2) void gemm_ar_kernel(const Params p) {
    extern __shared__ __attribute__((aligned(16))) char smem[];
    LAS char* lds = (LAS char*)smem;
    const int b = blockIdx.x;
    if (b >= NLINK) { gemm_wg(p, lds, b - NLINK); return; }
#define MAPMODE 2
    {
        const int lab = b & 7, e = b >> 3;
        const int dsel = (lab + e) & 7;
        if (dsel) { link_wg(p, lds, dsel, e); return; }
    }
    if (b == 0 && threadIdx.x < 64) {
        char* ws = p.ws[p.rank];
        const int lane = threadIdx.x;
        const unsigned call = __builtin_amdgcn_readfirstlane(ld_sys(ws + CALL_OFF)) + 1u;
        const unsigned long long t0 = __builtin_amdgcn_s_memrealtime();
        bool dead = false;
        for (unsigned polls = 1;; ++polls) {
            const unsigned v = ld_sys(ws + DONE_OFF + (lane & 7) * 128);
            if (__all((int)(v - call * 1024u) >= 0)) break;
            if ((polls & 63u) == 0u && give_up(ws + TMO_OFF, call, t0)) { dead = true; break; }
            __builtin_amdgcn_s_sleep(4);
        }
        __builtin_amdgcn_fence(__ATOMIC_ACQUIRE, "");
        if (lane == 0) {
            if (dead) st_sys(p.out[p.rank] + ((size_t)panel_row(p.rank, 8) * DIM + (size_t)panel_col(p.rank, 0)) * 2, 0x7fc07fc0u);
            st_sys(ws + CALL_OFF, call);
            STAMP(31);
        }
    }
}

void build_schedule(Params& p) {
#define BATCH_ORDER {0, 1, 4, 2, 5, 3, 6, 7}
    static const int base[8][6] = {{0, 1, 16, 17, 18, 19}, {2, 3, 20, 21, 22, 23}, {4, 5, 24, 25, 26, 27}, {6, 7, 28, 29, 30, 31},
                                   {8, 9, -1, -1, -1, -1}, {10, 11, -1, -1, -1, -1}, {12, 13, -1, -1, -1, -1}, {14, 15, -1, -1, -1, -1}};
    const int order[8] = BATCH_ORDER;
    int pos[32], ni = 0, nx = 32 * NI;
    unsigned char* un = (unsigned char*)p.units;
    unsigned short* tl = (unsigned short*)p.tiles;
    for (int b = 0; b < 8; ++b) {
        const int* pl = base[order[b]];
        for (int i = 0; i < 6; ++i) if (pl[i] >= 0) pos[pl[i]] = b;
        for (int c = 0; c < NI; ++c) for (int i = 0; i < 6; ++i) if (pl[i] >= 0) un[ni++] = (unsigned char)(pl[i] | c << 5);
        for (int c = 0; c < NX; ++c) for (int i = 0; i < 6; ++i) if (pl[i] >= 0) un[nx++] = (unsigned char)(pl[i] | c << 5);
    }
    for (int i = 0; i < NGEMM; ++i) p.tiles[i] = 0xffffffffu;
    for (int own = 0; own < 2; ++own) {
        int next = 0;
        for (int b = 0; b < 8; ++b) {
            int list[128], n = 0;
            for (int r = 0; r < 8; ++r) for (int c = 0; c < 16; ++c) {
                const int a = pos[own * 8 + r], bb = pos[16 + c];
                if ((a > bb ? a : bb) == b) list[n++] = (own * 8 + r) << 4 | c;
            }
            int k = 0;
            for (int x = 0; x < 8; ++x) for (int s = next; s < next + n; ++s) if ((s & 7) == x) {
                const int wg = own == 0 ? s : (s < NGEMM - 128 ? 128 + s : s - (NGEMM - 128)), rnd = (own == 1 && s >= NGEMM - 128) ? 1 : 0;
                tl[wg * 2 + rnd] = (unsigned short)list[k++];
            }
            next += n;
        }
    }
}
}

extern "C" void kernel_launch(int rank, int nranks, ncclComm_t comm,
                              hipStream_t stream,
                              void* const* d_in, const int* in_sizes, int n_in,
                              void* d_out, int out_size,
                              const int* in_dtypes, int out_dtype,
                              void* const* d_out_all,
                              void* const* d_ws_all, size_t ws_bytes) {
    (void)comm; (void)in_dtypes; (void)out_dtype; (void)d_out;
    if (nranks != NR || (unsigned)rank >= (unsigned)NR || n_in < 2 || in_sizes[0] != DIM * KL || in_sizes[1] != KL * DIM ||
        out_size != DIM * DIM || ws_bytes < WS_NEED) {
        fprintf(stderr, "kernel_launch: unexpected problem shape\n");
        return;
    }
    Params p;
    p.x = (const char*)d_in[0]; p.w = (const char*)d_in[1]; p.rank = rank;
    for (int r = 0; r < NR; ++r) { p.out[r] = (char*)d_out_all[r]; p.ws[r] = (char*)d_ws_all[r]; }
    build_schedule(p);
    HIP_CHECK(hipFuncSetAttribute((const void*)gemm_ar_kernel, hipFuncAttributeMaxDynamicSharedMemorySize, LDS_B));
    hipLaunchKernelGGL(gemm_ar_kernel, dim3(NLINK + NGEMM), dim3(512), LDS_B, stream, p);
    HIP_CHECK(hipGetLastError());
}
```

```cpp
#include <hip/hip_runtime.h>
#include <rccl/rccl.h>
#include <cstdio>

#define HIP_CHECK(expr) do { hipError_t _e = (expr); if (_e != hipSuccess) { \
    fprintf(stderr, "HIP error %s at %s:%d\n", hipGetErrorString(_e), __FILE__, __LINE__); \
    return; } } while (0)

#define LAS __attribute__((address_space(3)))
#define GAS __attribute__((address_space(1)))
typedef short bf16x8 __attribute__((ext_vector_type(8)));
typedef short s16x4 __attribute__((ext_vector_type(4)));
typedef float f32x4 __attribute__((ext_vector_type(4)));
typedef unsigned u32x4 __attribute__((ext_vector_type(4)));
typedef unsigned u32x2 __attribute__((ext_vector_type(2)));

namespace {
constexpr int NR = 8;
constexpr int DIM = 8192, KL = 1024;
constexpr int NI = 5, NX = 3;
constexpr unsigned KT_B = 32768, HT_B = 16384;
constexpr unsigned SLOT_B = 2 * KT_B;
constexpr unsigned PANEL_B = 32 * SLOT_B;
constexpr unsigned STAGE_B = 32 * PANEL_B;
constexpr unsigned TILE_B = 131072;
constexpr unsigned PIN_OFF = STAGE_B;
constexpr unsigned POUT_OFF = PIN_OFF + 128 * TILE_B;
constexpr unsigned DATA_B = POUT_OFF + 128 * TILE_B;
constexpr unsigned CNT_OFF = DATA_B;
constexpr unsigned PF_OFF = DATA_B + 4096;
constexpr unsigned FL_OFF = DATA_B + 8192;
constexpr unsigned PL_OFF = DATA_B + 12288;
constexpr unsigned DONE_OFF = DATA_B + 16384;
constexpr unsigned TAIL_OFF = DONE_OFF + 1024;
constexpr unsigned HEAD_OFF = DONE_OFF + 4096;
constexpr unsigned CALL_OFF = DONE_OFF + 2048;
constexpr unsigned TMO_OFF = CALL_OFF + 128;
constexpr size_t WS_NEED = (size_t)DATA_B + 32768;
constexpr unsigned OUT_B = (unsigned)DIM * DIM * 2;
#define LINK_WGS 8
#define SENDERS 2
constexpr int NE = LINK_WGS;
constexpr int NS = SENDERS;
constexpr int NLINK = 8 * NE, NGEMM = 256 - NLINK;
constexpr int LDS_B = 131072;
constexpr unsigned long long DEADLINE = 100000000ull;
constexpr int AUX_SYS = 17;

#define WIN 3
#define STAGE_AUX 16
#define DMA_GAP 3
#define POLL_SLEEP 64

struct Params {
    const char* x; const char* w;
    char* out[NR]; char* ws[NR];
    int rank;
    unsigned units[8 * (NI + NX)];
    unsigned tiles[NGEMM];
};

#define STR2(x) #x
#define STR(x) STR2(x)
#define WAIT_V(n) asm volatile("s_waitcnt vmcnt(" STR(n) ")" ::: "memory")
#define WAIT_L(n) asm volatile("s_waitcnt lgkmcnt(" STR(n) ")" ::: "memory")
#define BAR __builtin_amdgcn_s_barrier()
#define SCHED __builtin_amdgcn_sched_barrier(0)

#define STAMPV(i, v) do {} while (0)
#define STAMP(i) do {} while (0)

__device__ __forceinline__ unsigned ld_sys(const char* p) {
    return __hip_atomic_load((const GAS unsigned*)p, __ATOMIC_RELAXED, __HIP_MEMORY_SCOPE_SYSTEM);
}
__device__ __forceinline__ void st_sys(char* p, unsigned v) {
    __hip_atomic_store((GAS unsigned*)p, v, __ATOMIC_RELAXED, __HIP_MEMORY_SCOPE_SYSTEM);
}
__device__ __forceinline__ void st8_sys(char* p, unsigned v) {
    __hip_atomic_store((GAS unsigned char*)p, (unsigned char)v, __ATOMIC_RELAXED, __HIP_MEMORY_SCOPE_SYSTEM);
}
__device__ __forceinline__ unsigned call_tag(unsigned call) { return call % 255u + 1u; }
__device__ __forceinline__ void add_sys(char* p, unsigned v) {
    __hip_atomic_fetch_add((GAS unsigned*)p, v, __ATOMIC_RELAXED, __HIP_MEMORY_SCOPE_SYSTEM);
}
__device__ __forceinline__ unsigned cvt_pk_bf16(float lo, float hi) {
    unsigned r; asm volatile("v_cvt_pk_bf16_f32 %0, %1, %2" : "=v"(r) : "v"(lo), "v"(hi)); return r;
}
__device__ __forceinline__ int panel_row(int d, int P) {
    const int g = d >> 2, ai = (d >> 1) & 1, k = P & 7, u = k & 1, i = ((P >> 3) ? g : 1 - g) * 4 + (k >> 1);
    return (g == 0 ? ai * 2 + u : u * 2 + ai) * 2048 + i * 256;
}
__device__ __forceinline__ int panel_col(int d, int k) {
    const int g = d >> 2, aj = d & 1, v = k & 1;
    return (g == 0 ? aj * 2 + v : v * 2 + aj) * 2048 + (k >> 1) * 256;
}
__device__ __forceinline__ bool give_up(char* tmo, unsigned call, unsigned long long t0) {
    if (__builtin_amdgcn_readfirstlane(__hip_atomic_load((const GAS unsigned*)tmo, __ATOMIC_RELAXED, __HIP_MEMORY_SCOPE_AGENT)) == call
        || __builtin_amdgcn_s_memrealtime() - t0 > DEADLINE) {
        __hip_atomic_store((GAS unsigned*)tmo, call, __ATOMIC_RELAXED, __HIP_MEMORY_SCOPE_AGENT);
        return true;
    }
    return false;
}

struct Piece { const char* src; unsigned sA, sB, dst, pub; int kind; };

__device__ __forceinline__ Piece input_piece(const Params& p, int q, int dsel) {
    const int d = p.rank ^ dsel;
    const int ui = (q >> 2) + (dsel >= 4 ? 32 * NI : 0);
    const int u = (p.units[ui >> 2] >> ((ui & 3) * 8)) & 255;
    const int panel = u & 31, cc = u >> 5, pc = q & 3;
    const int c = dsel >= 4 ? NI + cc : cc;
    const int slot = dsel == 0 ? cc : (dsel >= 4 ? NI + cc * 4 + (dsel & 3) : NI + 4 * NX + cc * 3 + dsel - 1);
    const int k0 = c * 128 + (pc >> 1) * 64, h = pc & 1;
    Piece o; o.kind = panel >= 16;
    o.dst = (unsigned)panel * PANEL_B + (unsigned)slot * SLOT_B + (unsigned)pc * HT_B;
    o.pub = CNT_OFF + (unsigned)panel * 128 + (unsigned)slot * 4 + (unsigned)pc;
    if (o.kind) {
        o.src = p.w + ((size_t)k0 * DIM + panel_col(d, panel - 16) + h * 128) * 2;
        o.sA = 8u * DIM * 2; o.sB = 4u * DIM * 2;
    } else {
        o.src = p.x + ((size_t)(panel_row(d, panel) + h * 128) * KL + k0) * 2;
        o.sA = 16u * KL * 2; o.sB = 8u * KL * 2;
    }
    return o;
}
__device__ __forceinline__ Piece partial_piece(const Params& p, int id) {
    const int pm = (id >> 7) & 7, pn = (id >> 3) & 15, w = id & 7;
    const int ot = ((pm & 6) | ((p.rank >> 1) & 1)) * 16 + ((pn & 14) | (p.rank & 1));
    Piece o; o.kind = 2;
    o.src = p.ws[p.rank] + POUT_OFF + (size_t)(id & 1023) * HT_B;
    o.sA = 2048; o.sB = 1024;
    o.dst = PIN_OFF + (unsigned)(ot * 8 + w) * HT_B;
    o.pub = PF_OFF + (unsigned)(ot * 8 + w) * 4;
    return o;
}
__device__ __forceinline__ Piece output_piece(const Params& p, int id) {
    const int pm = (id >> 7) & 7, pn = (id >> 3) & 15, w = id & 7;
    Piece o; o.kind = 3;
    o.dst = (unsigned)(panel_row(p.rank, 8 + pm) + w * 32) * (DIM * 2) + (unsigned)panel_col(p.rank, pn) * 2;
    o.src = p.out[p.rank] + o.dst;
    o.sA = 4u * DIM * 2; o.sB = 2u * DIM * 2;
    o.pub = 0;
    return o;
}
__device__ __forceinline__ const char* piece_src(const Piece& pc, int lane) {
    const unsigned xl = (unsigned)(lane >> 3) * (KL * 2) + (unsigned)((lane & 7) ^ (lane >> 3)) * 16;
    const unsigned wl = (unsigned)(lane >> 4) * (DIM * 2) + (unsigned)((lane & 15) ^ ((lane >> 4) << 2)) * 16;
    const unsigned ol = (unsigned)(lane >> 5) * (DIM * 2) + (unsigned)(lane & 31) * 16;
    return pc.src + (pc.kind == 0 ? xl : pc.kind == 1 ? wl : pc.kind == 2 ? (unsigned)lane * 16 : ol);
}
__device__ __forceinline__ void piece_dma2(const Piece& pc, const char* s, LAS char* buf, int i) {
    __builtin_amdgcn_global_load_lds((const GAS unsigned*)(s + (size_t)i * pc.sA), (LAS unsigned*)(buf + i * 2048), 16, 0, 16);
    __builtin_amdgcn_global_load_lds((const GAS unsigned*)(s + (size_t)i * pc.sA + pc.sB), (LAS unsigned*)(buf + i * 2048 + 1024), 16, 0, 16);
}
__device__ __forceinline__ void piece_dma(const Piece& pc, LAS char* buf, int lane) {
    const char* s = piece_src(pc, lane);
#pragma unroll
    for (int i = 0; i < 8; ++i) piece_dma2(pc, s, buf, i);
}
__device__ __forceinline__ void piece_read_tr(u32x4 (&v)[16], LAS char* buf, int lane) {
    const int q = (lane >> 2) & 3;
    const unsigned trl = (unsigned)(8 * (lane >> 4) + q) * 256 + (unsigned)(lane & 3) * 16;
#pragma unroll
    for (int f = 0; f < 16; ++f) {
        const int nb = f >> 1, kb = f & 1;
        LAS char* a = buf + trl + (((nb >> 1) ^ q) * 64) + kb * 8192 + (nb & 1) * 8;
        const u32x2 lo = __builtin_bit_cast(u32x2, __builtin_amdgcn_ds_read_tr16_b64_v4i16((LAS s16x4*)a));
        const u32x2 hi = __builtin_bit_cast(u32x2, __builtin_amdgcn_ds_read_tr16_b64_v4i16((LAS s16x4*)(a + 1024)));
        v[f] = (u32x4){lo.x, lo.y, hi.x, hi.y};
    }
}
__device__ __forceinline__ void piece_read(u32x4 (&v)[16], bool isx, LAS char* buf, int lane) {
    const unsigned xb = (unsigned)(lane & 15) * 128 + (unsigned)((lane >> 4) ^ (lane & 7)) * 16;
    LAS char* b0 = buf + (isx ? xb : (unsigned)lane * 16);
    LAS char* b1 = buf + (isx ? xb ^ 64u : (unsigned)lane * 16 + 1024u);
#pragma unroll
    for (int f = 0; f < 16; ++f) asm volatile("ds_read_b128 %0, %1 offset:%2" : "=v"(v[f]) : "v"((f & 1) ? b1 : b0), "i"((f >> 1) * 2048) : "memory");
    asm volatile("s_waitcnt lgkmcnt(0)" : "+v"(v[0]), "+v"(v[1]), "+v"(v[2]), "+v"(v[3]), "+v"(v[4]), "+v"(v[5]), "+v"(v[6]), "+v"(v[7]),
                 "+v"(v[8]), "+v"(v[9]), "+v"(v[10]), "+v"(v[11]), "+v"(v[12]), "+v"(v[13]), "+v"(v[14]), "+v"(v[15]) :: "memory");
}

constexpr int KSL = 3;
constexpr int NMB = 3 * KSL;

__device__ __forceinline__ int next_task(LAS unsigned* mb, int lane) {
    for (;;) {
        unsigned v;
        asm volatile("ds_read_b32 %0, %1\n\ts_waitcnt lgkmcnt(0)" : "=&v"(v) : "v"(mb + (lane & 15)) : "memory");
        unsigned best = 0xffffffffu; int bl = 0;
#pragma unroll
        for (int l = 0; l < NMB; ++l) { const unsigned k = (unsigned)__builtin_amdgcn_readlane(v, l) - 1u; if (k < best) { best = k; bl = l; } }
        if (best == 0xffffffffu) return __builtin_amdgcn_readlane(v, NMB) ? -2 : -1;
        unsigned old = 0;
        if (lane == 0) asm volatile("ds_cmpst_rtn_b32 %0, %1, %2, %3\n\ts_waitcnt lgkmcnt(0)" : "=&v"(old) : "v"(mb + bl), "v"(best + 1u), "v"(0u) : "memory");
        if ((unsigned)__builtin_amdgcn_readfirstlane(old) == best + 1u) {
            const int cls = best >> 21;
            return cls << 12 | (cls == 1 ? (int)(best >> 10) & 2047 : (int)best & 1023);
        }
    }
}
__device__ __forceinline__ Piece task_piece(const Params& p, int code, int dsel) {
    const int cls = code >> 12, id = code & 4095;
    return cls == 1 ? input_piece(p, id, dsel) : cls == 0 ? partial_piece(p, id) : output_piece(p, id);
}

static_assert(WIN >= 1 && WIN <= 8 && NS >= 1 && NS <= 3, "window / senders");

__device__ __forceinline__ void link_wg(const Params& p, LAS char* lds, int dsel, int e) {
    const int wid = __builtin_amdgcn_readfirstlane(threadIdx.x >> 6), lane = threadIdx.x & 63;
    char* ws = p.ws[p.rank];
    LAS unsigned* mb = (LAS unsigned*)(lds + 7 * HT_B);
    const int sn = wid < NS ? wid : wid - NS;
    LAS char* buf = lds + sn * HT_B;
    LAS unsigned* slot = mb + 16 + sn * 4;
    STAMPV(0, dsel);
    Piece cur;
    if (wid >= NS && wid < 2 * NS) { cur = input_piece(p, sn * NE + e, dsel); piece_dma(cur, buf, lane); }
    else if (wid == 7) { mb[lane] = 0u; WAIT_L(0); }
    BAR; asm volatile("" ::: "memory");
    if (wid >= 2 * NS) {
        if (wid < 6) return;
        const unsigned call = __builtin_amdgcn_readfirstlane(ld_sys(ws + CALL_OFF)) + 1u;
        if (wid == 6) {
            char* dws = p.ws[p.rank ^ dsel];
            bool live = lane < NS;
            while (__any(live)) {
                const unsigned v = live ? __hip_atomic_load(mb + 32 + lane, __ATOMIC_RELAXED, __HIP_MEMORY_SCOPE_WORKGROUP) : 0u;
                if (v) {
                    __hip_atomic_store(mb + 32 + lane, 0u, __ATOMIC_RELAXED, __HIP_MEMORY_SCOPE_WORKGROUP);
                    if (v == 3u) live = false;
                    else if (v >= PF_OFF) st_sys(dws + v, call);
                    else st8_sys(dws + v, call_tag(call));
                }
                __builtin_amdgcn_s_sleep(1);
            }
            return;
        }
        const int cls = lane < KSL ? 0 : lane < 2 * KSL ? 1 : 2;
        const unsigned first = cls == 1 ? NS * NE : 0u;
        const unsigned n = cls == 0 ? 256u : cls == 2 ? 1024u : dsel >= 4 ? 32u * NX * 4 : 32u * NI * 4;
        const unsigned base = (call - 1u) * (n - first + NE * KSL) - first;
        char* head = ws + HEAD_OFF + (dsel * 4 + cls) * 128;
        const char* list = ws + (cls == 0 ? PL_OFF + (dsel & 3) * 1024 : FL_OFF);
        const unsigned tag = call & 0xfffffu;
        int st = (lane >= NMB || (cls == 0 && dsel < 4)) ? 3 : 0;
        unsigned pos = 0;
        const unsigned long long t0 = __builtin_amdgcn_s_memrealtime();
        for (unsigned polls = 1;; ++polls) {
            if (st == 2 && __hip_atomic_load(mb + lane, __ATOMIC_RELAXED, __HIP_MEMORY_SCOPE_WORKGROUP) == 0u) st = 0;
            unsigned t = 0, en = 0;
            if (st == 0) t = __hip_atomic_fetch_add((GAS unsigned*)head, 1u, __ATOMIC_RELAXED, __HIP_MEMORY_SCOPE_AGENT);
            if (st == 1) en = ld_sys(list + pos * 4);
            bool post = false;
            if (st == 0) { pos = t - base; st = pos >= n ? 3 : 1; post = st == 1 && cls == 1; }
            else if (st == 1) post = (en >> 12) == tag;
            if (post) {
                __hip_atomic_store(mb + lane, ((((unsigned)cls << 11 | pos) << 10) | (en & 1023u)) + 1u, __ATOMIC_RELAXED, __HIP_MEMORY_SCOPE_WORKGROUP);
                st = 2;
            }
            if (__all(st == 3) || ((polls & 63u) == 0u && give_up(ws + TMO_OFF, call, t0))) break;
            __builtin_amdgcn_s_sleep(2);
        }
        if (lane == 0) __hip_atomic_store(mb + NMB, 1u, __ATOMIC_RELAXED, __HIP_MEMORY_SCOPE_WORKGROUP);
        return;
    }
    if (wid >= NS) {
        for (;;) {
            WAIT_V(0);
            if (cur.kind == 1) {
                u32x4 v[16];
                piece_read_tr(v, buf, lane);
                WAIT_L(0);
#pragma unroll
                for (int f = 0; f < 16; ++f) asm volatile("" : "+v"(v[f]));
#pragma unroll
                for (int f = 0; f < 16; ++f) *(LAS u32x4*)(buf + f * 1024 + lane * 16) = v[f];
                WAIT_L(0);
            }
            if (lane == 0) asm volatile("ds_write_b128 %0, %1" :: "v"(slot), "v"((u32x4){(unsigned)cur.kind + 1u, cur.dst, cur.pub, 0u}) : "memory");
            for (;;) {
                unsigned f;
                asm volatile("ds_read_b32 %0, %1\n\ts_waitcnt lgkmcnt(0)" : "=&v"(f) : "v"(slot) : "memory");
                if (__builtin_amdgcn_readfirstlane(f) == 0) break;
                __builtin_amdgcn_s_sleep(2);
            }
            int code = next_task(mb, lane);
            while (code == -1) { __builtin_amdgcn_s_sleep(2); code = next_task(mb, lane); }
            if (code < 0) break;
            cur = task_piece(p, code, dsel);
            {
                const char* sp = piece_src(cur, lane);
#pragma nounroll
                for (int i = 0; i < 16; ++i) {
                    __builtin_amdgcn_global_load_lds((const GAS unsigned*)(sp + (size_t)i * cur.sB), (LAS unsigned*)(buf + i * 1024), 16, 0, 16);
                    __builtin_amdgcn_s_sleep(DMA_GAP);
                }
            }
        }
        if (lane == 0) asm volatile("ds_write_b32 %0, %1" :: "v"(slot), "v"(15u) : "memory");
        return;
    }
    LAS unsigned* pq = mb + 32 + sn;
    auto post = [&](unsigned v) {
        for (;;) {
            unsigned f;
            asm volatile("ds_read_b32 %0, %1\n\ts_waitcnt lgkmcnt(0)" : "=&v"(f) : "v"(pq) : "memory");
            if (__builtin_amdgcn_readfirstlane(f) == 0) break;
            __builtin_amdgcn_s_sleep(1);
        }
        if (lane == 0) asm volatile("ds_write_b32 %0, %1" :: "v"(pq), "v"(v) : "memory");
    };
    const int d = p.rank ^ dsel;
    char* dws = p.ws[d];
    const __amdgpu_buffer_rsrc_t rsw = __builtin_amdgcn_make_buffer_rsrc(dws, 0, POUT_OFF, 0x00020000);
    const __amdgpu_buffer_rsrc_t rso = __builtin_amdgcn_make_buffer_rsrc(p.out[d], 0, OUT_B, 0x00020000);
    const unsigned ol = (unsigned)(lane >> 5) * (DIM * 2) + (unsigned)(lane & 31) * 16;
    unsigned nout = 0, pend = 0;
    int si = 1, waited = 0; (void)si; (void)waited;
    for (;;) {
        u32x4 ds4;
        asm volatile("ds_read_b128 %0, %1\n\ts_waitcnt lgkmcnt(0)" : "=&v"(ds4) : "v"(slot) : "memory");
        const int kind = __builtin_amdgcn_readfirstlane(ds4.x) - 1;
        if (kind < 0 || kind > 3) {
            if (pend) { WAIT_V(0); post(pend); pend = 0; }
            if (kind > 3) break;
            waited = 8;
            __builtin_amdgcn_s_sleep(1);
            continue;
        }
        const unsigned dst = __builtin_amdgcn_readfirstlane(ds4.y), pub = __builtin_amdgcn_readfirstlane(ds4.z);
        u32x4 v[16];
        piece_read(v, kind == 0, buf, lane);
        if (lane == 0) asm volatile("ds_write_b32 %0, %1" :: "v"(slot), "v"(0u) : "memory");
        const bool o = kind == 3;
        const __amdgpu_buffer_rsrc_t rs = o ? rso : rsw;
        const unsigned ds = o ? 2u * DIM * 2 : 1024u;
        unsigned vo = dst + (o ? ol : (unsigned)lane * 16);
#pragma unroll
        for (int f = 0; f < 16; ++f) {
            __builtin_amdgcn_raw_buffer_store_b128(v[f], rs, vo, 0, AUX_SYS);
            vo += ds;
            WAIT_V(WIN - 1);
            if (f == (WIN > 2 ? WIN - 2 : 0) && pend) post(pend);
        }
        pend = pub; nout += o;
        if (si < 128) { STAMPV(si, (kind + 1) | waited); } ++si; waited = 0;
    }
    WAIT_V(0);
    if (lane == 0 && nout) add_sys(dws + DONE_OFF + p.rank * 128, nout);
    post(3u);
}

__device__ __forceinline__ int ready_prefix(unsigned v, unsigned target) {
    const unsigned long long m = __ballot(v == target);
    const unsigned both = (unsigned)m & (unsigned)(m >> 32);
    return both == 0xffffffffu ? 32 : __builtin_ctz(~both);
}
__device__ __noinline__ int wait_slot(const char* cptr, char* tmo, unsigned call, int need) {
    const unsigned long long t0 = __builtin_amdgcn_s_memrealtime();
    int known;
    for (unsigned polls = 1;; ++polls) {
        known = ready_prefix(ld_sys(cptr), call_tag(call) * 0x01010101u);
        if (known > need) break;
        if ((polls & 63u) == 0u && give_up(tmo, call, t0)) { known = 32; break; }
        __builtin_amdgcn_s_sleep(POLL_SLEEP);
    }
    return known;
}

__device__ __forceinline__ void gemm_wg(const Params& p, LAS char* lds, int gb) {
    const int tid = threadIdx.x, wid = __builtin_amdgcn_readfirstlane(tid >> 6), lane = tid & 63, wr = wid >> 2, wc = wid & 3;
    char* ws = p.ws[p.rank];
    STAMP(0);
    const unsigned call = __builtin_amdgcn_readfirstlane(ld_sys(ws + CALL_OFF)) + 1u;
    {
        const int q = wid * NGEMM + gb;
        if (q < 32 * NI * 4) {
            const Piece pc = input_piece(p, q, 0);
            LAS char* buf = lds + wid * HT_B;
            piece_dma(pc, buf, lane);
            WAIT_V(0);
            u32x4 v[16];
            if (pc.kind) piece_read_tr(v, buf, lane); else piece_read(v, true, buf, lane);
            const __amdgpu_buffer_rsrc_t rs = __builtin_amdgcn_make_buffer_rsrc(ws, 0, STAGE_B, 0x00020000);
#pragma unroll
            for (int f = 0; f < 16; ++f) __builtin_amdgcn_raw_buffer_store_b128(v[f], rs, pc.dst + lane * 16 + f * 1024, 0, AUX_SYS);
            WAIT_V(0);
            if (lane == 0) st8_sys(ws + pc.pub, call_tag(call));
        }
        BAR;
    }
    STAMP(1);
    const unsigned target = call_tag(call) * 0x01010101u;
    const unsigned voff = (unsigned)tid * 16;
    const unsigned ldsw = (unsigned)wid * 1024u;
    const int aoff = wr * 8192 + lane * 16, boff = wc * 4096 + lane * 16;
#define SA(b, h) (((b) * 2 + (h)) * HT_B)
#define SB(b, h) ((4 + (b) * 2 + (h)) * HT_B)
#define STAGE(bufoff, gbase) do { _Pragma("unroll") for (int _i = 0; _i < 2; ++_i) \
        __builtin_amdgcn_global_load_lds((const GAS unsigned*)((const char*)(gbase) + voff + _i * 8192), (LAS unsigned*)(lds + (bufoff) + ldsw + _i * 8192), 16, 0, STAGE_AUX); } while (0)
#define LDA(dst, b, h) do { _Pragma("unroll") for (int m = 0; m < 4; ++m) _Pragma("unroll") for (int k = 0; k < 2; ++k) dst[m][k] = *(const LAS bf16x8*)(lds + SA(b, h) + aoff + m * 2048 + k * 1024); } while (0)
#define LDB(dst, b, h) do { _Pragma("unroll") for (int n = 0; n < 2; ++n) _Pragma("unroll") for (int k = 0; k < 2; ++k) dst[n][k] = *(const LAS bf16x8*)(lds + SB(b, h) + boff + n * 2048 + k * 1024); } while (0)
#define MMA(ai, bj, At, Bt) do { __builtin_amdgcn_s_setprio(1); _Pragma("unroll") for (int m = 0; m < 4; ++m) _Pragma("unroll") for (int n = 0; n < 2; ++n) _Pragma("unroll") for (int k = 0; k < 2; ++k) \
        acc[ai][bj][m][n] = __builtin_amdgcn_mfma_f32_16x16x32_bf16(Bt[n][k], At[m][k], acc[ai][bj][m][n], 0, 0, 0); __builtin_amdgcn_s_setprio(0); } while (0)
    for (int rnd = 0; rnd < 2; ++rnd) {
        const int code = (p.tiles[gb] >> (16 * rnd)) & 0xffff;
        if (code == 0xffff) break;
        const int pm = (code >> 4) & 15, pn = code & 15;
        const char* Ab = ws + (size_t)pm * PANEL_B;
        const char* Bb = ws + (size_t)(16 + pn) * PANEL_B;
        const unsigned coff = CNT_OFF + (unsigned)(lane < 32 ? pm : 16 + pn) * 128 + (unsigned)(lane & 31) * 4;
        f32x4 acc[2][2][4][2];
#pragma unroll
        for (int a = 0; a < 2; ++a)
#pragma unroll
            for (int b = 0; b < 2; ++b)
#pragma unroll
                for (int m = 0; m < 4; ++m)
#pragma unroll
                    for (int n = 0; n < 2; ++n) acc[a][b][m][n] = (f32x4){0.f, 0.f, 0.f, 0.f};
        bf16x8 At[4][2], B0[2][2], B1[2][2];
        int known = wait_slot(ws + coff, ws + TMO_OFF, call, 0);
        STAMP(2 + rnd * 12);
        STAGE(SB(0, 0), Bb); STAGE(SA(0, 0), Ab); STAGE(SB(0, 1), Bb + HT_B); STAGE(SA(0, 1), Ab + HT_B);
        if (wr == 1) BAR;
        WAIT_V(4); BAR;
        STAGE(SB(1, 0), Bb + KT_B); STAGE(SA(1, 0), Ab + KT_B); STAGE(SB(1, 1), Bb + KT_B + HT_B);
        WAIT_V(6); BAR;
        for (int T = 0; T < 32; ++T) {
            if (T < 31 && known <= T + 1) known = wait_slot(ws + coff, ws + TMO_OFF, call, T + 1);
            const unsigned o1 = (unsigned)T * SLOT_B, o2 = (unsigned)(T < 31 ? T + 1 : T) * SLOT_B;
            const char* a1 = Ab + o1 + KT_B; const char* a2 = Ab + o2; const char* b2 = Bb + o2;
            const char* a3 = a2 + KT_B; const char* b3 = b2 + KT_B;
            unsigned pv;
            LDB(B0, 0, 0); SCHED; LDA(At, 0, 0);
            asm volatile("global_load_dword %0, %1, %2 sc0 sc1" : "=v"(pv) : "v"(coff), "s"(ws) : "memory");
            STAGE(SA(1, 1), a1 + HT_B);
            WAIT_L(8); BAR; WAIT_L(0); MMA(0, 0, At, B0); BAR; SCHED;
            LDB(B1, 0, 1); STAGE(SB(0, 0), b2);
            BAR; WAIT_L(0); MMA(0, 1, At, B1); BAR;
            LDA(At, 0, 1); STAGE(SA(0, 0), a2);
            BAR; WAIT_L(0); MMA(1, 0, At, B0); BAR; SCHED;
            STAGE(SB(0, 1), b2 + HT_B);
            WAIT_V(6);
            asm volatile("" : "+v"(pv));
            { const int k2 = ready_prefix(pv, target); known = k2 > known ? k2 : known; }
            BAR; MMA(1, 1, At, B1); BAR;
            LDB(B0, 1, 0); SCHED; LDA(At, 1, 0); STAGE(SA(0, 1), a2 + HT_B);
            WAIT_L(8); BAR; WAIT_L(0); MMA(0, 0, At, B0); BAR; SCHED;
            LDB(B1, 1, 1); STAGE(SB(1, 0), b3);
            BAR; WAIT_L(0); MMA(0, 1, At, B1); BAR;
            LDA(At, 1, 1); STAGE(SA(1, 0), a3);
            BAR; WAIT_L(0); MMA(1, 0, At, B0); BAR; SCHED;
            STAGE(SB(1, 1), b3 + HT_B);
            WAIT_V(6); BAR; MMA(1, 1, At, B1); BAR;
        }
        WAIT_V(0);
        if (wr == 0) BAR;
        BAR;
        STAMP(11 + rnd * 12);
        int tz = threadIdx.x; asm volatile("" : "+v"(tz));
        const int ln = tz & 63;
        const int id = ((pm & 7) * 16 + pn) * 8 + wid;
        if (pm < 8) {
            const int j = (((pm & 1) * 2 + (pn & 1)) ^ p.rank) & 3;
            unsigned pos = 0;
            if (ln == 0) pos = __hip_atomic_fetch_add((GAS unsigned*)(ws + TAIL_OFF + j * 128), 1u, __ATOMIC_RELAXED, __HIP_MEMORY_SCOPE_AGENT);
            const bool bad = __builtin_amdgcn_readfirstlane(__hip_atomic_load((const GAS unsigned*)(ws + TMO_OFF), __ATOMIC_RELAXED, __HIP_MEMORY_SCOPE_AGENT)) == call;
            const __amdgpu_buffer_rsrc_t rp = __builtin_amdgcn_make_buffer_rsrc(ws + POUT_OFF, 0, 128 * TILE_B, 0x00020000);
            const unsigned base = (unsigned)id * HT_B + (unsigned)ln * 16;
#pragma unroll
            for (int ai = 0; ai < 2; ++ai)
#pragma unroll
                for (int m = 0; m < 4; ++m)
#pragma unroll
                    for (int bj = 0; bj < 2; ++bj) {
                        const f32x4 v0 = acc[ai][bj][m][0], v1 = acc[ai][bj][m][1];
                        u32x4 w; w.x = cvt_pk_bf16(v0[0], v0[1]); w.y = cvt_pk_bf16(v0[2], v0[3]); w.z = cvt_pk_bf16(v1[0], v1[1]); w.w = cvt_pk_bf16(v1[2], v1[3]);
                        if (bad) w = (u32x4){0x7fc07fc0u, 0x7fc07fc0u, 0x7fc07fc0u, 0x7fc07fc0u};
                        __builtin_amdgcn_raw_buffer_store_b128(w, rp, base + (unsigned)((ai * 4 + m) * 2 + bj) * 1024, 0, AUX_SYS);
                    }
            WAIT_V(0);
            pos &= 255u;
            if (ln == 0) st_sys(ws + PL_OFF + (j * 256 + pos) * 4, (call & 0xfffffu) << 12 | (unsigned)id);
            STAMP(12 + rnd * 12);
            continue;
        }
        {
            const unsigned long long t0 = __builtin_amdgcn_s_memrealtime();
            for (unsigned polls = 1;; ++polls) {
                if ((int)(__builtin_amdgcn_readfirstlane(ld_sys(ws + PF_OFF + id * 4)) - call) >= 0) break;
                if ((polls & 63u) == 0u && give_up(ws + TMO_OFF, call, t0)) break;
                __builtin_amdgcn_s_sleep(16);
            }
            asm volatile("" ::: "memory");
            STAMP(26);
            const bool bad = __builtin_amdgcn_readfirstlane(__hip_atomic_load((const GAS unsigned*)(ws + TMO_OFF), __ATOMIC_RELAXED, __HIP_MEMORY_SCOPE_AGENT)) == call;
            const __amdgpu_buffer_rsrc_t rp = __builtin_amdgcn_make_buffer_rsrc(ws + PIN_OFF, 0, 128 * TILE_B, 0x00020000);
            const unsigned base = (unsigned)id * HT_B + (unsigned)ln * 16;
            u32x4 pr[16];
#pragma unroll
            for (int f = 0; f < 16; ++f) pr[f] = __builtin_amdgcn_raw_buffer_load_b128(rp, base + f * 1024, 0, AUX_SYS);
            const int fr = ln & 15, fq = ln >> 4;
#pragma unroll
            for (int ai = 0; ai < 2; ++ai)
#pragma unroll
                for (int m = 0; m < 4; ++m) {
                    const int row = ai * 128 + wr * 64 + m * 16 + fr;
#pragma unroll
                    for (int bj = 0; bj < 2; ++bj) {
                        const u32x4 q = pr[(ai * 4 + m) * 2 + bj];
                        const f32x4 v0 = acc[ai][bj][m][0], v1 = acc[ai][bj][m][1];
                        u32x4 w;
                        w.x = cvt_pk_bf16(v0[0] + __uint_as_float(q.x << 16), v0[1] + __uint_as_float(q.x & 0xffff0000u));
                        w.y = cvt_pk_bf16(v0[2] + __uint_as_float(q.y << 16), v0[3] + __uint_as_float(q.y & 0xffff0000u));
                        w.z = cvt_pk_bf16(v1[0] + __uint_as_float(q.z << 16), v1[1] + __uint_as_float(q.z & 0xffff0000u));
                        w.w = cvt_pk_bf16(v1[2] + __uint_as_float(q.w << 16), v1[3] + __uint_as_float(q.w & 0xffff0000u));
                        if (bad) w = (u32x4){0x7fc07fc0u, 0x7fc07fc0u, 0x7fc07fc0u, 0x7fc07fc0u};
                        const int ch = (bj * 16 + wc * 4 + fq) ^ (fr & 7);
                        *(LAS u32x4*)(lds + row * 512 + ch * 16) = w;
                    }
                }
        }
        WAIT_L(0); BAR; asm volatile("" ::: "memory");
        {
            unsigned pos = 0;
            if (ln == 0) pos = __hip_atomic_fetch_add((GAS unsigned*)(ws + TAIL_OFF + 4 * 128), 1u, __ATOMIC_RELAXED, __HIP_MEMORY_SCOPE_AGENT);
            const __amdgpu_buffer_rsrc_t ro = __builtin_amdgcn_make_buffer_rsrc(p.out[p.rank], 0, OUT_B, 0x00020000);
            const unsigned vo = (unsigned)(panel_row(p.rank, pm) + wid * 32 + (ln >> 5)) * (DIM * 2) + (unsigned)panel_col(p.rank, pn) * 2 + (unsigned)(ln & 31) * 16;
            const int hi = ln >> 5, chn = ln & 31;
#pragma unroll
            for (int f = 0; f < 16; ++f) {
                const u32x4 v = *(const LAS u32x4*)(lds + (wid * 32 + 2 * f + hi) * 512 + ((chn ^ ((2 * f + hi) & 7)) * 16));
                __builtin_amdgcn_raw_buffer_store_b128(v, ro, vo + (unsigned)(2 * f) * (DIM * 2), 0, AUX_SYS);
            }
            WAIT_V(0);
            pos &= 1023u;
            if (ln == 0) {
                st_sys(ws + FL_OFF + pos * 4, (call & 0xfffffu) << 12 | (unsigned)id);
                add_sys(ws + DONE_OFF + p.rank * 128, 1u);
            }
            STAMP(27);
        }
        break;
    }
#undef SA
#undef SB
#undef STAGE
#undef LDA
#undef LDB
#undef MMA
}

__global__ __launch_bounds__(512, 2) void gemm_ar_kernel(const Params p) {
    extern __shared__ __attribute__((aligned(16))) char smem[];
    LAS char* lds = (LAS char*)smem;
    const int b = blockIdx.x;
    if (b >= NLINK) { gemm_wg(p, lds, b - NLINK); return; }
#define MAPMODE 2
    {
        const int lab = b & 7, e = b >> 3;
        const int dsel = (lab + e) & 7;
        if (dsel) { link_wg(p, lds, dsel, e); return; }
    }
    if (b == 0 && threadIdx.x < 64) {
        char* ws = p.ws[p.rank];
        const int lane = threadIdx.x;
        const unsigned call = __builtin_amdgcn_readfirstlane(ld_sys(ws + CALL_OFF)) + 1u;
        const unsigned long long t0 = __builtin_amdgcn_s_memrealtime();
        bool dead = false;
        for (unsigned polls = 1;; ++polls) {
            const unsigned v = ld_sys(ws + DONE_OFF + (lane & 7) * 128);
            if (__all((int)(v - call * 1024u) >= 0)) break;
            if ((polls & 63u) == 0u && give_up(ws + TMO_OFF, call, t0)) { dead = true; break; }
            __builtin_amdgcn_s_sleep(4);
        }
        __builtin_amdgcn_fence(__ATOMIC_ACQUIRE, "");
        if (lane == 0) {
            if (dead) st_sys(p.out[p.rank] + ((size_t)panel_row(p.rank, 8) * DIM + (size_t)panel_col(p.rank, 0)) * 2, 0x7fc07fc0u);
            st_sys(ws + CALL_OFF, call);
            STAMP(31);
        }
    }
}

void build_schedule(Params& p) {
#define BATCH_ORDER {0, 1, 4, 2, 5, 3, 6, 7}
    static const int base[8][6] = {{0, 1, 16, 17, 18, 19}, {2, 3, 20, 21, 22, 23}, {4, 5, 24, 25, 26, 27}, {6, 7, 28, 29, 30, 31},
                                   {8, 9, -1, -1, -1, -1}, {10, 11, -1, -1, -1, -1}, {12, 13, -1, -1, -1, -1}, {14, 15, -1, -1, -1, -1}};
    const int order[8] = BATCH_ORDER;
    int pos[32], ni = 0, nx = 32 * NI;
    unsigned char* un = (unsigned char*)p.units;
    unsigned short* tl = (unsigned short*)p.tiles;
    for (int b = 0; b < 8; ++b) {
        const int* pl = base[order[b]];
        for (int i = 0; i < 6; ++i) if (pl[i] >= 0) pos[pl[i]] = b;
        for (int c = 0; c < NI; ++c) for (int i = 0; i < 6; ++i) if (pl[i] >= 0) un[ni++] = (unsigned char)(pl[i] | c << 5);
        for (int c = 0; c < NX; ++c) for (int i = 0; i < 6; ++i) if (pl[i] >= 0) un[nx++] = (unsigned char)(pl[i] | c << 5);
    }
    for (int i = 0; i < NGEMM; ++i) p.tiles[i] = 0xffffffffu;
    for (int own = 0; own < 2; ++own) {
        int next = 0;
        for (int b = 0; b < 8; ++b) {
            int list[128], n = 0;
            for (int r = 0; r < 8; ++r) for (int c = 0; c < 16; ++c) {
                const int a = pos[own * 8 + r], bb = pos[16 + c];
                if ((a > bb ? a : bb) == b) list[n++] = (own * 8 + r) << 4 | c;
            }
            int k = 0;
            for (int x = 0; x < 8; ++x) for (int s = next; s < next + n; ++s) if ((s & 7) == x) {
                const int wg = own == 0 ? s : (s < NGEMM - 128 ? 128 + s : s - (NGEMM - 128)), rnd = (own == 1 && s >= NGEMM - 128) ? 1 : 0;
                tl[wg * 2 + rnd] = (unsigned short)list[k++];
            }
            next += n;
        }
    }
}
}

extern "C" void kernel_launch(int rank, int nranks, ncclComm_t comm,
                              hipStream_t stream,
                              void* const* d_in, const int* in_sizes, int n_in,
                              void* d_out, int out_size,
                              const int* in_dtypes, int out_dtype,
                              void* const* d_out_all,
                              void* const* d_ws_all, size_t ws_bytes) {
    (void)comm; (void)in_dtypes; (void)out_dtype; (void)d_out;
    if (nranks != NR || (unsigned)rank >= (unsigned)NR || n_in < 2 || in_sizes[0] != DIM * KL || in_sizes[1] != KL * DIM ||
        out_size != DIM * DIM || ws_bytes < WS_NEED) {
        fprintf(stderr, "kernel_launch: unexpected problem shape\n");
        return;
    }
    Params p;
    p.x = (const char*)d_in[0]; p.w = (const char*)d_in[1]; p.rank = rank;
    for (int r = 0; r < NR; ++r) { p.out[r] = (char*)d_out_all[r]; p.ws[r] = (char*)d_ws_all[r]; }
    build_schedule(p);
    HIP_CHECK(hipFuncSetAttribute((const void*)gemm_ar_kernel, hipFuncAttributeMaxDynamicSharedMemorySize, LDS_B));
    hipLaunchKernelGGL(gemm_ar_kernel, dim3(NLINK + NGEMM), dim3(512), LDS_B, stream, p);
    HIP_CHECK(hipGetLastError());
}
```

```cpp
#include <hip/hip_runtime.h>
#include <rccl/rccl.h>
#include <cstdio>

#define HIP_CHECK(expr) do { hipError_t _e = (expr); if (_e != hipSuccess) { \
    fprintf(stderr, "HIP error %s at %s:%d\n", hipGetErrorString(_e), __FILE__, __LINE__); \
    return; } } while (0)

#define LAS __attribute__((address_space(3)))
#define GAS __attribute__((address_space(1)))
typedef short bf16x8 __attribute__((ext_vector_type(8)));
typedef short s16x4 __attribute__((ext_vector_type(4)));
typedef float f32x4 __attribute__((ext_vector_type(4)));
typedef unsigned u32x4 __attribute__((ext_vector_type(4)));
typedef unsigned u32x2 __attribute__((ext_vector_type(2)));

namespace {
constexpr int NR = 8;
constexpr int DIM = 8192, KL = 1024;
constexpr int NI = 5, NX = 3;
constexpr unsigned KT_B = 32768, HT_B = 16384;
constexpr unsigned SLOT_B = 2 * KT_B;
constexpr unsigned PANEL_B = 32 * SLOT_B;
constexpr unsigned STAGE_B = 32 * PANEL_B;
constexpr unsigned TILE_B = 131072;
constexpr unsigned PIN_OFF = STAGE_B;
constexpr unsigned POUT_OFF = PIN_OFF + 128 * TILE_B;
constexpr unsigned SEND_OFF = POUT_OFF + 128 * TILE_B;
constexpr unsigned DATA_B = SEND_OFF + 2048 * HT_B;
constexpr unsigned CNT_OFF = DATA_B;
constexpr unsigned PF_OFF = DATA_B + 4096;
constexpr unsigned IR_OFF = DATA_B + 8192;
constexpr unsigned PL_OFF = DATA_B + 16384;
constexpr unsigned FL_OFF = DATA_B + 32768;
constexpr unsigned DONE_OFF = DATA_B + 65536;
constexpr unsigned TAIL_OFF = DONE_OFF + 1024;
constexpr unsigned CALL_OFF = DONE_OFF + 2048;
constexpr unsigned TMO_OFF = CALL_OFF + 128;
constexpr unsigned HEAD_OFF = DONE_OFF + 4096;
constexpr size_t WS_NEED = (size_t)DONE_OFF + 8192;
constexpr unsigned OUT_B = (unsigned)DIM * DIM * 2;
#define LINK_WGS 8
#define SENDERS 2
constexpr int NE = LINK_WGS;
constexpr int NS = SENDERS;
constexpr int NLINK = 8 * NE, NGEMM = 256 - NLINK;
constexpr int LDS_B = 131072;
constexpr unsigned long long DEADLINE = 100000000ull;
constexpr int AUX_SYS = 17;

#define WIN 3
#define STAGE_AUX 16
#define DMA_GAP 3
#define RT0 3
#define RT_DELAY 4
#define RT_STEP 1
#define NLATE 96
#define WIDE_RULE false
#define TAILN 16
#define TAILWIN 6
#define PUSH_WIN 3
#define ST_WIN 16
#define POLL_SLEEP 64

struct Params {
    const char* x; const char* w;
    char* out[NR]; char* ws[NR];
    int rank;
    unsigned units[8 * (NI + NX)];
    unsigned late[NGEMM / 4];
    unsigned tiles[NGEMM];
};

#define STR2(x) #x
#define STR(x) STR2(x)
#define WAIT_V(n) asm volatile("s_waitcnt vmcnt(" STR(n) ")" ::: "memory")
#define WAIT_L(n) asm volatile("s_waitcnt lgkmcnt(" STR(n) ")" ::: "memory")
#define BAR __builtin_amdgcn_s_barrier()
#define SCHED __builtin_amdgcn_sched_barrier(0)

#define STAMPV(i, v) do {} while (0)
#define STAMP(i) do {} while (0)

__device__ __forceinline__ unsigned ld_sys(const char* p) {
    return __hip_atomic_load((const GAS unsigned*)p, __ATOMIC_RELAXED, __HIP_MEMORY_SCOPE_SYSTEM);
}
__device__ __forceinline__ void st_sys(char* p, unsigned v) {
    __hip_atomic_store((GAS unsigned*)p, v, __ATOMIC_RELAXED, __HIP_MEMORY_SCOPE_SYSTEM);
}
__device__ __forceinline__ void st8_sys(char* p, unsigned v) {
    __hip_atomic_store((GAS unsigned char*)p, (unsigned char)v, __ATOMIC_RELAXED, __HIP_MEMORY_SCOPE_SYSTEM);
}
__device__ __forceinline__ unsigned call_tag(unsigned call) { return call % 255u + 1u; }
__device__ __forceinline__ void add_sys(char* p, unsigned v) {
    __hip_atomic_fetch_add((GAS unsigned*)p, v, __ATOMIC_RELAXED, __HIP_MEMORY_SCOPE_SYSTEM);
}
__device__ __forceinline__ unsigned cvt_pk_bf16(float lo, float hi) {
    unsigned r; asm volatile("v_cvt_pk_bf16_f32 %0, %1, %2" : "=v"(r) : "v"(lo), "v"(hi)); return r;
}
__device__ __forceinline__ int panel_row(int d, int P) {
    const int g = d >> 2, ai = (d >> 1) & 1, k = P & 7, u = k & 1, i = ((P >> 3) ? g : 1 - g) * 4 + (k >> 1);
    return (g == 0 ? ai * 2 + u : u * 2 + ai) * 2048 + i * 256;
}
__device__ __forceinline__ int panel_col(int d, int k) {
    const int g = d >> 2, aj = d & 1, v = k & 1;
    return (g == 0 ? aj * 2 + v : v * 2 + aj) * 2048 + (k >> 1) * 256;
}
__device__ __forceinline__ bool give_up(char* tmo, unsigned call, unsigned long long t0) {
    if (__builtin_amdgcn_readfirstlane(__hip_atomic_load((const GAS unsigned*)tmo, __ATOMIC_RELAXED, __HIP_MEMORY_SCOPE_AGENT)) == call
        || __builtin_amdgcn_s_memrealtime() - t0 > DEADLINE) {
        __hip_atomic_store((GAS unsigned*)tmo, call, __ATOMIC_RELAXED, __HIP_MEMORY_SCOPE_AGENT);
        return true;
    }
    return false;
}

struct Piece { const char* src; unsigned sA, sB, dst, pub; int kind; };

__device__ __forceinline__ Piece input_piece(const Params& p, int q, int dsel) {
    const int d = p.rank ^ dsel;
    const int ui = (q >> 2) + (dsel >= 4 ? 32 * NI : 0);
    const int u = (p.units[ui >> 2] >> ((ui & 3) * 8)) & 255;
    const int panel = u & 31, cc = u >> 5, pc = q & 3;
    const int c = dsel >= 4 ? NI + cc : cc;
    const int slot = dsel == 0 ? cc : (dsel >= 4 ? NI + cc * 4 + (dsel & 3) : NI + 4 * NX + cc * 3 + dsel - 1);
    const int k0 = c * 128 + (pc >> 1) * 64, h = pc & 1;
    Piece o; o.kind = panel >= 16;
    o.dst = (unsigned)panel * PANEL_B + (unsigned)slot * SLOT_B + (unsigned)pc * HT_B;
    o.pub = CNT_OFF + (unsigned)panel * 128 + (unsigned)slot * 4 + (unsigned)pc;
    if (o.kind) {
        o.src = p.w + ((size_t)k0 * DIM + panel_col(d, panel - 16) + h * 128) * 2;
        o.sA = 8u * DIM * 2; o.sB = 4u * DIM * 2;
    } else {
        o.src = p.x + ((size_t)(panel_row(d, panel) + h * 128) * KL + k0) * 2;
        o.sA = 16u * KL * 2; o.sB = 8u * KL * 2;
    }
    return o;
}
__device__ __forceinline__ unsigned send_off(int d, int panel, int c, int pc) {
    const int gp = panel >= 16 ? 32 + (panel_col(d, panel - 16) >> 8) : panel_row(d, panel) >> 8;
    return SEND_OFF + (unsigned)((gp * 8 + c) * 4 + pc) * HT_B;
}
__device__ __forceinline__ Piece pull_piece(const Params& p, int code, int dsel) {
    const int cls = (code >> 12) & 3, id = code & 4095, s = p.rank ^ dsel;
    const int pm = (id >> 7) & 7, pn = (id >> 3) & 15, w = id & 7;
    Piece o; o.kind = 2; o.sA = 2048; o.sB = 1024;
    if (cls == 1) {
        const int ui = (id >> 2) + (dsel >= 4 ? 32 * NI : 0);
        const int u = (p.units[ui >> 2] >> ((ui & 3) * 8)) & 255;
        const int panel = u & 31, cc = u >> 5, pc = id & 3;
        const int slot = dsel >= 4 ? NI + cc * 4 + (dsel & 3) : NI + 4 * NX + cc * 3 + dsel - 1;
        o.src = p.ws[s] + send_off(p.rank, panel, dsel >= 4 ? NI + cc : cc, pc);
        o.dst = (unsigned)panel * PANEL_B + (unsigned)slot * SLOT_B + (unsigned)pc * HT_B;
        o.pub = CNT_OFF + (unsigned)panel * 128 + (unsigned)slot * 4 + (unsigned)pc;
    } else if (cls == 0) {
        const int ot = ((pm & 6) | ((s >> 1) & 1)) * 16 + ((pn & 14) | (s & 1));
        o.src = p.ws[s] + POUT_OFF + (size_t)(id & 1023) * HT_B;
        o.dst = PIN_OFF + (unsigned)(ot * 8 + w) * HT_B;
        o.pub = PF_OFF + (unsigned)(ot * 8 + w) * 4;
    } else {
        o.kind = 3; o.sA = 4u * DIM * 2; o.sB = 2u * DIM * 2;
        o.dst = (unsigned)(panel_row(s, 8 + pm) + w * 32) * (DIM * 2) + (unsigned)panel_col(s, pn) * 2;
        o.src = p.out[s] + o.dst;
        o.pub = 0;
    }
    return o;
}
__device__ __forceinline__ const char* piece_src(const Piece& pc, int lane) {
    const unsigned xl = (unsigned)(lane >> 3) * (KL * 2) + (unsigned)((lane & 7) ^ (lane >> 3)) * 16;
    const unsigned wl = (unsigned)(lane >> 4) * (DIM * 2) + (unsigned)((lane & 15) ^ ((lane >> 4) << 2)) * 16;
    const unsigned ol = (unsigned)(lane >> 5) * (DIM * 2) + (unsigned)(lane & 31) * 16;
    return pc.src + (pc.kind == 0 ? xl : pc.kind == 1 ? wl : pc.kind == 2 ? (unsigned)lane * 16 : ol);
}
__device__ __forceinline__ void piece_dma2(const Piece& pc, const char* s, LAS char* buf, int i) {
    __builtin_amdgcn_global_load_lds((const GAS unsigned*)(s + (size_t)i * pc.sA), (LAS unsigned*)(buf + i * 2048), 16, 0, 16);
    __builtin_amdgcn_global_load_lds((const GAS unsigned*)(s + (size_t)i * pc.sA + pc.sB), (LAS unsigned*)(buf + i * 2048 + 1024), 16, 0, 16);
}
__device__ __forceinline__ void piece_dma(const Piece& pc, LAS char* buf, int lane) {
    const char* s = piece_src(pc, lane);
#pragma unroll
    for (int i = 0; i < 8; ++i) piece_dma2(pc, s, buf, i);
}
__device__ __forceinline__ void piece_read_tr(u32x4 (&v)[16], LAS char* buf, int lane) {
    const int q = (lane >> 2) & 3;
    const unsigned trl = (unsigned)(8 * (lane >> 4) + q) * 256 + (unsigned)(lane & 3) * 16;
#pragma unroll
    for (int f = 0; f < 16; ++f) {
        const int nb = f >> 1, kb = f & 1;
        LAS char* a = buf + trl + (((nb >> 1) ^ q) * 64) + kb * 8192 + (nb & 1) * 8;
        const u32x2 lo = __builtin_bit_cast(u32x2, __builtin_amdgcn_ds_read_tr16_b64_v4i16((LAS s16x4*)a));
        const u32x2 hi = __builtin_bit_cast(u32x2, __builtin_amdgcn_ds_read_tr16_b64_v4i16((LAS s16x4*)(a + 1024)));
        v[f] = (u32x4){lo.x, lo.y, hi.x, hi.y};
    }
}
__device__ __forceinline__ void piece_read(u32x4 (&v)[16], bool isx, LAS char* buf, int lane) {
    const unsigned xb = (unsigned)(lane & 15) * 128 + (unsigned)((lane >> 4) ^ (lane & 7)) * 16;
    LAS char* b0 = buf + (isx ? xb : (unsigned)lane * 16);
    LAS char* b1 = buf + (isx ? xb ^ 64u : (unsigned)lane * 16 + 1024u);
#pragma unroll
    for (int f = 0; f < 16; ++f) asm volatile("ds_read_b128 %0, %1 offset:%2" : "=v"(v[f]) : "v"((f & 1) ? b1 : b0), "i"((f >> 1) * 2048) : "memory");
    asm volatile("s_waitcnt lgkmcnt(0)" : "+v"(v[0]), "+v"(v[1]), "+v"(v[2]), "+v"(v[3]), "+v"(v[4]), "+v"(v[5]), "+v"(v[6]), "+v"(v[7]),
                 "+v"(v[8]), "+v"(v[9]), "+v"(v[10]), "+v"(v[11]), "+v"(v[12]), "+v"(v[13]), "+v"(v[14]), "+v"(v[15]) :: "memory");
}

#define LOOKAHEAD 3
constexpr int KSL = LOOKAHEAD;
constexpr int NMB = 3 * KSL;

__device__ __forceinline__ int next_task(LAS unsigned* mb, int lane) {
    for (;;) {
        unsigned v;
        asm volatile("ds_read_b32 %0, %1\n\ts_waitcnt lgkmcnt(0)" : "=&v"(v) : "v"(mb + (lane & 15)) : "memory");
        unsigned best = 0xffffffffu; int bl = 0;
#pragma unroll
        for (int l = 0; l < NMB; ++l) { const unsigned k = (unsigned)__builtin_amdgcn_readlane(v, l) - 1u; if (k < best) { best = k; bl = l; } }
        if (best == 0xffffffffu) return __builtin_amdgcn_readlane(v, NMB) ? -2 : -1;
        unsigned old = 0;
        if (lane == 0) asm volatile("ds_cmpst_rtn_b32 %0, %1, %2, %3\n\ts_waitcnt lgkmcnt(0)" : "=&v"(old) : "v"(mb + bl), "v"(best + 1u), "v"(0u) : "memory");
        if ((unsigned)__builtin_amdgcn_readfirstlane(old) == best + 1u) {
            const int cls = best >> 21;
            const int pos = (int)(best >> 10) & 2047;
            return cls << 12 | (cls == 1 ? pos : (int)best & 1023) | (cls == 2 && pos >= 1024 - TAILN ? 1 << 14 : 0);
        }
    }
}
static_assert(WIN >= 2 && WIN <= 8 && NS >= 1 && NS <= 3, "window / pullers");
__device__ __forceinline__ void lds_wait_zero(LAS unsigned* a) {
    for (;;) {
        unsigned f;
        asm volatile("ds_read_b32 %0, %1\n\ts_waitcnt lgkmcnt(0)" : "=&v"(f) : "v"(a) : "memory");
        if (__builtin_amdgcn_readfirstlane(f) == 0) break;
        __builtin_amdgcn_s_sleep(1);
    }
}

__device__ __forceinline__ void link_wg(const Params& p, LAS char* lds, int dsel, int e) {
    const int wid = __builtin_amdgcn_readfirstlane(threadIdx.x >> 6), lane = threadIdx.x & 63;
    char* ws = p.ws[p.rank];
    LAS unsigned* mb = (LAS unsigned*)(lds + 7 * HT_B);
    const int sn = wid < NS ? wid : wid - NS;
    LAS char* buf = lds + sn * 2 * HT_B;
    LAS unsigned* slot = mb + 16 + sn * 8;
    STAMPV(0, dsel);
    Piece first;
    if (wid >= NS && wid < 2 * NS) { first = input_piece(p, sn * NE + e, dsel); piece_dma(first, buf + HT_B, lane); }
    else if (wid == 7) {
        const unsigned c = ld_sys(ws + CALL_OFF) + 1u;
        mb[lane] = lane == 15 ? c : (lane >= 16 && (lane & 7) == 4) ? 14u : 0u; mb[64 + lane] = p.units[lane]; WAIT_L(0);
        if (lane == 0) add_sys(ws + DONE_OFF, 1u);
    }
    BAR; asm volatile("" ::: "memory");
    unsigned call;
    asm volatile("ds_read_b32 %0, %1 offset:60\n\ts_waitcnt lgkmcnt(0)" : "=&v"(call) : "v"(mb) : "memory");
    call = __builtin_amdgcn_readfirstlane(call);
    if (wid >= 2 * NS) {
        if (wid != 7) return;
        const int cls = lane < KSL ? 0 : lane < 2 * KSL ? 1 : 2;
        const unsigned nfirst = cls == 1 ? NS * NE : 0u;
        const unsigned n = cls == 0 ? 256u : cls == 2 ? 1024u : dsel >= 4 ? 32u * NX * 4 : 32u * NI * 4;
        const unsigned base = (call - 1u) * (n - nfirst + NE * KSL) - nfirst;
        char* head = ws + HEAD_OFF + (dsel * 4 + cls) * 128;
        const char* list = ws + (cls == 0 ? PL_OFF + (dsel & 3) * 1024 : cls == 2 ? FL_OFF + dsel * 4096 : IR_OFF + dsel * 1024);
        const unsigned tag = call & 0xfffffu, tag8 = call_tag(call);
        int st = (lane >= NMB || (cls == 0 && dsel < 4)) ? 3 : 0;
        unsigned pos = 0, lo = 0;
        const unsigned long long t0 = __builtin_amdgcn_s_memrealtime();
        for (unsigned polls = 1;; ++polls) {
            if (st == 2 && __hip_atomic_load(mb + lane, __ATOMIC_RELAXED, __HIP_MEMORY_SCOPE_WORKGROUP) == 0u) st = 0;
            unsigned t = 0, en = 0;
            if (st == 0) t = __hip_atomic_fetch_add((GAS unsigned*)head, 1u, __ATOMIC_RELAXED, __HIP_MEMORY_SCOPE_AGENT);
            if (st == 1) en = ld_sys(list + lo);
            if (st == 0) {
                pos = t - base; st = pos >= n ? 3 : 1; lo = pos * 4;
                if (st == 1 && cls == 1) { const unsigned u = ((LAS unsigned char*)(mb + 64))[(pos >> 2) + (dsel >= 4 ? 32 * NI : 0)]; lo = (u & 31u) * 32 + (u >> 5) * 4; }
            } else if (st == 1 && (cls == 1 ? ((en >> ((pos & 3u) * 8)) & 255u) == tag8 : (en >> 12) == tag)) {
                __hip_atomic_store(mb + lane, ((((unsigned)cls << 11 | pos) << 10) | (en & 1023u)) + 1u, __ATOMIC_RELAXED, __HIP_MEMORY_SCOPE_WORKGROUP);
                st = 2;
            }
            if (__all(st == 3)) break;
            if ((polls & 63u) == 0u && give_up(ws + TMO_OFF, call, t0)) {
                if (lane == 0) st_sys(p.out[p.rank] + ((size_t)panel_row(p.rank, 8) * DIM + (size_t)panel_col(p.rank, 0)) * 2, 0x7fc07fc0u);
                break;
            }
            __builtin_amdgcn_s_sleep(2);
        }
        if (lane == 0) __hip_atomic_store(mb + NMB, 1u, __ATOMIC_RELAXED, __HIP_MEMORY_SCOPE_WORKGROUP);
        return;
    }
    if (wid < NS) {
        int k = 0, code = -1, si = 1, waited = 0; (void)si; (void)waited;
        bool pend = false;
        u32x4 desc = {0u, 0u, 0u, 0u};
        Piece cur;
        for (;;) {
            if (code < 0) {
                if (code == -1) code = next_task(mb, lane);
                if (code < 0) {
                    if (pend) { WAIT_V(0); if (lane == 0) asm volatile("ds_write_b128 %0, %1" :: "v"(slot + ((k - 1) & 1) * 4), "v"(desc) : "memory"); pend = false; }
                    while (code == -1) { __builtin_amdgcn_s_sleep(2); code = next_task(mb, lane); }
                    if (code < 0) break;
                    waited = 8;
                }
                cur = pull_piece(p, code, dsel);
            }
            lds_wait_zero(slot + (k & 1) * 4);
            LAS char* b = buf + (k & 1) * HT_B;
            const char* sp = piece_src(cur, lane);
            const unsigned stride = cur.sB;
            const u32x4 ndesc = {(unsigned)cur.kind + 1u, cur.dst, cur.pub, 0u};
            const int cls0 = (code >> 12) & 3; (void)cls0;
            const bool fast = code >> 14;
            const bool wide = WIDE_RULE;
#pragma unroll
            for (int f = 0; f < 16; ++f) {
                __builtin_amdgcn_global_load_lds((const GAS unsigned*)sp, (LAS unsigned*)(b + f * 1024), 16, 0, AUX_SYS);
                sp += stride;
                asm volatile("" ::: "memory");
                if (f == 10) { code = next_task(mb, lane); if (code >= 0) cur = pull_piece(p, code, dsel); }
                if (f > WIN - 2 && fast) WAIT_V(TAILWIN - 1); else if (f != WIN - 2 && wide) WAIT_V(WIN); else WAIT_V(WIN - 1);
                if (f == WIN - 2 && pend && lane == 0)
                    asm volatile("ds_write_b128 %0, %1" :: "v"(slot + ((k - 1) & 1) * 4), "v"(desc) : "memory");
            }
            desc = ndesc; pend = true; ++k;
            if (si < 128) { STAMPV(si, (cls0 == 1 ? 1 : cls0 == 0 ? 3 : 4) | waited); } ++si; waited = 0;
        }
        lds_wait_zero(slot + (k & 1) * 4);
        if (lane == 0) asm volatile("ds_write_b32 %0, %1" :: "v"(slot + (k & 1) * 4), "v"(15u) : "memory");
        return;
    }
    {
        char* dws = p.ws[p.rank ^ dsel];
        const __amdgpu_buffer_rsrc_t rs = __builtin_amdgcn_make_buffer_rsrc(dws, 0, STAGE_B, 0x00020000);
        WAIT_V(0);
        u32x4 v[16];
        if (first.kind) { piece_read_tr(v, buf + HT_B, lane); WAIT_L(0);
#pragma unroll
            for (int f = 0; f < 16; ++f) asm volatile("" : "+v"(v[f]));
        } else piece_read(v, true, buf + HT_B, lane);
        if (lane == 0) asm volatile("ds_write_b32 %0, %1" :: "v"(slot + 4), "v"(0u) : "memory");
#pragma unroll
        for (int f = 0; f < 16; ++f) { __builtin_amdgcn_raw_buffer_store_b128(v[f], rs, first.dst + lane * 16 + f * 1024, 0, AUX_SYS); WAIT_V(PUSH_WIN - 1); }
        WAIT_V(0);
        if (lane == 0) st8_sys(dws + first.pub, call_tag(call));
    }
    const __amdgpu_buffer_rsrc_t rsw = __builtin_amdgcn_make_buffer_rsrc(ws, 0, POUT_OFF, 0x00020000);
    const __amdgpu_buffer_rsrc_t rso = __builtin_amdgcn_make_buffer_rsrc(p.out[p.rank], 0, OUT_B, 0x00020000);
    const unsigned ol = (unsigned)(lane >> 5) * (DIM * 2) + (unsigned)(lane & 31) * 16;
    for (int k = 0;;) {
        u32x4 ds4;
        asm volatile("ds_read_b128 %0, %1\n\ts_waitcnt lgkmcnt(0)" : "=&v"(ds4) : "v"(slot + (k & 1) * 4) : "memory");
        const int kind = __builtin_amdgcn_readfirstlane(ds4.x) - 1;
        if (kind > 3) break;
        if (kind < 0) { __builtin_amdgcn_s_sleep(1); continue; }
        const unsigned dst = __builtin_amdgcn_readfirstlane(ds4.y), pub = __builtin_amdgcn_readfirstlane(ds4.z);
        u32x4 v[16];
        piece_read(v, false, buf + (k & 1) * HT_B, lane);
        if (lane == 0) asm volatile("ds_write_b32 %0, %1" :: "v"(slot + (k & 1) * 4), "v"(0u) : "memory");
        const bool o = kind == 3;
        const __amdgpu_buffer_rsrc_t rs = o ? rso : rsw;
        const unsigned ds = o ? 2u * DIM * 2 : 1024u;
        unsigned vo = dst + (o ? ol : (unsigned)lane * 16);
#pragma unroll
        for (int f = 0; f < 16; ++f) {
            __builtin_amdgcn_raw_buffer_store_b128(v[f], rs, vo, 0, AUX_SYS);
            vo += ds;
            WAIT_V(ST_WIN - 1);
        }
        if (pub) {
            WAIT_V(0);
            if (lane == 0) { if (pub >= PF_OFF) st_sys(ws + pub, call); else st8_sys(ws + pub, call_tag(call)); }
        }
        ++k;
    }
    STAMP(1);
}

__device__ __forceinline__ int ready_prefix(unsigned v, unsigned target) {
    const unsigned long long m = __ballot(v == target);
    const unsigned both = (unsigned)m & (unsigned)(m >> 32);
    return both == 0xffffffffu ? 32 : __builtin_ctz(~both);
}
__device__ __noinline__ int wait_slot(const char* cptr, char* tmo, unsigned call, int need) {
    const unsigned long long t0 = __builtin_amdgcn_s_memrealtime();
    int known;
    for (unsigned polls = 1;; ++polls) {
        known = ready_prefix(ld_sys(cptr), call_tag(call) * 0x01010101u);
        if (known > need) break;
        if ((polls & 63u) == 0u && give_up(tmo, call, t0)) { known = 32; break; }
        __builtin_amdgcn_s_sleep(POLL_SLEEP);
    }
    return known;
}

__device__ __forceinline__ void gemm_wg(const Params& p, LAS char* lds, int gb) {
    const int tid = threadIdx.x, wid = __builtin_amdgcn_readfirstlane(tid >> 6), lane = tid & 63, wr = wid >> 2, wc = wid & 3;
    char* ws = p.ws[p.rank];
    STAMP(0);
    const unsigned call = __builtin_amdgcn_readfirstlane(ld_sys(ws + CALL_OFF)) + 1u;
    const int late = (int)(signed char)((p.late[gb >> 2] >> ((gb & 3) * 8)) & 255);
    const bool bulk = wid >= RT0;
    if (bulk && late >= 0) for (int i = 0; i < RT_DELAY + (wid - RT0) * RT_STEP; ++i) __builtin_amdgcn_s_sleep(32);
    for (int r = !bulk ? wid * NGEMM + gb : late >= 0 ? RT0 * NGEMM + (wid - RT0) * NLATE + late : 2048; r < 2048; r = bulk ? r + (8 - RT0) * NLATE : 2048) {
        const int cross = r < 1536 ? (r >> 3) & 1 : 0, n = r < 1536 ? r >> 4 : 96 + ((r - 1536) >> 3), half = (r >> 2) & 1, pcs = r & 3;
        const int ui = n + (cross ? 32 * NI : 0);
        const int u = (p.units[ui >> 2] >> ((ui & 3) * 8)) & 255, panel = u & 31, cc = u >> 5;
        const int d0 = (((p.rank >> 2) ^ cross) << 2) + (panel >= 16 ? half : half * 2), d1 = d0 + (panel >= 16 ? 2 : 1);
        const bool own = d0 == p.rank || d1 == p.rank;
        const Piece pc = input_piece(p, n * 4 + pcs, own ? 0 : p.rank ^ d0);
        LAS char* buf = lds + wid * HT_B;
        piece_dma(pc, buf, lane);
        WAIT_V(0);
        u32x4 v[16];
        if (pc.kind) piece_read_tr(v, buf, lane); else piece_read(v, true, buf, lane);
        const __amdgpu_buffer_rsrc_t rs = __builtin_amdgcn_make_buffer_rsrc(ws, 0, DATA_B, 0x00020000);
        const unsigned so = send_off(d0, panel, cross ? NI + cc : cc, pcs);
#pragma unroll
        for (int f = 0; f < 16; ++f) __builtin_amdgcn_raw_buffer_store_b128(v[f], rs, so + lane * 16 + f * 1024, 0, AUX_SYS);
        if (own) {
#pragma unroll
            for (int f = 0; f < 16; ++f) __builtin_amdgcn_raw_buffer_store_b128(v[f], rs, pc.dst + lane * 16 + f * 1024, 0, AUX_SYS);
        }
        WAIT_V(0);
        if (lane == 0) {
            const unsigned fo = (unsigned)(panel * 32 + cc * 4 + pcs);
            if (d0 != p.rank) st8_sys(p.ws[d0] + IR_OFF + (p.rank ^ d0) * 1024 + fo, call_tag(call));
            if (d1 != p.rank) st8_sys(p.ws[d1] + IR_OFF + (p.rank ^ d1) * 1024 + fo, call_tag(call));
            if (own) st8_sys(ws + pc.pub, call_tag(call));
        }
    }
    BAR;
    if (tid == 0) add_sys(ws + DONE_OFF, 1u);
    STAMP(1);
    const unsigned target = call_tag(call) * 0x01010101u;
    const unsigned voff = (unsigned)tid * 16;
    const unsigned ldsw = (unsigned)wid * 1024u;
    const int aoff = wr * 8192 + lane * 16, boff = wc * 4096 + lane * 16;
#define SA(b, h) (((b) * 2 + (h)) * HT_B)
#define SB(b, h) ((4 + (b) * 2 + (h)) * HT_B)
#define STAGE(bufoff, gbase) do { _Pragma("unroll") for (int _i = 0; _i < 2; ++_i) \
        __builtin_amdgcn_global_load_lds((const GAS unsigned*)((const char*)(gbase) + voff + _i * 8192), (LAS unsigned*)(lds + (bufoff) + ldsw + _i * 8192), 16, 0, STAGE_AUX); } while (0)
#define LDA(dst, b, h) do { _Pragma("unroll") for (int m = 0; m < 4; ++m) _Pragma("unroll") for (int k = 0; k < 2; ++k) dst[m][k] = *(const LAS bf16x8*)(lds + SA(b, h) + aoff + m * 2048 + k * 1024); } while (0)
#define LDB(dst, b, h) do { _Pragma("unroll") for (int n = 0; n < 2; ++n) _Pragma("unroll") for (int k = 0; k < 2; ++k) dst[n][k] = *(const LAS bf16x8*)(lds + SB(b, h) + boff + n * 2048 + k * 1024); } while (0)
#define MMA(ai, bj, At, Bt) do { __builtin_amdgcn_s_setprio(1); _Pragma("unroll") for (int m = 0; m < 4; ++m) _Pragma("unroll") for (int n = 0; n < 2; ++n) _Pragma("unroll") for (int k = 0; k < 2; ++k) \
        acc[ai][bj][m][n] = __builtin_amdgcn_mfma_f32_16x16x32_bf16(Bt[n][k], At[m][k], acc[ai][bj][m][n], 0, 0, 0); __builtin_amdgcn_s_setprio(0); } while (0)
    for (int rnd = 0; rnd < 2; ++rnd) {
        const int code = (p.tiles[gb] >> (16 * rnd)) & 0xffff;
        if (code == 0xffff) break;
        const int pm = (code >> 4) & 15, pn = code & 15;
        const char* Ab = ws + (size_t)pm * PANEL_B;
        const char* Bb = ws + (size_t)(16 + pn) * PANEL_B;
        const unsigned coff = CNT_OFF + (unsigned)(lane < 32 ? pm : 16 + pn) * 128 + (unsigned)(lane & 31) * 4;
        f32x4 acc[2][2][4][2];
#pragma unroll
        for (int a = 0; a < 2; ++a)
#pragma unroll
            for (int b = 0; b < 2; ++b)
#pragma unroll
                for (int m = 0; m < 4; ++m)
#pragma unroll
                    for (int n = 0; n < 2; ++n) acc[a][b][m][n] = (f32x4){0.f, 0.f, 0.f, 0.f};
        bf16x8 At[4][2], B0[2][2], B1[2][2];
        int known = wait_slot(ws + coff, ws + TMO_OFF, call, 0);
        STAMP(2 + rnd * 12);
        STAGE(SB(0, 0), Bb); STAGE(SA(0, 0), Ab); STAGE(SB(0, 1), Bb + HT_B); STAGE(SA(0, 1), Ab + HT_B);
        if (wr == 1) BAR;
        WAIT_V(4); BAR;
        STAGE(SB(1, 0), Bb + KT_B); STAGE(SA(1, 0), Ab + KT_B); STAGE(SB(1, 1), Bb + KT_B + HT_B);
        WAIT_V(6); BAR;
        for (int T = 0; T < 32; ++T) {
            if (T < 31 && known <= T + 1) known = wait_slot(ws + coff, ws + TMO_OFF, call, T + 1);
            const unsigned o1 = (unsigned)T * SLOT_B, o2 = (unsigned)(T < 31 ? T + 1 : T) * SLOT_B;
            const char* a1 = Ab + o1 + KT_B; const char* a2 = Ab + o2; const char* b2 = Bb + o2;
            const char* a3 = a2 + KT_B; const char* b3 = b2 + KT_B;
            unsigned pv;
            LDB(B0, 0, 0); SCHED; LDA(At, 0, 0);
            asm volatile("global_load_dword %0, %1, %2 sc0 sc1" : "=v"(pv) : "v"(coff), "s"(ws) : "memory");
            STAGE(SA(1, 1), a1 + HT_B);
            WAIT_L(8); BAR; WAIT_L(0); MMA(0, 0, At, B0); BAR; SCHED;
            LDB(B1, 0, 1); STAGE(SB(0, 0), b2);
            BAR; WAIT_L(0); MMA(0, 1, At, B1); BAR;
            LDA(At, 0, 1); STAGE(SA(0, 0), a2);
            BAR; WAIT_L(0); MMA(1, 0, At, B0); BAR; SCHED;
            STAGE(SB(0, 1), b2 + HT_B);
            WAIT_V(6);
            asm volatile("" : "+v"(pv));
            { const int k2 = ready_prefix(pv, target); known = k2 > known ? k2 : known; }
            BAR; MMA(1, 1, At, B1); BAR;
            LDB(B0, 1, 0); SCHED; LDA(At, 1, 0); STAGE(SA(0, 1), a2 + HT_B);
            WAIT_L(8); BAR; WAIT_L(0); MMA(0, 0, At, B0); BAR; SCHED;
            LDB(B1, 1, 1); STAGE(SB(1, 0), b3);
            BAR; WAIT_L(0); MMA(0, 1, At, B1); BAR;
            LDA(At, 1, 1); STAGE(SA(1, 0), a3);
            BAR; WAIT_L(0); MMA(1, 0, At, B0); BAR; SCHED;
            STAGE(SB(1, 1), b3 + HT_B);
            WAIT_V(6); BAR; MMA(1, 1, At, B1); BAR;
        }
        WAIT_V(0);
        if (wr == 0) BAR;
        BAR;
        STAMP(11 + rnd * 12);
        int tz = threadIdx.x; asm volatile("" : "+v"(tz));
        const int ln = tz & 63;
        const int id = ((pm & 7) * 16 + pn) * 8 + wid;
        if (pm < 8) {
            const int j = (((pm & 1) * 2 + (pn & 1)) ^ p.rank) & 3;
            unsigned pos = 0;
            if (ln == 0) pos = __hip_atomic_fetch_add((GAS unsigned*)(ws + TAIL_OFF + j * 128), 1u, __ATOMIC_RELAXED, __HIP_MEMORY_SCOPE_AGENT);
            const bool bad = __builtin_amdgcn_readfirstlane(__hip_atomic_load((const GAS unsigned*)(ws + TMO_OFF), __ATOMIC_RELAXED, __HIP_MEMORY_SCOPE_AGENT)) == call;
            const __amdgpu_buffer_rsrc_t rp = __builtin_amdgcn_make_buffer_rsrc(ws + POUT_OFF, 0, 128 * TILE_B, 0x00020000);
            const unsigned base = (unsigned)id * HT_B + (unsigned)ln * 16;
#pragma unroll
            for (int ai = 0; ai < 2; ++ai)
#pragma unroll
                for (int m = 0; m < 4; ++m)
#pragma unroll
                    for (int bj = 0; bj < 2; ++bj) {
                        const f32x4 v0 = acc[ai][bj][m][0], v1 = acc[ai][bj][m][1];
                        u32x4 w; w.x = cvt_pk_bf16(v0[0], v0[1]); w.y = cvt_pk_bf16(v0[2], v0[3]); w.z = cvt_pk_bf16(v1[0], v1[1]); w.w = cvt_pk_bf16(v1[2], v1[3]);
                        if (bad) w = (u32x4){0x7fc07fc0u, 0x7fc07fc0u, 0x7fc07fc0u, 0x7fc07fc0u};
                        __builtin_amdgcn_raw_buffer_store_b128(w, rp, base + (unsigned)((ai * 4 + m) * 2 + bj) * 1024, 0, AUX_SYS);
                    }
            WAIT_V(0);
            pos &= 255u;
            if (ln == 0) st_sys(p.ws[p.rank ^ (4 + j)] + PL_OFF + (j * 256 + pos) * 4, (call & 0xfffffu) << 12 | (unsigned)id);
            STAMP(12 + rnd * 12);
            continue;
        }
        {
            const unsigned long long t0 = __builtin_amdgcn_s_memrealtime();
            for (unsigned polls = 1;; ++polls) {
                if ((int)(__builtin_amdgcn_readfirstlane(ld_sys(ws + PF_OFF + id * 4)) - call) >= 0) break;
                if ((polls & 63u) == 0u && give_up(ws + TMO_OFF, call, t0)) break;
                __builtin_amdgcn_s_sleep(16);
            }
            asm volatile("" ::: "memory");
            STAMP(26);
            const bool bad = __builtin_amdgcn_readfirstlane(__hip_atomic_load((const GAS unsigned*)(ws + TMO_OFF), __ATOMIC_RELAXED, __HIP_MEMORY_SCOPE_AGENT)) == call;
            const __amdgpu_buffer_rsrc_t rp = __builtin_amdgcn_make_buffer_rsrc(ws + PIN_OFF, 0, 128 * TILE_B, 0x00020000);
            const unsigned base = (unsigned)id * HT_B + (unsigned)ln * 16;
            u32x4 pr[16];
#pragma unroll
            for (int f = 0; f < 16; ++f) pr[f] = __builtin_amdgcn_raw_buffer_load_b128(rp, base + f * 1024, 0, AUX_SYS);
            const int fr = ln & 15, fq = ln >> 4;
#pragma unroll
            for (int ai = 0; ai < 2; ++ai)
#pragma unroll
                for (int m = 0; m < 4; ++m) {
                    const int row = ai * 128 + wr * 64 + m * 16 + fr;
#pragma unroll
                    for (int bj = 0; bj < 2; ++bj) {
                        const u32x4 q = pr[(ai * 4 + m) * 2 + bj];
                        const f32x4 v0 = acc[ai][bj][m][0], v1 = acc[ai][bj][m][1];
                        u32x4 w;
                        w.x = cvt_pk_bf16(v0[0] + __uint_as_float(q.x << 16), v0[1] + __uint_as_float(q.x & 0xffff0000u));
                        w.y = cvt_pk_bf16(v0[2] + __uint_as_float(q.y << 16), v0[3] + __uint_as_float(q.y & 0xffff0000u));
                        w.z = cvt_pk_bf16(v1[0] + __uint_as_float(q.z << 16), v1[1] + __uint_as_float(q.z & 0xffff0000u));
                        w.w = cvt_pk_bf16(v1[2] + __uint_as_float(q.w << 16), v1[3] + __uint_as_float(q.w & 0xffff0000u));
                        if (bad) w = (u32x4){0x7fc07fc0u, 0x7fc07fc0u, 0x7fc07fc0u, 0x7fc07fc0u};
                        const int ch = (bj * 16 + wc * 4 + fq) ^ (fr & 7);
                        *(LAS u32x4*)(lds + row * 512 + ch * 16) = w;
                    }
                }
        }
        WAIT_L(0); BAR; asm volatile("" ::: "memory");
        {
            unsigned pos = 0;
            if (ln == 0) pos = __hip_atomic_fetch_add((GAS unsigned*)(ws + TAIL_OFF + 4 * 128), 1u, __ATOMIC_RELAXED, __HIP_MEMORY_SCOPE_AGENT);
            const __amdgpu_buffer_rsrc_t ro = __builtin_amdgcn_make_buffer_rsrc(p.out[p.rank], 0, OUT_B, 0x00020000);
            const unsigned vo = (unsigned)(panel_row(p.rank, pm) + wid * 32 + (ln >> 5)) * (DIM * 2) + (unsigned)panel_col(p.rank, pn) * 2 + (unsigned)(ln & 31) * 16;
            const int hi = ln >> 5, chn = ln & 31;
#pragma unroll
            for (int f = 0; f < 16; ++f) {
                const u32x4 v = *(const LAS u32x4*)(lds + (wid * 32 + 2 * f + hi) * 512 + ((chn ^ ((2 * f + hi) & 7)) * 16));
                __builtin_amdgcn_raw_buffer_store_b128(v, ro, vo + (unsigned)(2 * f) * (DIM * 2), 0, AUX_SYS);
            }
            WAIT_V(0);
            pos &= 1023u;
            if (ln == 0) {
#pragma unroll
                for (int q = 1; q < NR; ++q) st_sys(p.ws[p.rank ^ q] + FL_OFF + (q * 1024 + pos) * 4, (call & 0xfffffu) << 12 | (unsigned)id);
            }
            STAMP(27);
        }
        break;
    }
#undef SA
#undef SB
#undef STAGE
#undef LDA
#undef LDB
#undef MMA
}

__global__ __launch_bounds__(512, 2) void gemm_ar_kernel(const Params p) {
    extern __shared__ __attribute__((aligned(16))) char smem[];
    LAS char* lds = (LAS char*)smem;
    const int b = blockIdx.x;
    if (b >= NLINK) { gemm_wg(p, lds, b - NLINK); return; }
#define MAPMODE 2
    {
        const int lab = b & 7, e = b >> 3;
        const int dsel = (lab + e) & 7;
        if (dsel) { link_wg(p, lds, dsel, e); return; }
    }
    if (b == 0 && threadIdx.x < 64) {
        char* ws = p.ws[p.rank];
        const unsigned call = __builtin_amdgcn_readfirstlane(ld_sys(ws + CALL_OFF)) + 1u;
        const unsigned long long t0 = __builtin_amdgcn_s_memrealtime();
        for (unsigned polls = 1;; ++polls) {
            if ((int)(__builtin_amdgcn_readfirstlane(ld_sys(ws + DONE_OFF)) - call * (unsigned)(7 * NE + NGEMM)) >= 0) break;
            if ((polls & 63u) == 0u && give_up(ws + TMO_OFF, call, t0)) break;
            __builtin_amdgcn_s_sleep(32);
        }
        if (threadIdx.x == 0) st_sys(ws + CALL_OFF, call);
    }
}

void build_schedule(Params& p) {
#define BATCH_ORDER {0, 1, 4, 2, 5, 3, 6, 7}
    static const int base[8][6] = {{0, 1, 16, 17, 18, 19}, {2, 3, 20, 21, 22, 23}, {4, 5, 24, 25, 26, 27}, {6, 7, 28, 29, 30, 31},
                                   {8, 9, -1, -1, -1, -1}, {10, 11, -1, -1, -1, -1}, {12, 13, -1, -1, -1, -1}, {14, 15, -1, -1, -1, -1}};
    const int order[8] = BATCH_ORDER;
    int pos[32], ni = 0, nx = 32 * NI;
    unsigned char* un = (unsigned char*)p.units;
    unsigned short* tl = (unsigned short*)p.tiles;
    for (int b = 0; b < 8; ++b) {
        const int* pl = base[order[b]];
        for (int i = 0; i < 6; ++i) if (pl[i] >= 0) pos[pl[i]] = b;
        for (int c = 0; c < NI; ++c) for (int i = 0; i < 6; ++i) if (pl[i] >= 0) un[ni++] = (unsigned char)(pl[i] | c << 5);
        for (int c = 0; c < NX; ++c) for (int i = 0; i < 6; ++i) if (pl[i] >= 0) un[nx++] = (unsigned char)(pl[i] | c << 5);
    }
    for (int i = 0; i < NGEMM; ++i) p.tiles[i] = 0xffffffffu;
    for (int own = 0; own < 2; ++own) {
        int next = 0;
        for (int b = 0; b < 8; ++b) {
            int list[128], n = 0;
            for (int r = 0; r < 8; ++r) for (int c = 0; c < 16; ++c) {
                const int a = pos[own * 8 + r], bb = pos[16 + c];
                if ((a > bb ? a : bb) == b) list[n++] = (own * 8 + r) << 4 | c;
            }
            int k = 0;
            for (int x = 0; x < 8; ++x) for (int s = next; s < next + n; ++s) if ((s & 7) == x) {
                const int wg = own == 0 ? s : (s < NGEMM - 128 ? 128 + s : s - (NGEMM - 128)), rnd = (own == 1 && s >= NGEMM - 128) ? 1 : 0;
                tl[wg * 2 + rnd] = (unsigned short)list[k++];
            }
            next += n;
        }
    }
    signed char* lt = (signed char*)p.late;
    int nl = 0;
    for (int i = 0; i < NGEMM; ++i) lt[i] = -1;
    for (int b = 7; b >= 0; --b) for (int i = NGEMM - 1; i >= 0 && nl < NLATE; --i) {
        const int a = pos[tl[2 * i] >> 4], bb = pos[16 + (tl[2 * i] & 15)];
        if ((a > bb ? a : bb) == b) lt[i] = (signed char)nl++;
    }
}
}

extern "C" void kernel_launch(int rank, int nranks, ncclComm_t comm,
                              hipStream_t stream,
                              void* const* d_in, const int* in_sizes, int n_in,
                              void* d_out, int out_size,
                              const int* in_dtypes, int out_dtype,
                              void* const* d_out_all,
                              void* const* d_ws_all, size_t ws_bytes) {
    (void)comm; (void)in_dtypes; (void)out_dtype; (void)d_out;
    if (nranks != NR || (unsigned)rank >= (unsigned)NR || n_in < 2 || in_sizes[0] != DIM * KL || in_sizes[1] != KL * DIM ||
        out_size != DIM * DIM || ws_bytes < WS_NEED) {
        fprintf(stderr, "kernel_launch: unexpected problem shape\n");
        return;
    }
    Params p;
    p.x = (const char*)d_in[0]; p.w = (const char*)d_in[1]; p.rank = rank;
    for (int r = 0; r < NR; ++r) { p.out[r] = (char*)d_out_all[r]; p.ws[r] = (char*)d_ws_all[r]; }
    build_schedule(p);
    HIP_CHECK(hipFuncSetAttribute((const void*)gemm_ar_kernel, hipFuncAttributeMaxDynamicSharedMemorySize, LDS_B));
    hipLaunchKernelGGL(gemm_ar_kernel, dim3(NLINK + NGEMM), dim3(512), LDS_B, stream, p);
    HIP_CHECK(hipGetLastError());
}
```

```cpp
#include <hip/hip_runtime.h>
#include <rccl/rccl.h>
#include <cstdio>

#define HIP_CHECK(expr) do { hipError_t _e = (expr); if (_e != hipSuccess) { \
    fprintf(stderr, "HIP error %s at %s:%d\n", hipGetErrorString(_e), __FILE__, __LINE__); \
    return; } } while (0)

#define LAS __attribute__((address_space(3)))
#define GAS __attribute__((address_space(1)))
typedef short bf16x8 __attribute__((ext_vector_type(8)));
typedef short s16x4 __attribute__((ext_vector_type(4)));
typedef float f32x4 __attribute__((ext_vector_type(4)));
typedef unsigned u32x4 __attribute__((ext_vector_type(4)));
typedef unsigned u32x2 __attribute__((ext_vector_type(2)));

namespace {
constexpr int NR = 8;
constexpr int DIM = 8192, KL = 1024;
constexpr unsigned KT_B = 32768, HT_B = 16384;
constexpr unsigned SLOT_B = 2 * KT_B;
constexpr unsigned PANEL_B = 32 * SLOT_B;
constexpr unsigned STAGE_B = 32 * PANEL_B;
constexpr unsigned TILE_B = 131072;
constexpr unsigned PIN_OFF = STAGE_B;
constexpr unsigned POUT_OFF = PIN_OFF + 128 * TILE_B;
constexpr unsigned SEND_OFF = POUT_OFF + 128 * TILE_B;
constexpr unsigned SCR_OFF = SEND_OFF + 2048 * HT_B;
constexpr unsigned DATA_B = SCR_OFF + 256 * TILE_B;
constexpr unsigned CNT_OFF = DATA_B;
constexpr unsigned PF_OFF = DATA_B + 4096;
constexpr unsigned IR_OFF = DATA_B + 8192;
constexpr unsigned PL_OFF = DATA_B + 16384;
constexpr unsigned SF_OFF = DATA_B + 20480;
constexpr unsigned FL_OFF = DATA_B + 32768;
constexpr unsigned HV_OFF = DATA_B + 65536;
constexpr unsigned DONE_OFF = DATA_B + 98304;
constexpr unsigned TAIL_OFF = DONE_OFF + 1024;
constexpr unsigned CALL_OFF = DONE_OFF + 2048;
constexpr unsigned TMO_OFF = CALL_OFF + 128;
constexpr unsigned HEAD_OFF = DONE_OFF + 4096;
constexpr size_t WS_NEED = (size_t)DONE_OFF + 8192;
constexpr unsigned OUT_B = (unsigned)DIM * DIM * 2;
#define LINK_WGS 8
#define SENDERS 2
constexpr int NE = LINK_WGS;
constexpr int NS = SENDERS;
constexpr int NLINK = 8 * NE, NGEMM = 256 - NLINK;
constexpr int NURG = NGEMM / 4;
constexpr int LDS_B = 131072 + 256;
constexpr unsigned long long DEADLINE = 100000000ull;
constexpr int AUX_SYS = 17;

#define WIN 3
#define STAGE_AUX 16
#define ROUND_SWAP 1
#define RT_DELAY 8
#define URG0 48
#define URG_DELAY 2
#define URG_STEP 1
#define RT_STEP 1
#define NLATE 96
#define FLEX_AT 32
#define PF_SLEEP 127
#define SCHED_SLEEP 4
#define TAILN 64
#define NSPLIT 40
#define TAILWIN 6
#define PUSH_WIN 3
#define ST_WIN 16
#define POLL_SLEEP 32

struct Params {
    const char* x; const char* w;
    char* out[NR]; char* ws[NR];
    int rank;
    unsigned units[32];
    unsigned late[NGEMM / 4];
    unsigned tiles[NGEMM];
};

#define STR2(x) #x
#define STR(x) STR2(x)
#define WAIT_V(n) asm volatile("s_waitcnt vmcnt(" STR(n) ")" ::: "memory")
#define WAIT_L(n) asm volatile("s_waitcnt lgkmcnt(" STR(n) ")" ::: "memory")
#define BAR __builtin_amdgcn_s_barrier()
#define SCHED __builtin_amdgcn_sched_barrier(0)

#define STAMPV(i, v) do {} while (0)
#define STAMP(i) do {} while (0)

__device__ __forceinline__ unsigned ld_sys(const char* p) {
    return __hip_atomic_load((const GAS unsigned*)p, __ATOMIC_RELAXED, __HIP_MEMORY_SCOPE_SYSTEM);
}
__device__ __forceinline__ void st_sys(char* p, unsigned v) {
    __hip_atomic_store((GAS unsigned*)p, v, __ATOMIC_RELAXED, __HIP_MEMORY_SCOPE_SYSTEM);
}
__device__ __forceinline__ void st8_sys(char* p, unsigned v) {
    __hip_atomic_store((GAS unsigned char*)p, (unsigned char)v, __ATOMIC_RELAXED, __HIP_MEMORY_SCOPE_SYSTEM);
}
__device__ __forceinline__ unsigned call_tag(unsigned call) { return call % 255u + 1u; }
__device__ __forceinline__ void add_sys(char* p, unsigned v) {
    __hip_atomic_fetch_add((GAS unsigned*)p, v, __ATOMIC_RELAXED, __HIP_MEMORY_SCOPE_SYSTEM);
}
__device__ __forceinline__ unsigned cvt_pk_bf16(float lo, float hi) {
    unsigned r; asm volatile("v_cvt_pk_bf16_f32 %0, %1, %2" : "=v"(r) : "v"(lo), "v"(hi)); return r;
}
__device__ __forceinline__ int panel_row(int d, int P) { return ((d >> 1) & 1) * 4096 + ((((P >> 3) ^ (d >> 2) ^ 1) & 1) * 8 + (P & 7)) * 256; }
__device__ __forceinline__ int panel_col(int d, int k) { return (d & 1) * 4096 + k * 256; }
__device__ __forceinline__ int link_slot(int dsel, int cc) { return 4 + (((cc & 1) ^ ROUND_SWAP) * 2 + (cc >> 1)) * 6 + (dsel < 4 ? dsel - 1 : dsel - 2); }
__device__ __forceinline__ bool give_up(char* tmo, unsigned call, unsigned long long t0) {
    if (__builtin_amdgcn_readfirstlane(__hip_atomic_load((const GAS unsigned*)tmo, __ATOMIC_RELAXED, __HIP_MEMORY_SCOPE_AGENT)) == call
        || __builtin_amdgcn_s_memrealtime() - t0 > DEADLINE) {
        __hip_atomic_store((GAS unsigned*)tmo, call, __ATOMIC_RELAXED, __HIP_MEMORY_SCOPE_AGENT);
        return true;
    }
    return false;
}

struct Piece { const char* src; unsigned sA, sB, dst, pub; int kind; };

__device__ __forceinline__ Piece shard_piece(const Params& p, int d, int panel, int cc, int pc, int dp, int slot) {
    const int k0 = ((d >> 2) * 4 + cc) * 128 + (pc >> 1) * 64, h = pc & 1;
    Piece o; o.kind = panel >= 16;
    o.dst = (unsigned)dp * PANEL_B + (unsigned)slot * SLOT_B + (unsigned)pc * HT_B;
    o.pub = CNT_OFF + (unsigned)dp * 128 + (unsigned)slot * 4 + (unsigned)pc;
    if (o.kind) {
        o.src = p.w + ((size_t)k0 * DIM + panel_col(d, panel - 16) + h * 128) * 2;
        o.sA = 8u * DIM * 2; o.sB = 4u * DIM * 2;
    } else {
        o.src = p.x + ((size_t)(panel_row(d, panel) + h * 128) * KL + k0) * 2;
        o.sA = 16u * KL * 2; o.sB = 8u * KL * 2;
    }
    return o;
}
__device__ __forceinline__ int unit_at(const Params& p, int n) { return (p.units[n >> 2] >> ((n & 3) * 8)) & 255; }
__device__ __forceinline__ unsigned send_off(int d, int panel, int cc, int pc) {
    const int gp = panel >= 16 ? 32 + (panel_col(d, panel - 16) >> 8) : panel_row(d, panel) >> 8;
    return SEND_OFF + (unsigned)((gp * 8 + (d >> 2) * 4 + cc) * 4 + pc) * HT_B;
}
__device__ __forceinline__ Piece pull_piece(const Params& p, int code, int dsel) {
    const int cls = code >> 21, pos = (code >> 10) & 2047, id = code & 1023, s = p.rank ^ dsel;
    Piece o; o.kind = 2; o.sA = 2048; o.sB = 1024;
    if (cls == 1) {
        const int u = unit_at(p, (pos >> 2) & 127), panel = u & 31, cc = u >> 5, pc = pos & 3, slot = link_slot(dsel, cc);
        o.src = p.ws[s] + send_off(p.rank, panel, cc, pc);
        o.dst = (unsigned)panel * PANEL_B + (unsigned)slot * SLOT_B + (unsigned)pc * HT_B;
        o.pub = CNT_OFF + (unsigned)panel * 128 + (unsigned)slot * 4 + (unsigned)pc;
    } else if (cls == 0) {
        o.src = p.ws[s] + POUT_OFF + (size_t)id * HT_B;
        o.dst = PIN_OFF + (unsigned)id * HT_B;
        o.pub = PF_OFF + (unsigned)id * 4;
    } else {
        const int ow = cls == 2 ? s : p.rank ^ 4;
        o.kind = 3; o.sA = 4u * DIM * 2; o.sB = 2u * DIM * 2;
        o.dst = (unsigned)(panel_row(ow, 8 + (id >> 7)) + (id & 7) * 32) * (DIM * 2) + (unsigned)panel_col(ow, (id >> 3) & 15) * 2;
        o.src = p.out[s] + o.dst;
        o.pub = cls == 2 ? (unsigned)(code & 0xfffff) + 1u : 0u;
    }
    return o;
}
__device__ __forceinline__ const char* piece_src(const Piece& pc, int lane) {
    const unsigned xl = (unsigned)(lane >> 3) * (KL * 2) + (unsigned)((lane & 7) ^ (lane >> 3)) * 16;
    const unsigned wl = (unsigned)(lane >> 4) * (DIM * 2) + (unsigned)((lane & 15) ^ ((lane >> 4) << 2)) * 16;
    const unsigned ol = (unsigned)(lane >> 5) * (DIM * 2) + (unsigned)(lane & 31) * 16;
    return pc.src + (pc.kind == 0 ? xl : pc.kind == 1 ? wl : pc.kind == 2 ? (unsigned)lane * 16 : ol);
}
__device__ __forceinline__ void piece_dma2(const Piece& pc, const char* s, LAS char* buf, int i) {
    __builtin_amdgcn_global_load_lds((const GAS unsigned*)(s + (size_t)i * pc.sA), (LAS unsigned*)(buf + i * 2048), 16, 0, 16);
    __builtin_amdgcn_global_load_lds((const GAS unsigned*)(s + (size_t)i * pc.sA + pc.sB), (LAS unsigned*)(buf + i * 2048 + 1024), 16, 0, 16);
}
__device__ __forceinline__ void piece_dma(const Piece& pc, LAS char* buf, int lane) {
    const char* s = piece_src(pc, lane);
#pragma unroll
    for (int i = 0; i < 8; ++i) piece_dma2(pc, s, buf, i);
}
__device__ __forceinline__ void piece_read_tr(u32x4 (&v)[16], LAS char* buf, int lane) {
    const int q = (lane >> 2) & 3;
    const unsigned trl = (unsigned)(8 * (lane >> 4) + q) * 256 + (unsigned)(lane & 3) * 16;
#pragma unroll
    for (int f = 0; f < 16; ++f) {
        const int nb = f >> 1, kb = f & 1;
        LAS char* a = buf + trl + (((nb >> 1) ^ q) * 64) + kb * 8192 + (nb & 1) * 8;
        const u32x2 lo = __builtin_bit_cast(u32x2, __builtin_amdgcn_ds_read_tr16_b64_v4i16((LAS s16x4*)a));
        const u32x2 hi = __builtin_bit_cast(u32x2, __builtin_amdgcn_ds_read_tr16_b64_v4i16((LAS s16x4*)(a + 1024)));
        v[f] = (u32x4){lo.x, lo.y, hi.x, hi.y};
    }
}
__device__ __forceinline__ void piece_read(u32x4 (&v)[16], bool isx, LAS char* buf, int lane) {
    const unsigned xb = (unsigned)(lane & 15) * 128 + (unsigned)((lane >> 4) ^ (lane & 7)) * 16;
    LAS char* b0 = buf + (isx ? xb : (unsigned)lane * 16);
    LAS char* b1 = buf + (isx ? xb ^ 64u : (unsigned)lane * 16 + 1024u);
#pragma unroll
    for (int f = 0; f < 16; ++f) asm volatile("ds_read_b128 %0, %1 offset:%2" : "=v"(v[f]) : "v"((f & 1) ? b1 : b0), "i"((f >> 1) * 2048) : "memory");
    asm volatile("s_waitcnt lgkmcnt(0)" : "+v"(v[0]), "+v"(v[1]), "+v"(v[2]), "+v"(v[3]), "+v"(v[4]), "+v"(v[5]), "+v"(v[6]), "+v"(v[7]),
                 "+v"(v[8]), "+v"(v[9]), "+v"(v[10]), "+v"(v[11]), "+v"(v[12]), "+v"(v[13]), "+v"(v[14]), "+v"(v[15]) :: "memory");
}

#define LOOKAHEAD 3
#define LOOKAHEAD3 2
constexpr int KSL = LOOKAHEAD, KS3 = LOOKAHEAD3;
constexpr int NMB = 2 * KSL + KS3;

__device__ __forceinline__ int next_task(LAS unsigned* mb, int lane) {
    for (;;) {
        unsigned v;
        asm volatile("ds_read_b32 %0, %1\n\ts_waitcnt lgkmcnt(0)" : "=&v"(v) : "v"(mb + (lane & 15)) : "memory");
        unsigned best = 0xffffffffu; int bl = 0;
#pragma unroll
        for (int l = 0; l < NMB; ++l) { const unsigned k = (unsigned)__builtin_amdgcn_readlane(v, l) - 1u; if (k < best) { best = k; bl = l; } }
        if (best == 0xffffffffu) return __builtin_amdgcn_readlane(v, NMB) ? -2 : -1;
        unsigned old = 0;
        if (lane == 0) asm volatile("ds_cmpst_rtn_b32 %0, %1, %2, %3\n\ts_waitcnt lgkmcnt(0)" : "=&v"(old) : "v"(mb + bl), "v"(best + 1u), "v"(0u) : "memory");
        if ((unsigned)__builtin_amdgcn_readfirstlane(old) == best + 1u) return (int)(best & 0x7fffffu);
    }
}
static_assert(WIN >= 2 && WIN <= 8 && NS >= 1 && NS <= 3, "window / pullers");
__device__ __forceinline__ void lds_wait_zero(LAS unsigned* a) {
    for (;;) {
        unsigned f;
        asm volatile("ds_read_b32 %0, %1\n\ts_waitcnt lgkmcnt(0)" : "=&v"(f) : "v"(a) : "memory");
        if (__builtin_amdgcn_readfirstlane(f) == 0) break;
        __builtin_amdgcn_s_sleep(1);
    }
}

__device__ __forceinline__ void link_wg(const Params& p, LAS char* lds, int dsel, int e) {
    const int wid = __builtin_amdgcn_readfirstlane(threadIdx.x >> 6), lane = threadIdx.x & 63;
    char* ws = p.ws[p.rank];
    LAS unsigned* mb = (LAS unsigned*)(lds + 7 * HT_B);
    const int sn = wid < NS ? wid : wid - NS;
    LAS char* buf = lds + sn * 2 * HT_B;
    LAS unsigned* slot = mb + 16 + sn * 8;
    STAMPV(0, dsel);
    Piece first;
    if (wid >= NS && wid < 2 * NS) {
        if (dsel != 4) {
            const int q = sn * NE + e, u = unit_at(p, q >> 2);
            first = shard_piece(p, p.rank ^ dsel, u & 31, u >> 5, q & 3, u & 31, link_slot(dsel, u >> 5));
            piece_dma(first, buf + HT_B, lane);
        }
    } else if (wid == 7) {
        const unsigned c = ld_sys(ws + CALL_OFF) + 1u;
        mb[lane] = lane == 15 ? c : (lane >= 16 && (lane & 7) == 4 && dsel != 4) ? 14u : 0u; WAIT_L(0);
        if (lane == 0) add_sys(ws + DONE_OFF, 1u);
    }
    BAR; asm volatile("" ::: "memory");
    unsigned call;
    asm volatile("ds_read_b32 %0, %1 offset:60\n\ts_waitcnt lgkmcnt(0)" : "=&v"(call) : "v"(mb) : "memory");
    call = __builtin_amdgcn_readfirstlane(call);
    if (wid >= 2 * NS) {
        if (wid != 7) return;
        const int cls = lane < KSL ? (dsel == 4 ? 0 : 1) : lane < 2 * KSL ? 2 : 3;
        const unsigned nfirst = cls == 1 ? (NS + KSL) * NE : 0u;
        const unsigned n = cls == 1 ? 512u : 1024u;
        const unsigned base = (call - 1u) * (n - nfirst + NE * (cls == 3 ? 7 * KS3 : KSL)) - nfirst;
        char* head = ws + HEAD_OFF + (cls == 3 ? 0 : dsel * 2 + (cls >> 1)) * 128;
        const char* list = (cls == 3 && dsel != 4) ? p.ws[p.rank ^ dsel] + HV_OFF + (dsel ^ 4) * 4096
                                                     : ws + (cls == 0 ? PL_OFF : cls == 1 ? IR_OFF + dsel * 1024 : FL_OFF + dsel * 4096);
        const unsigned tag = call & 0xfffffu;
        int st = (lane >= NMB || (cls == 2 && dsel == 4)) ? 3 : (cls == 3 && dsel != 4) ? 4 : 0;
        unsigned pos = (NS + lane) * NE + e, lo = pos & ~3u;
        if (cls == 1) st = 1;
        const unsigned long long t0 = __builtin_amdgcn_s_memrealtime();
        for (unsigned polls = 1;; ++polls) {
            if (st == 2 && __hip_atomic_load(mb + lane, __ATOMIC_RELAXED, __HIP_MEMORY_SCOPE_WORKGROUP) == 0u) st = 0;
            unsigned t = 0, en = 0;
            if (st == 0) t = __hip_atomic_fetch_add((GAS unsigned*)head, 1u, __ATOMIC_RELAXED, __HIP_MEMORY_SCOPE_AGENT);
            if (st == 1) en = ld_sys(list + lo);
            if (st == 0) {
                pos = t - base; st = pos >= n ? 3 : 1;
                lo = cls != 1 ? pos * 4 : (pos < 4 * NURG ? pos & ~3u : 4u * NURG);
            } else if (st == 1 && (en >> 12) == tag) {
                __hip_atomic_store(mb + lane, ((((unsigned)cls << 11 | pos) << 10) | (en & 1023u)) + 1u, __ATOMIC_RELAXED, __HIP_MEMORY_SCOPE_WORKGROUP);
                st = 2;
            }
            if (__any(cls == 2 && st != 0 && pos >= 1024u - FLEX_AT) && st == 4) st = 0;
            if (__all(st == 3)) break;
            if ((polls & 63u) == 0u && give_up(ws + TMO_OFF, call, t0)) {
                if (lane == 0) st_sys(p.out[p.rank] + ((size_t)panel_row(p.rank, 8) * DIM + (size_t)panel_col(p.rank, 0)) * 2, 0x7fc07fc0u);
                break;
            }
            __builtin_amdgcn_s_sleep(SCHED_SLEEP);
        }
        if (lane == 0) __hip_atomic_store(mb + NMB, 1u, __ATOMIC_RELAXED, __HIP_MEMORY_SCOPE_WORKGROUP);
        return;
    }
    if (wid < NS) {
        int k = 0, code = -1, si = 1, waited = 0; (void)si; (void)waited;
        bool pend = false;
        u32x4 desc = {0u, 0u, 0u, 0u};
        Piece cur;
        for (;;) {
            if (code < 0) {
                if (code == -1) code = next_task(mb, lane);
                if (code < 0) {
                    if (pend) { WAIT_V(0); if (lane == 0) asm volatile("ds_write_b128 %0, %1" :: "v"(slot + ((k - 1) & 1) * 4), "v"(desc) : "memory"); pend = false; }
                    while (code == -1) { __builtin_amdgcn_s_sleep(2); code = next_task(mb, lane); }
                    if (code < 0) break;
                    waited = 8;
                }
                cur = pull_piece(p, code, dsel);
            }
            if (k == 0) { STAMP(127); }
            lds_wait_zero(slot + (k & 1) * 4);
            LAS char* b = buf + (k & 1) * HT_B;
            const char* sp = piece_src(cur, lane);
            const unsigned stride = cur.sB;
            const u32x4 ndesc = {(unsigned)cur.kind + 1u, cur.dst, cur.pub, 0u};
            const int cls0 = code >> 21; (void)cls0;
            const bool fast = (code >> 10) >= (3 << 11 | (1024 - TAILN));
#pragma unroll
            for (int f = 0; f < 16; ++f) {
                __builtin_amdgcn_global_load_lds((const GAS unsigned*)sp, (LAS unsigned*)(b + f * 1024), 16, 0, AUX_SYS);
                sp += stride;
                asm volatile("" ::: "memory");
                if (f == 10) { code = next_task(mb, lane); if (code >= 0) cur = pull_piece(p, code, dsel); }
                if (f > WIN - 2 && fast) WAIT_V(TAILWIN - 1); else WAIT_V(WIN - 1);
                if (f == WIN - 2 && pend && lane == 0)
                    asm volatile("ds_write_b128 %0, %1" :: "v"(slot + ((k - 1) & 1) * 4), "v"(desc) : "memory");
            }
            desc = ndesc; pend = true; ++k;
            if (si < 127) { STAMPV(si, (cls0 == 1 ? 1 : cls0 == 0 ? 3 : cls0 == 2 ? 4 : 2) | waited); } ++si; waited = 0;
        }
        lds_wait_zero(slot + (k & 1) * 4);
        if (lane == 0) asm volatile("ds_write_b32 %0, %1" :: "v"(slot + (k & 1) * 4), "v"(15u) : "memory");
        return;
    }
    if (dsel != 4) {
        char* dws = p.ws[p.rank ^ dsel];
        const __amdgpu_buffer_rsrc_t rs = __builtin_amdgcn_make_buffer_rsrc(dws, 0, STAGE_B, 0x00020000);
        STAMP(4);
        WAIT_V(0);
        STAMP(2);
        u32x4 v[16];
        if (first.kind) { piece_read_tr(v, buf + HT_B, lane); WAIT_L(0);
#pragma unroll
            for (int f = 0; f < 16; ++f) asm volatile("" : "+v"(v[f]));
        } else piece_read(v, true, buf + HT_B, lane);
        if (lane == 0) asm volatile("ds_write_b32 %0, %1" :: "v"(slot + 4), "v"(0u) : "memory");
#pragma unroll
        for (int f = 0; f < 16; ++f) { __builtin_amdgcn_raw_buffer_store_b128(v[f], rs, first.dst + lane * 16 + f * 1024, 0, AUX_SYS); WAIT_V(PUSH_WIN - 1); }
        WAIT_V(0);
        if (lane == 0) st8_sys(dws + first.pub, call_tag(call));
        STAMP(3);
    }
    const __amdgpu_buffer_rsrc_t rsw = __builtin_amdgcn_make_buffer_rsrc(ws, 0, POUT_OFF, 0x00020000);
    const __amdgpu_buffer_rsrc_t rso = __builtin_amdgcn_make_buffer_rsrc(p.out[p.rank], 0, OUT_B, 0x00020000);
    const unsigned ol = (unsigned)(lane >> 5) * (DIM * 2) + (unsigned)(lane & 31) * 16;
    char* fwd = ws + HV_OFF + dsel * 4096;
    for (int k = 0;;) {
        u32x4 ds4;
        asm volatile("ds_read_b128 %0, %1\n\ts_waitcnt lgkmcnt(0)" : "=&v"(ds4) : "v"(slot + (k & 1) * 4) : "memory");
        const int kind = __builtin_amdgcn_readfirstlane(ds4.x) - 1;
        if (kind > 3) break;
        if (kind < 0) { __builtin_amdgcn_s_sleep(1); continue; }
        const unsigned dst = __builtin_amdgcn_readfirstlane(ds4.y), pub = __builtin_amdgcn_readfirstlane(ds4.z);
        u32x4 v[16];
        piece_read(v, false, buf + (k & 1) * HT_B, lane);
        if (lane == 0) asm volatile("ds_write_b32 %0, %1" :: "v"(slot + (k & 1) * 4), "v"(0u) : "memory");
        const bool o = kind == 3;
        const __amdgpu_buffer_rsrc_t rs = o ? rso : rsw;
        const unsigned ds = o ? 2u * DIM * 2 : 1024u;
        unsigned vo = dst + (o ? ol : (unsigned)lane * 16);
#pragma unroll
        for (int f = 0; f < 16; ++f) {
            __builtin_amdgcn_raw_buffer_store_b128(v[f], rs, vo, 0, AUX_SYS);
            vo += ds;
            WAIT_V(ST_WIN - 1);
        }
        if (pub) {
            WAIT_V(0);
            if (lane == 0) {
                if (o) st_sys(fwd + ((pub - 1u) >> 10) * 4, (call & 0xfffffu) << 12 | ((pub - 1u) & 1023u));
                else if (pub >= PF_OFF) st_sys(ws + pub, call);
                else st8_sys(ws + pub, call_tag(call));
            }
        }
        ++k;
    }
    STAMP(1);
}

__device__ __forceinline__ int ready_prefix(unsigned v, unsigned target, unsigned low) {
    const unsigned long long m = __ballot(v == target);
    const unsigned both = ((unsigned)m & (unsigned)(m >> 32)) | low;
    return both == 0xffffffffu ? 32 : __builtin_ctz(~both);
}
__device__ __forceinline__ void kn_put(LAS unsigned* kn, unsigned v) { asm volatile("ds_write_b32 %0, %1" :: "v"(kn), "v"(v) : "memory"); }
__device__ __noinline__ int wait_slot(const char* cptr, char* tmo, unsigned call, int need, unsigned low, LAS unsigned* kn, int job, int duty) {
    int known;
    if (__builtin_amdgcn_readfirstlane(duty)) {
        const unsigned long long t0 = __builtin_amdgcn_s_memrealtime();
        for (unsigned polls = 1;; ++polls) {
            known = ready_prefix(ld_sys(cptr), call_tag(call) * 0x01010101u, low);
            if (known > need) break;
            if ((polls & 63u) == 0u && give_up(tmo, call, t0)) { known = 32; break; }
            __builtin_amdgcn_s_sleep(POLL_SLEEP);
        }
        kn_put(kn, (unsigned)(job + known));
    } else {
        for (;;) {
            unsigned f;
            asm volatile("ds_read_b32 %0, %1\n\ts_waitcnt lgkmcnt(0)" : "=&v"(f) : "v"(kn) : "memory");
            known = (int)f - job;
            if (known > need) break;
            __builtin_amdgcn_s_sleep(4);
        }
    }
    return known;
}

__device__ __forceinline__ void gemm_wg(const Params& p, LAS char* lds, int gb) {
    const int tid = threadIdx.x, wid = __builtin_amdgcn_readfirstlane(tid >> 6), lane = tid & 63, wr = wid >> 2, wc = wid & 3;
    char* ws = p.ws[p.rank];
    STAMP(0);
    const unsigned call = __builtin_amdgcn_readfirstlane(ld_sys(ws + CALL_OFF)) + 1u;
    const int late = (int)(signed char)((p.late[gb >> 2] >> ((gb & 3) * 8)) & 255);
    const bool bulk = wid >= 4;
    LAS unsigned* ucnt = (LAS unsigned*)(lds + 131072);
    if (tid < 2) ucnt[tid] = 0u;
    WAIT_L(0); BAR;
    for (int i = bulk ? (late >= 0 ? RT_DELAY + (wid - 4) * RT_STEP : 0) : (gb < URG0 ? 0 : URG_DELAY + ((gb - URG0) >> 5) * URG_STEP); i > 0; --i) __builtin_amdgcn_s_sleep(32);
    unsigned nbulk = 0;
    for (int r = !bulk ? gb * 4 + wid : late >= 0 ? 4 * NGEMM + (wid - 4) * NLATE + late : 2048; r < 2048; r = bulk ? r + 4 * NLATE : 2048) {
        const int other = (r >> 3) & 1, half = (r >> 2) & 1, pcs = r & 3;
        const int u = unit_at(p, r >> 4), panel = u & 31, cc = u >> 5;
        const int d0 = (((p.rank >> 2) ^ other) << 2) + (panel >= 16 ? half : half * 2), d1 = d0 + (panel >= 16 ? 2 : 1);
        const bool own0 = ((d0 ^ p.rank) & 3) == 0, own1 = ((d1 ^ p.rank) & 3) == 0, own = own0 || own1;
        const Piece pc = shard_piece(p, d0, panel, cc, pcs, panel < 16 && other ? panel ^ 8 : panel, other * 28 + cc);
        LAS char* buf = lds + wid * HT_B;
        piece_dma(pc, buf, lane);
        WAIT_V(0);
        u32x4 v[16];
        if (pc.kind) piece_read_tr(v, buf, lane); else piece_read(v, true, buf, lane);
        const __amdgpu_buffer_rsrc_t rs = __builtin_amdgcn_make_buffer_rsrc(ws, 0, DATA_B, 0x00020000);
        const unsigned so = send_off(d0, panel, cc, pcs);
#pragma unroll
        for (int f = 0; f < 16; ++f) __builtin_amdgcn_raw_buffer_store_b128(v[f], rs, so + lane * 16 + f * 1024, 0, AUX_SYS);
        if (own) {
#pragma unroll
            for (int f = 0; f < 16; ++f) __builtin_amdgcn_raw_buffer_store_b128(v[f], rs, pc.dst + lane * 16 + f * 1024, 0, AUX_SYS);
        }
        WAIT_V(0);
        if (!bulk) { STAMP(28); }
        ++nbulk;
        if (lane == 0) {
            if (own) st8_sys(ws + pc.pub, call_tag(call));
            if (!bulk && __hip_atomic_fetch_add(ucnt, 1u, __ATOMIC_RELAXED, __HIP_MEMORY_SCOPE_WORKGROUP) == 3u) {
                if (!own0) st_sys(p.ws[d0] + IR_OFF + (p.rank ^ d0) * 1024 + (r >> 4) * 4, (call & 0xfffffu) << 12);
                if (!own1) st_sys(p.ws[d1] + IR_OFF + (p.rank ^ d1) * 1024 + (r >> 4) * 4, (call & 0xfffffu) << 12);
            }
        }
    }
    if (bulk && nbulk && lane == 0 &&
        __hip_atomic_fetch_add((GAS unsigned*)(ws + TAIL_OFF + 256), nbulk, __ATOMIC_RELAXED, __HIP_MEMORY_SCOPE_AGENT) + nbulk == call * (2048u - 4u * NGEMM)) {
#pragma unroll
        for (int q = 1; q < NR; ++q) if (q != 4) st_sys(p.ws[p.rank ^ q] + IR_OFF + q * 1024 + NURG * 4, (call & 0xfffffu) << 12);
    }
    BAR;
    if (tid == 0) add_sys(ws + DONE_OFF, 1u);
    STAMP(1);
    const unsigned target = call_tag(call) * 0x01010101u;
    const unsigned voff = (unsigned)tid * 16;
    const unsigned ldsw = (unsigned)wid * 1024u;
    const int aoff = wr * 8192 + lane * 16, boff = wc * 4096 + lane * 16;
#define SA(b, h) (((b) * 2 + (h)) * HT_B)
#define SB(b, h) ((4 + (b) * 2 + (h)) * HT_B)
#define STAGE(bufoff, gbase) do { _Pragma("unroll") for (int _i = 0; _i < 2; ++_i) \
        __builtin_amdgcn_global_load_lds((const GAS unsigned*)((const char*)(gbase) + voff + _i * 8192), (LAS unsigned*)(lds + (bufoff) + ldsw + _i * 8192), 16, 0, STAGE_AUX); } while (0)
#define LDA(dst, b, h) do { _Pragma("unroll") for (int m = 0; m < 4; ++m) _Pragma("unroll") for (int k = 0; k < 2; ++k) dst[m][k] = *(const LAS bf16x8*)(lds + SA(b, h) + aoff + m * 2048 + k * 1024); } while (0)
#define LDB(dst, b, h) do { _Pragma("unroll") for (int n = 0; n < 2; ++n) _Pragma("unroll") for (int k = 0; k < 2; ++k) dst[n][k] = *(const LAS bf16x8*)(lds + SB(b, h) + boff + n * 2048 + k * 1024); } while (0)
#define MMA(ai, bj, At, Bt) do { __builtin_amdgcn_s_setprio(1); _Pragma("unroll") for (int m = 0; m < 4; ++m) _Pragma("unroll") for (int n = 0; n < 2; ++n) _Pragma("unroll") for (int k = 0; k < 2; ++k) \
        acc[ai][bj][m][n] = __builtin_amdgcn_mfma_f32_16x16x32_bf16(Bt[n][k], At[m][k], acc[ai][bj][m][n], 0, 0, 0); __builtin_amdgcn_s_setprio(0); } while (0)
    for (int rnd = 0; rnd < 2; ++rnd) {
        const int code = (p.tiles[gb] >> (16 * rnd)) & 0xffff;
        if (code == 0xffff) break;
        const int pm = (code >> 4) & 15, pn = code & 15, part = code >> 8;
        const int Tb = part == 2 ? 16 : 0, Te = part == 1 ? 16 : 32;
        const unsigned low = part == 2 ? 0xffffu : 0u;
        const char* Ab = ws + (size_t)pm * PANEL_B;
        const char* Bb = ws + (size_t)(16 + pn) * PANEL_B;
        const unsigned coff = CNT_OFF + (unsigned)(lane < 32 ? pm : 16 + pn) * 128 + (unsigned)(lane & 31) * 4;
        f32x4 acc[2][2][4][2];
#pragma unroll
        for (int a = 0; a < 2; ++a)
#pragma unroll
            for (int b = 0; b < 2; ++b)
#pragma unroll
                for (int m = 0; m < 4; ++m)
#pragma unroll
                    for (int n = 0; n < 2; ++n) acc[a][b][m][n] = (f32x4){0.f, 0.f, 0.f, 0.f};
        bf16x8 At[4][2], B0[2][2], B1[2][2];
        int known = __builtin_amdgcn_readfirstlane(wait_slot(ws + coff, ws + TMO_OFF, call, Tb, low, ucnt + 1, rnd * 64, wid == 0));
        STAMP(2 + rnd * 12);
        {
            const char* a0 = Ab + (unsigned)Tb * SLOT_B; const char* b0 = Bb + (unsigned)Tb * SLOT_B;
            STAGE(SB(0, 0), b0); STAGE(SA(0, 0), a0); STAGE(SB(0, 1), b0 + HT_B); STAGE(SA(0, 1), a0 + HT_B);
            if (wr == 1) BAR;
            WAIT_V(4); BAR;
            STAGE(SB(1, 0), b0 + KT_B); STAGE(SA(1, 0), a0 + KT_B); STAGE(SB(1, 1), b0 + KT_B + HT_B);
            WAIT_V(6); BAR;
        }
        for (int T = Tb; T < Te; ++T) {
            if (T + 1 < Te && known <= T + 1) known = __builtin_amdgcn_readfirstlane(wait_slot(ws + coff, ws + TMO_OFF, call, T + 1, low, ucnt + 1, rnd * 64, wid == 0));
            const unsigned o1 = (unsigned)T * SLOT_B, o2 = (unsigned)(T + 1 < Te ? T + 1 : T) * SLOT_B;
            const char* a1 = Ab + o1 + KT_B; const char* a2 = Ab + o2; const char* b2 = Bb + o2;
            const char* a3 = a2 + KT_B; const char* b3 = b2 + KT_B;
            unsigned pv = 0;
            const bool ask = known < Te;
            LDB(B0, 0, 0); SCHED; LDA(At, 0, 0);
            if (ask) asm volatile("global_load_dword %0, %1, %2 sc0 sc1" : "+v"(pv) : "v"(coff), "s"(ws) : "memory");
            STAGE(SA(1, 1), a1 + HT_B);
            WAIT_L(8); BAR; WAIT_L(0); MMA(0, 0, At, B0); BAR; SCHED;
            LDB(B1, 0, 1); STAGE(SB(0, 0), b2);
            BAR; WAIT_L(0); MMA(0, 1, At, B1); BAR;
            LDA(At, 0, 1); STAGE(SA(0, 0), a2);
            BAR; WAIT_L(0); MMA(1, 0, At, B0); BAR; SCHED;
            STAGE(SB(0, 1), b2 + HT_B);
            WAIT_V(6);
            asm volatile("" : "+v"(pv));
            { const int k2 = ready_prefix(pv, target, low); if (k2 > known) { known = k2; if (wid == 0) kn_put(ucnt + 1, (unsigned)(rnd * 64 + k2)); } }
            BAR; MMA(1, 1, At, B1); BAR;
            LDB(B0, 1, 0); SCHED; LDA(At, 1, 0); STAGE(SA(0, 1), a2 + HT_B);
            WAIT_L(8); BAR; WAIT_L(0); MMA(0, 0, At, B0); BAR; SCHED;
            LDB(B1, 1, 1); STAGE(SB(1, 0), b3);
            BAR; WAIT_L(0); MMA(0, 1, At, B1); BAR;
            LDA(At, 1, 1); STAGE(SA(1, 0), a3);
            BAR; WAIT_L(0); MMA(1, 0, At, B0); BAR; SCHED;
            STAGE(SB(1, 1), b3 + HT_B);
            WAIT_V(6); BAR; MMA(1, 1, At, B1); BAR;
        }
        WAIT_V(0);
        if (wr == 0) BAR;
        BAR;
        STAMP(11 + rnd * 12);
        int tz = threadIdx.x; asm volatile("" : "+v"(tz));
        const int ln = tz & 63;
        const int id = ((pm & 7) * 16 + pn) * 8 + wid;
        if (part == 1) {
            const __amdgpu_buffer_rsrc_t rq = __builtin_amdgcn_make_buffer_rsrc(ws + SCR_OFF, 0, 256 * TILE_B, 0x00020000);
            const unsigned base = (unsigned)id * (2 * HT_B) + (unsigned)ln * 16;
#pragma unroll
            for (int f = 0; f < 32; ++f) __builtin_amdgcn_raw_buffer_store_b128(__builtin_bit_cast(u32x4, acc[f >> 4][(f >> 1) & 1][(f >> 2) & 3][f & 1]), rq, base + f * 1024, 0, AUX_SYS);
            WAIT_V(0);
            if (ln == 0) st_sys(ws + SF_OFF + id * 4, call);
            STAMP(12 + rnd * 12);
            continue;
        }
        if (pm < 8) {
            if (part) {
                const unsigned long long t0 = __builtin_amdgcn_s_memrealtime();
                for (unsigned polls = 1;; ++polls) {
                    if ((int)(__builtin_amdgcn_readfirstlane(ld_sys(ws + SF_OFF + id * 4)) - call) >= 0) break;
                    if ((polls & 63u) == 0u && give_up(ws + TMO_OFF, call, t0)) break;
                    __builtin_amdgcn_s_sleep(8);
                }
                asm volatile("" ::: "memory");
            }
            unsigned pos = 0;
            if (tz == 0) pos = __hip_atomic_fetch_add((GAS unsigned*)(ws + TAIL_OFF), 8u, __ATOMIC_RELAXED, __HIP_MEMORY_SCOPE_AGENT);
            const bool bad = __builtin_amdgcn_readfirstlane(__hip_atomic_load((const GAS unsigned*)(ws + TMO_OFF), __ATOMIC_RELAXED, __HIP_MEMORY_SCOPE_AGENT)) == call;
            const __amdgpu_buffer_rsrc_t rq = __builtin_amdgcn_make_buffer_rsrc(ws + SCR_OFF, 0, part ? 256 * TILE_B : 0, 0x00020000);
            const __amdgpu_buffer_rsrc_t rp = __builtin_amdgcn_make_buffer_rsrc(ws + POUT_OFF, 0, 128 * TILE_B, 0x00020000);
            const unsigned base = (unsigned)id * HT_B + (unsigned)ln * 16;
#pragma unroll
            for (int ai = 0; ai < 2; ++ai) {
                u32x4 pr[16];
#pragma unroll
                for (int f = 0; f < 16; ++f) pr[f] = __builtin_amdgcn_raw_buffer_load_b128(rq, base + (unsigned)id * HT_B + (ai * 16 + f) * 1024, 0, AUX_SYS);
#pragma unroll
                for (int m = 0; m < 4; ++m)
#pragma unroll
                    for (int bj = 0; bj < 2; ++bj) {
                        const f32x4 v0 = acc[ai][bj][m][0] + __builtin_bit_cast(f32x4, pr[m * 4 + bj * 2]), v1 = acc[ai][bj][m][1] + __builtin_bit_cast(f32x4, pr[m * 4 + bj * 2 + 1]);
                        u32x4 w; w.x = cvt_pk_bf16(v0[0], v0[1]); w.y = cvt_pk_bf16(v0[2], v0[3]); w.z = cvt_pk_bf16(v1[0], v1[1]); w.w = cvt_pk_bf16(v1[2], v1[3]);
                        if (bad) w = (u32x4){0x7fc07fc0u, 0x7fc07fc0u, 0x7fc07fc0u, 0x7fc07fc0u};
                        __builtin_amdgcn_raw_buffer_store_b128(w, rp, base + (unsigned)((ai * 4 + m) * 2 + bj) * 1024, 0, AUX_SYS);
                    }
                asm volatile("" ::: "memory");
            }
            WAIT_V(0); BAR;
            pos = __builtin_amdgcn_readfirstlane(pos) & 1023u;
            if (tz < 8) st_sys(p.ws[p.rank ^ 4] + PL_OFF + (pos + tz) * 4, (call & 0xfffffu) << 12 | (unsigned)(id + tz));
            STAMP(12 + rnd * 12);
            continue;
        }
        {
            const unsigned long long t0 = __builtin_amdgcn_s_memrealtime();
            for (unsigned polls = 1;; ++polls) {
                if ((int)(__builtin_amdgcn_readfirstlane(ld_sys(ws + PF_OFF + id * 4)) - call) >= 0) break;
                if ((polls & 63u) == 0u && give_up(ws + TMO_OFF, call, t0)) break;
                __builtin_amdgcn_s_sleep(PF_SLEEP);
            }
            asm volatile("" ::: "memory");
            STAMP(26);
            const bool bad = __builtin_amdgcn_readfirstlane(__hip_atomic_load((const GAS unsigned*)(ws + TMO_OFF), __ATOMIC_RELAXED, __HIP_MEMORY_SCOPE_AGENT)) == call;
            const __amdgpu_buffer_rsrc_t rp = __builtin_amdgcn_make_buffer_rsrc(ws + PIN_OFF, 0, 128 * TILE_B, 0x00020000);
            const unsigned base = (unsigned)id * HT_B + (unsigned)ln * 16;
            u32x4 pr[16];
#pragma unroll
            for (int f = 0; f < 16; ++f) pr[f] = __builtin_amdgcn_raw_buffer_load_b128(rp, base + f * 1024, 0, AUX_SYS);
            const int fr = ln & 15, fq = ln >> 4;
#pragma unroll
            for (int ai = 0; ai < 2; ++ai)
#pragma unroll
                for (int m = 0; m < 4; ++m) {
                    const int row = ai * 128 + wr * 64 + m * 16 + fr;
#pragma unroll
                    for (int bj = 0; bj < 2; ++bj) {
                        const u32x4 q = pr[(ai * 4 + m) * 2 + bj];
                        const f32x4 v0 = acc[ai][bj][m][0], v1 = acc[ai][bj][m][1];
                        u32x4 w;
                        w.x = cvt_pk_bf16(v0[0] + __uint_as_float(q.x << 16), v0[1] + __uint_as_float(q.x & 0xffff0000u));
                        w.y = cvt_pk_bf16(v0[2] + __uint_as_float(q.y << 16), v0[3] + __uint_as_float(q.y & 0xffff0000u));
                        w.z = cvt_pk_bf16(v1[0] + __uint_as_float(q.z << 16), v1[1] + __uint_as_float(q.z & 0xffff0000u));
                        w.w = cvt_pk_bf16(v1[2] + __uint_as_float(q.w << 16), v1[3] + __uint_as_float(q.w & 0xffff0000u));
                        if (bad) w = (u32x4){0x7fc07fc0u, 0x7fc07fc0u, 0x7fc07fc0u, 0x7fc07fc0u};
                        const int ch = (bj * 16 + wc * 4 + fq) ^ (fr & 7);
                        *(LAS u32x4*)(lds + row * 512 + ch * 16) = w;
                    }
                }
        }
        WAIT_L(0); BAR; asm volatile("" ::: "memory");
        {
            unsigned pos = 0;
            if (tz == 0) pos = __hip_atomic_fetch_add((GAS unsigned*)(ws + TAIL_OFF + 128), 8u, __ATOMIC_RELAXED, __HIP_MEMORY_SCOPE_AGENT);
            const __amdgpu_buffer_rsrc_t ro = __builtin_amdgcn_make_buffer_rsrc(p.out[p.rank], 0, OUT_B, 0x00020000);
            const unsigned vo = (unsigned)(panel_row(p.rank, pm) + wid * 32 + (ln >> 5)) * (DIM * 2) + (unsigned)panel_col(p.rank, pn) * 2 + (unsigned)(ln & 31) * 16;
            const int hi = ln >> 5, chn = ln & 31;
#pragma unroll
            for (int f = 0; f < 16; ++f) {
                const u32x4 v = *(const LAS u32x4*)(lds + (wid * 32 + 2 * f + hi) * 512 + ((chn ^ ((2 * f + hi) & 7)) * 16));
                __builtin_amdgcn_raw_buffer_store_b128(v, ro, vo + (unsigned)(2 * f) * (DIM * 2), 0, AUX_SYS);
            }
            WAIT_V(0); BAR;
            pos = __builtin_amdgcn_readfirstlane(pos) & 1023u;
            if (tz < 8) {
#pragma unroll
                for (int q = 1; q < NR; ++q) st_sys(p.ws[p.rank ^ q] + FL_OFF + (q * 1024 + pos + tz) * 4, (call & 0xfffffu) << 12 | (unsigned)(id + tz));
            }
            STAMP(27);
        }
        break;
    }
#undef SA
#undef SB
#undef STAGE
#undef LDA
#undef LDB
#undef MMA
}

__global__ __launch_bounds__(512, 2) void gemm_ar_kernel(const Params p) {
    extern __shared__ __attribute__((aligned(16))) char smem[];
    LAS char* lds = (LAS char*)smem;
    const int b = blockIdx.x;
    if (b >= NLINK) { gemm_wg(p, lds, b - NLINK); return; }
    if ((b + (b >> 3)) & 7) { link_wg(p, lds, (b + (b >> 3)) & 7, b >> 3); return; }
    if (b == 0 && threadIdx.x < 64) {
        char* ws = p.ws[p.rank];
        const unsigned call = __builtin_amdgcn_readfirstlane(ld_sys(ws + CALL_OFF)) + 1u;
        const unsigned long long t0 = __builtin_amdgcn_s_memrealtime();
        for (unsigned polls = 1;; ++polls) {
            if ((int)(__builtin_amdgcn_readfirstlane(ld_sys(ws + DONE_OFF)) - call * (unsigned)(7 * NE + NGEMM)) >= 0) break;
            if ((polls & 63u) == 0u && give_up(ws + TMO_OFF, call, t0)) break;
            __builtin_amdgcn_s_sleep(32);
        }
        if (threadIdx.x == 0) st_sys(ws + CALL_OFF, call);
    }
}

void build_schedule(Params& p) {
#define BATCHES {0, 1, 16, 17, -1, 2, 18, 19, -1, 3, 20, 21, -1, 8, 9, -1, 4, 22, 23, -1, 5, 24, 25, -1, 10, 11, -1, 6, 26, 27, -1, 7, 28, 29, -1, 12, 13, -1, 30, 31, -1, 14, 15, -1}
    static const int bt[] = BATCHES;
    int pos[32], nu = 0, nb = 0;
    unsigned char* un = (unsigned char*)p.units;
    unsigned short* tl = (unsigned short*)p.tiles;
    for (int i = 0; i < (int)(sizeof(bt) / sizeof(bt[0])); ++nb) {
        int e = i; while (bt[e] >= 0) pos[bt[e++]] = nb;
        for (int c = 0; c < 4; ++c) for (int k = i; k < e; ++k) un[nu++] = (unsigned char)(bt[k] | c << 5);
        i = e + 1;
    }
    for (int i = 0; i < NGEMM; ++i) p.tiles[i] = 0xffffffffu;
    int next = 0, nf = 0;
    for (int own = 0; own < 2; ++own)
        for (int b = 0; b < nb; ++b) {
            int list[256], n = 0;
            for (int r = 0; r < 8; ++r) for (int c = 0; c < 16; ++c) {
                const int a = pos[own * 8 + r], bb = pos[16 + c], t = (own * 8 + r) << 4 | c;
                if ((a > bb ? a : bb) != b) continue;
                if (!own && nf++ < NSPLIT) { list[n++] = t | 1 << 8; list[n++] = t | 2 << 8; } else list[n++] = t;
            }
            int k = 0;
            for (int x = 0; x < 8; ++x) for (int s = next; s < next + n; ++s) if ((s & 7) == x)
                tl[s < NGEMM ? 2 * s : 2 * (s - NGEMM) + 1] = (unsigned short)list[k++];
            next += n;
        }
    signed char* lt = (signed char*)p.late;
    int nl = 0;
    for (int i = 0; i < NGEMM; ++i) lt[i] = -1;
    for (int b = nb - 1; b >= 0; --b) for (int i = NGEMM - 1; i >= 0 && nl < NLATE; --i) {
        const int a = pos[(tl[2 * i] >> 4) & 15], bb = pos[16 + (tl[2 * i] & 15)];
        if ((a > bb ? a : bb) == b) lt[i] = (signed char)nl++;
    }
}
static_assert(128 + NSPLIT <= NGEMM && NGEMM % 8 == 0, "foreign jobs are first jobs");
}

extern "C" void kernel_launch(int rank, int nranks, ncclComm_t comm,
                              hipStream_t stream,
                              void* const* d_in, const int* in_sizes, int n_in,
                              void* d_out, int out_size,
                              const int* in_dtypes, int out_dtype,
                              void* const* d_out_all,
                              void* const* d_ws_all, size_t ws_bytes) {
    (void)comm; (void)in_dtypes; (void)out_dtype; (void)d_out;
    if (nranks != NR || (unsigned)rank >= (unsigned)NR || n_in < 2 || in_sizes[0] != DIM * KL || in_sizes[1] != KL * DIM ||
        out_size != DIM * DIM || ws_bytes < WS_NEED) {
        fprintf(stderr, "kernel_launch: unexpected problem shape\n");
        return;
    }
    Params p;
    p.x = (const char*)d_in[0]; p.w = (const char*)d_in[1]; p.rank = rank;
    for (int r = 0; r < NR; ++r) { p.out[r] = (char*)d_out_all[r]; p.ws[r] = (char*)d_ws_all[r]; }
    build_schedule(p);
    HIP_CHECK(hipFuncSetAttribute((const void*)gemm_ar_kernel, hipFuncAttributeMaxDynamicSharedMemorySize, LDS_B));
    hipLaunchKernelGGL(gemm_ar_kernel, dim3(NLINK + NGEMM), dim3(512), LDS_B, stream, p);
    HIP_CHECK(hipGetLastError());
}
```

```cpp
#include <hip/hip_runtime.h>
#include <rccl/rccl.h>
#include <cstdio>

#define HIP_CHECK(expr) do { hipError_t _e = (expr); if (_e != hipSuccess) { \
    fprintf(stderr, "HIP error %s at %s:%d\n", hipGetErrorString(_e), __FILE__, __LINE__); \
    return; } } while (0)

#define LAS __attribute__((address_space(3)))
#define GAS __attribute__((address_space(1)))
typedef short bf16x8 __attribute__((ext_vector_type(8)));
typedef short s16x4 __attribute__((ext_vector_type(4)));
typedef float f32x4 __attribute__((ext_vector_type(4)));
typedef unsigned u32x4 __attribute__((ext_vector_type(4)));
typedef unsigned u32x2 __attribute__((ext_vector_type(2)));

namespace {
constexpr int NR = 8;
constexpr int DIM = 8192, KL = 1024;
constexpr unsigned KT_B = 32768, HT_B = 16384;
constexpr unsigned SLOT_B = 2 * KT_B;
constexpr unsigned PANEL_B = 32 * SLOT_B;
constexpr unsigned STAGE_B = 32 * PANEL_B;
constexpr unsigned TILE_B = 131072;
constexpr unsigned PIN_OFF = STAGE_B;
constexpr unsigned POUT_OFF = PIN_OFF + 128 * TILE_B;
constexpr unsigned SEND_OFF = POUT_OFF + 128 * TILE_B;
constexpr unsigned SCR_OFF = SEND_OFF + 2048 * HT_B;
constexpr unsigned SCR_B = 256 * TILE_B;
constexpr unsigned DATA_B = SCR_OFF + SCR_B;
constexpr unsigned CNT_OFF = DATA_B;
constexpr unsigned PF_OFF = DATA_B + 4096;
constexpr unsigned IR_OFF = DATA_B + 8192;
constexpr unsigned PL_OFF = DATA_B + 16384;
constexpr unsigned SF_OFF = DATA_B + 20480;
constexpr unsigned FL_OFF = DATA_B + 32768;
constexpr unsigned HV_OFF = DATA_B + 65536;
constexpr unsigned DONE_OFF = DATA_B + 98304;
constexpr unsigned TAIL_OFF = DONE_OFF + 1024;
constexpr unsigned CALL_OFF = DONE_OFF + 2048;
constexpr unsigned TMO_OFF = CALL_OFF + 128;
constexpr unsigned HEAD_OFF = DONE_OFF + 4096;
constexpr size_t WS_NEED = (size_t)DONE_OFF + 8192;
constexpr unsigned OUT_B = (unsigned)DIM * DIM * 2;
constexpr int NE = 8;
constexpr int NS = 2;
constexpr int NLINK = 8 * NE, NGEMM = 256 - NLINK;
constexpr int NURG = NGEMM / 4;
constexpr int LDS_B = 131072 + 256;
constexpr unsigned long long DEADLINE = 100000000ull;
constexpr int AUX_SYS = 17;

constexpr int AUX_DEV = 16;
#define WIN 3
#define TAILWIN 6
constexpr int TAILN = 64;
constexpr int CLAIM_AT = 10;
constexpr int FLEX_AT = 32;
constexpr int KSL = 3, KS3 = 2;
constexpr int NSPLIT = 40;
constexpr int NLATE = 96;
constexpr int URG0 = 48;

struct Params {
    const char* x; const char* w;
    char* myws;
    int rank;
    char* out[NR]; char* ws[NR];
    unsigned units[32];
    unsigned late[NGEMM / 4];
    unsigned tiles[NGEMM];
};

#define STR2(x) #x
#define STR(x) STR2(x)
#define WAIT_V(n) asm volatile("s_waitcnt vmcnt(" STR(n) ")" ::: "memory")
#define WAIT_L(n) asm volatile("s_waitcnt lgkmcnt(" STR(n) ")" ::: "memory")
#define BAR __builtin_amdgcn_s_barrier()
#define SCHED __builtin_amdgcn_sched_barrier(0)


__device__ __forceinline__ void touch_args(const Params& p, int u) {
    const int a = p.rank; const unsigned b = p.units[u]; char* const c = p.out[4]; char* const d = p.ws[4];
    asm volatile("" :: "s"(a), "s"(b), "s"(c), "s"(d));
}
__device__ __forceinline__ unsigned ld_sys(const char* p) {
    return __hip_atomic_load((const GAS unsigned*)p, __ATOMIC_RELAXED, __HIP_MEMORY_SCOPE_SYSTEM);
}
__device__ __forceinline__ void st_sys(char* p, unsigned v) {
    __hip_atomic_store((GAS unsigned*)p, v, __ATOMIC_RELAXED, __HIP_MEMORY_SCOPE_SYSTEM);
}
__device__ __forceinline__ void st8_sys(char* p, unsigned v) {
    __hip_atomic_store((GAS unsigned char*)p, (unsigned char)v, __ATOMIC_RELAXED, __HIP_MEMORY_SCOPE_SYSTEM);
}
__device__ __forceinline__ unsigned call_tag(unsigned call) { return call % 255u + 1u; }
__device__ __forceinline__ void add_sys(char* p, unsigned v) {
    __hip_atomic_fetch_add((GAS unsigned*)p, v, __ATOMIC_RELAXED, __HIP_MEMORY_SCOPE_SYSTEM);
}
__device__ __forceinline__ unsigned cvt_pk_bf16(float lo, float hi) {
    unsigned r; asm volatile("v_cvt_pk_bf16_f32 %0, %1, %2" : "=v"(r) : "v"(lo), "v"(hi)); return r;
}
__device__ __forceinline__ int panel_row(int d, int P) { return ((d >> 1) & 1) * 4096 + ((((P >> 3) ^ (d >> 2) ^ 1) & 1) * 8 + (P & 7)) * 256; }
__device__ __forceinline__ int panel_col(int d, int k) { return (d & 1) * 4096 + k * 256; }
__device__ __forceinline__ int link_slot(int dsel, int cc) { return (cc == 3 ? 8 : 4) + cc * 6 + (dsel < 4 ? dsel - 1 : dsel - 2); }
__device__ __forceinline__ bool give_up(char* tmo, unsigned call, unsigned long long t0) {
    if (__builtin_amdgcn_readfirstlane(__hip_atomic_load((const GAS unsigned*)tmo, __ATOMIC_RELAXED, __HIP_MEMORY_SCOPE_AGENT)) == call
        || __builtin_amdgcn_s_memrealtime() - t0 > DEADLINE) {
        __hip_atomic_store((GAS unsigned*)tmo, call, __ATOMIC_RELAXED, __HIP_MEMORY_SCOPE_AGENT);
        return true;
    }
    return false;
}

struct Piece { const char* src; unsigned stride, dst, pub; int kind; };

__device__ __forceinline__ Piece shard_piece(const Params& p, int d, int panel, int cc, int pc, int dp, int slot) {
    const int k0 = ((d >> 2) * 4 + cc) * 128 + (pc >> 1) * 64, h = pc & 1;
    Piece o; o.kind = panel >= 16;
    o.dst = (unsigned)dp * PANEL_B + (unsigned)slot * SLOT_B + (unsigned)pc * HT_B;
    o.pub = CNT_OFF + (unsigned)dp * 128 + (unsigned)slot * 4 + (unsigned)pc;
    if (o.kind) {
        o.src = p.w + ((size_t)k0 * DIM + panel_col(d, panel - 16) + h * 128) * 2;
        o.stride = 4u * DIM * 2;
    } else {
        o.src = p.x + ((size_t)(panel_row(d, panel) + h * 128) * KL + k0) * 2;
        o.stride = 8u * KL * 2;
    }
    return o;
}
__device__ __forceinline__ int unit_at(const Params& p, int n) { return (p.units[n >> 2] >> ((n & 3) * 8)) & 255; }
__device__ __forceinline__ unsigned send_off(int d, int panel, int cc, int pc) {
    const int gp = panel >= 16 ? 32 + (panel_col(d, panel - 16) >> 8) : panel_row(d, panel) >> 8;
    return SEND_OFF + (unsigned)((gp * 8 + (d >> 2) * 4 + cc) * 4 + pc) * HT_B;
}
__device__ __forceinline__ Piece pull_piece(const Params& p, int code, int dsel) {
    const int cls = code >> 21, pos = (code >> 10) & 2047, id = code & 1023, s = p.rank ^ dsel;
    Piece o; o.kind = 2; o.stride = 1024;
    if (cls == 1) {
        const int u = unit_at(p, (pos >> 2) & 127), panel = u & 31, cc = u >> 5, pc = pos & 3, slot = link_slot(dsel, cc);
        o.src = p.ws[s] + send_off(p.rank, panel, cc, pc);
        o.dst = (unsigned)panel * PANEL_B + (unsigned)slot * SLOT_B + (unsigned)pc * HT_B;
        o.pub = CNT_OFF + (unsigned)panel * 128 + (unsigned)slot * 4 + (unsigned)pc;
    } else if (cls == 0) {
        o.src = p.ws[s] + POUT_OFF + (size_t)id * HT_B;
        o.dst = PIN_OFF + (unsigned)id * HT_B;
        o.pub = PF_OFF + (unsigned)id * 4;
    } else {
        const int ow = cls == 2 ? s : p.rank ^ 4;
        o.kind = 3; o.stride = 2u * DIM * 2;
        o.dst = (unsigned)(panel_row(ow, 8 + (id >> 7)) + (id & 7) * 32) * (DIM * 2) + (unsigned)panel_col(ow, (id >> 3) & 15) * 2;
        o.src = p.out[s] + o.dst;
        o.pub = cls == 2 ? (unsigned)(code & 0xfffff) + 1u : 0u;
    }
    return o;
}
__device__ __forceinline__ const char* piece_src(const Piece& pc, int lane) {
    const unsigned xl = (unsigned)(lane >> 3) * (KL * 2) + (unsigned)((lane & 7) ^ (lane >> 3)) * 16;
    const unsigned wl = (unsigned)(lane >> 4) * (DIM * 2) + (unsigned)((lane & 15) ^ ((lane >> 4) << 2)) * 16;
    const unsigned ol = (unsigned)(lane >> 5) * (DIM * 2) + (unsigned)(lane & 31) * 16;
    return pc.src + (pc.kind == 0 ? xl : pc.kind == 1 ? wl : pc.kind == 2 ? (unsigned)lane * 16 : ol);
}
__device__ __forceinline__ void piece_dma(const Piece& pc, LAS char* buf, int lane) {
    const char* s = piece_src(pc, lane);
#pragma unroll
    for (int f = 0; f < 16; ++f) __builtin_amdgcn_global_load_lds((const GAS unsigned*)(s + (size_t)f * pc.stride), (LAS unsigned*)(buf + f * 1024), 16, 0, AUX_DEV);
}
__device__ __forceinline__ void piece_read_tr(u32x4 (&v)[16], LAS char* buf, int lane) {
    const int q = (lane >> 2) & 3;
    const unsigned trl = (unsigned)(8 * (lane >> 4) + q) * 256 + (unsigned)(lane & 3) * 16;
#pragma unroll
    for (int f = 0; f < 16; ++f) {
        const int nb = f >> 1, kb = f & 1;
        LAS char* a = buf + trl + (((nb >> 1) ^ q) * 64) + kb * 8192 + (nb & 1) * 8;
        const u32x2 lo = __builtin_bit_cast(u32x2, __builtin_amdgcn_ds_read_tr16_b64_v4i16((LAS s16x4*)a));
        const u32x2 hi = __builtin_bit_cast(u32x2, __builtin_amdgcn_ds_read_tr16_b64_v4i16((LAS s16x4*)(a + 1024)));
        v[f] = (u32x4){lo.x, lo.y, hi.x, hi.y};
    }
}
__device__ __forceinline__ void piece_read(u32x4 (&v)[16], bool isx, LAS char* buf, int lane) {
    const unsigned xb = (unsigned)(lane & 15) * 128 + (unsigned)((lane >> 4) ^ (lane & 7)) * 16;
    LAS char* b0 = buf + (isx ? xb : (unsigned)lane * 16);
    LAS char* b1 = buf + (isx ? xb ^ 64u : (unsigned)lane * 16 + 1024u);
#pragma unroll
    for (int f = 0; f < 16; ++f) asm volatile("ds_read_b128 %0, %1 offset:%2" : "=v"(v[f]) : "v"((f & 1) ? b1 : b0), "i"((f >> 1) * 2048) : "memory");
    asm volatile("s_waitcnt lgkmcnt(0)" : "+v"(v[0]), "+v"(v[1]), "+v"(v[2]), "+v"(v[3]), "+v"(v[4]), "+v"(v[5]), "+v"(v[6]), "+v"(v[7]),
                 "+v"(v[8]), "+v"(v[9]), "+v"(v[10]), "+v"(v[11]), "+v"(v[12]), "+v"(v[13]), "+v"(v[14]), "+v"(v[15]) :: "memory");
}

constexpr int NMB = 2 * KSL + KS3;

__device__ __forceinline__ int next_task(LAS unsigned* mb, int lane) {
    for (;;) {
        unsigned v;
        asm volatile("ds_read_b32 %0, %1\n\ts_waitcnt lgkmcnt(0)" : "=&v"(v) : "v"(mb + (lane & 15)) : "memory");
        unsigned best = 0xffffffffu; int bl = 0;
#pragma unroll
        for (int l = 0; l < NMB; ++l) { const unsigned k = (unsigned)__builtin_amdgcn_readlane(v, l) - 1u; if (k < best) { best = k; bl = l; } }
        if (best == 0xffffffffu) return __builtin_amdgcn_readlane(v, NMB) ? -2 : -1;
        unsigned old = 0;
        if (lane == 0) asm volatile("ds_cmpst_rtn_b32 %0, %1, %2, %3\n\ts_waitcnt lgkmcnt(0)" : "=&v"(old) : "v"(mb + bl), "v"(best + 1u), "v"(0u) : "memory");
        if ((unsigned)__builtin_amdgcn_readfirstlane(old) == best + 1u) return (int)(best & 0x7fffffu);
    }
}
static_assert(WIN >= 2 && WIN <= 8 && NS >= 1 && NS <= 3, "window / pullers");
__device__ __forceinline__ void lds_wait_zero(LAS unsigned* a) {
    for (;;) {
        unsigned f;
        asm volatile("ds_read_b32 %0, %1\n\ts_waitcnt lgkmcnt(0)" : "=&v"(f) : "v"(a) : "memory");
        if (__builtin_amdgcn_readfirstlane(f) == 0) break;
        __builtin_amdgcn_s_sleep(1);
    }
}

__device__ __forceinline__ void link_wg(const Params& p, LAS char* lds, int dsel, int e) {
    const int wid = __builtin_amdgcn_readfirstlane(threadIdx.x >> 6), lane = threadIdx.x & 63;
    char* ws = p.myws;
    LAS unsigned* mb = (LAS unsigned*)(lds + 7 * HT_B);
    const int sn = wid < NS ? wid : wid - NS;
    LAS char* buf = lds + sn * 2 * HT_B;
    LAS unsigned* slot = mb + 16 + sn * 8;
    touch_args(p, 0);
    Piece first;
    if (wid >= NS && wid < 2 * NS) {
        if (dsel != 4) {
            const int q = sn * NE + e, u = unit_at(p, q >> 2);
            first = shard_piece(p, p.rank ^ dsel, u & 31, u >> 5, q & 3, u & 31, link_slot(dsel, u >> 5));
            piece_dma(first, buf + HT_B, lane);
        }
    } else if (wid == 7) {
        const unsigned c = ld_sys(ws + CALL_OFF) + 1u;
        mb[lane] = lane == 15 ? c : (lane >= 16 && (lane & 7) == 4 && dsel != 4) ? 14u : 0u; WAIT_L(0);
        if (lane == 0) add_sys(ws + DONE_OFF, 1u);
    }
    BAR; asm volatile("" ::: "memory");
    unsigned call;
    asm volatile("ds_read_b32 %0, %1 offset:60\n\ts_waitcnt lgkmcnt(0)" : "=&v"(call) : "v"(mb) : "memory");
    call = __builtin_amdgcn_readfirstlane(call);
    if (wid >= 2 * NS) {
        if (wid != 7) return;
        const int cls = lane < KSL ? (dsel == 4 ? 0 : 1) : lane < 2 * KSL ? 2 : 3;
        const unsigned nfirst = cls == 1 ? (NS + KSL) * NE : 0u;
        const unsigned n = cls == 1 ? 512u : 1024u;
        const unsigned base = (call - 1u) * (n - nfirst + NE * (cls == 3 ? 7 * KS3 : KSL)) - nfirst;
        char* head = ws + HEAD_OFF + (cls == 3 ? 0 : dsel * 2 + (cls >> 1)) * 128;
        const char* list = (cls == 3 && dsel != 4) ? p.ws[p.rank ^ dsel] + HV_OFF + (dsel ^ 4) * 4096
                                                     : ws + (cls == 0 ? PL_OFF : cls == 1 ? IR_OFF + dsel * 1024 : FL_OFF + dsel * 4096);
        const unsigned tag = call & 0xfffffu;
        int st = (lane >= NMB || (cls == 2 && dsel == 4)) ? 3 : (cls == 3 && dsel != 4) ? 4 : 0;
        unsigned pos = (NS + lane) * NE + e, lo = pos & ~3u;
        if (cls == 1) st = 1;
        const unsigned long long t0 = __builtin_amdgcn_s_memrealtime();
        for (unsigned polls = 1;; ++polls) {
            if (st == 2 && __hip_atomic_load(mb + lane, __ATOMIC_RELAXED, __HIP_MEMORY_SCOPE_WORKGROUP) == 0u) st = 0;
            unsigned t = 0, en = 0;
            if (st == 0) t = __hip_atomic_fetch_add((GAS unsigned*)head, 1u, __ATOMIC_RELAXED, __HIP_MEMORY_SCOPE_AGENT);
            if (st == 1) en = ld_sys(list + lo);
            if (st == 0) {
                pos = t - base; st = pos >= n ? 3 : 1;
                lo = cls != 1 ? pos * 4 : (pos < 4 * NURG ? pos & ~3u : 4u * NURG);
            } else if (st == 1 && (en >> 12) == tag) {
                __hip_atomic_store(mb + lane, ((((unsigned)cls << 11 | pos) << 10) | (en & 1023u)) + 1u, __ATOMIC_RELAXED, __HIP_MEMORY_SCOPE_WORKGROUP);
                st = 2;
            }
            if (__any(cls == 2 && st != 0 && pos >= 1024u - FLEX_AT) && st == 4) st = 0;
            if (__all(st == 3)) break;
            if ((polls & 63u) == 0u && give_up(ws + TMO_OFF, call, t0)) {
                if (lane == 0) st_sys(p.out[p.rank] + ((size_t)panel_row(p.rank, 8) * DIM + (size_t)panel_col(p.rank, 0)) * 2, 0x7fc07fc0u);
                break;
            }
            __builtin_amdgcn_s_sleep(4);
        }
        if (lane == 0) __hip_atomic_store(mb + NMB, 1u, __ATOMIC_RELAXED, __HIP_MEMORY_SCOPE_WORKGROUP);
        return;
    }
    if (wid < NS) {
        int k = 0, code = -1;
        bool pend = false;
        u32x4 desc = {0u, 0u, 0u, 0u};
        Piece cur;
        for (;;) {
            if (code < 0) {
                if (code == -1) code = next_task(mb, lane);
                if (code < 0) {
                    if (pend) { WAIT_V(0); if (lane == 0) asm volatile("ds_write_b128 %0, %1" :: "v"(slot + ((k - 1) & 1) * 4), "v"(desc) : "memory"); pend = false; }
                    while (code == -1) { __builtin_amdgcn_s_sleep(2); code = next_task(mb, lane); }
                    if (code < 0) break;
                }
                cur = pull_piece(p, code, dsel);
            }
            lds_wait_zero(slot + (k & 1) * 4);
            LAS char* b = buf + (k & 1) * HT_B;
            const char* sp = piece_src(cur, lane);
            const unsigned stride = cur.stride;
            const u32x4 ndesc = {(unsigned)cur.kind + 1u, cur.dst, cur.pub, 0u};
            const bool fast = (code >> 10) >= (3 << 11 | (1024 - TAILN));
#pragma unroll
            for (int f = 0; f < 16; ++f) {
                __builtin_amdgcn_global_load_lds((const GAS unsigned*)sp, (LAS unsigned*)(b + f * 1024), 16, 0, AUX_SYS);
                sp += stride;
                asm volatile("" ::: "memory");
                if (f == CLAIM_AT) { code = next_task(mb, lane); if (code >= 0) cur = pull_piece(p, code, dsel); }
                if (f > WIN - 2 && fast) WAIT_V(TAILWIN - 1); else WAIT_V(WIN - 1);
                if (f == WIN - 2 && pend && lane == 0)
                    asm volatile("ds_write_b128 %0, %1" :: "v"(slot + ((k - 1) & 1) * 4), "v"(desc) : "memory");
            }
            desc = ndesc; pend = true; ++k;
        }
        lds_wait_zero(slot + (k & 1) * 4);
        if (lane == 0) asm volatile("ds_write_b32 %0, %1" :: "v"(slot + (k & 1) * 4), "v"(15u) : "memory");
        return;
    }
    if (dsel != 4) {
        char* dws = p.ws[p.rank ^ dsel];
        const __amdgpu_buffer_rsrc_t rs = __builtin_amdgcn_make_buffer_rsrc(dws, 0, STAGE_B, 0x00020000);
        WAIT_V(0);
        u32x4 v[16];
        if (first.kind) { piece_read_tr(v, buf + HT_B, lane); WAIT_L(0);
#pragma unroll
            for (int f = 0; f < 16; ++f) asm volatile("" : "+v"(v[f]));
        } else piece_read(v, true, buf + HT_B, lane);
        if (lane == 0) asm volatile("ds_write_b32 %0, %1" :: "v"(slot + 4), "v"(0u) : "memory");
#pragma unroll
        for (int f = 0; f < 16; ++f) { __builtin_amdgcn_raw_buffer_store_b128(v[f], rs, first.dst + lane * 16 + f * 1024, 0, AUX_SYS); WAIT_V(WIN - 1); }
        WAIT_V(0);
        if (lane == 0) st8_sys(dws + first.pub, call_tag(call));
    }
    const __amdgpu_buffer_rsrc_t rsw = __builtin_amdgcn_make_buffer_rsrc(ws, 0, POUT_OFF, 0x00020000);
    const __amdgpu_buffer_rsrc_t rso = __builtin_amdgcn_make_buffer_rsrc(p.out[p.rank], 0, OUT_B, 0x00020000);
    const unsigned ol = (unsigned)(lane >> 5) * (DIM * 2) + (unsigned)(lane & 31) * 16;
    char* fwd = ws + HV_OFF + dsel * 4096;
    for (int k = 0;;) {
        u32x4 ds4;
        asm volatile("ds_read_b128 %0, %1\n\ts_waitcnt lgkmcnt(0)" : "=&v"(ds4) : "v"(slot + (k & 1) * 4) : "memory");
        const int kind = __builtin_amdgcn_readfirstlane(ds4.x) - 1;
        if (kind > 3) break;
        if (kind < 0) { __builtin_amdgcn_s_sleep(1); continue; }
        const unsigned dst = __builtin_amdgcn_readfirstlane(ds4.y), pub = __builtin_amdgcn_readfirstlane(ds4.z);
        u32x4 v[16];
        piece_read(v, false, buf + (k & 1) * HT_B, lane);
        if (lane == 0) asm volatile("ds_write_b32 %0, %1" :: "v"(slot + (k & 1) * 4), "v"(0u) : "memory");
        const bool o = kind == 3;
        const __amdgpu_buffer_rsrc_t rs = o ? rso : rsw;
        const unsigned ds = o ? 2u * DIM * 2 : 1024u;
        unsigned vo = dst + (o ? ol : (unsigned)lane * 16);
#pragma unroll
        for (int f = 0; f < 16; ++f) {
            __builtin_amdgcn_raw_buffer_store_b128(v[f], rs, vo, 0, AUX_SYS);
            vo += ds;
        }
        if (pub) {
            WAIT_V(0);
            if (lane == 0) {
                if (o) st_sys(fwd + ((pub - 1u) >> 10) * 4, (call & 0xfffffu) << 12 | ((pub - 1u) & 1023u));
                else if (pub >= PF_OFF) st_sys(ws + pub, call);
                else st8_sys(ws + pub, call_tag(call));
            }
        }
        ++k;
    }
}

__device__ __forceinline__ int ready_prefix(unsigned v, unsigned target, unsigned low) {
    const unsigned long long m = __ballot(v == target);
    const unsigned both = ((unsigned)m & (unsigned)(m >> 32)) | low;
    return both == 0xffffffffu ? 32 : __builtin_ctz(~both);
}
__device__ __forceinline__ void kn_put(LAS unsigned* kn, unsigned v) { asm volatile("ds_write_b32 %0, %1" :: "v"(kn), "v"(v) : "memory"); }
__device__ __noinline__ int wait_slot(const char* cptr, char* tmo, unsigned call, int need, unsigned low, LAS unsigned* kn, int job, int duty) {
    int known;
    if (__builtin_amdgcn_readfirstlane(duty)) {
        const unsigned long long t0 = __builtin_amdgcn_s_memrealtime();
        for (unsigned polls = 1;; ++polls) {
            known = ready_prefix(ld_sys(cptr), call_tag(call) * 0x01010101u, low);
            if (known > need) break;
            if ((polls & 63u) == 0u && give_up(tmo, call, t0)) { known = 32; break; }
            __builtin_amdgcn_s_sleep(32);
        }
        kn_put(kn, (unsigned)(job + known));
    } else {
        for (;;) {
            unsigned f;
            asm volatile("ds_read_b32 %0, %1\n\ts_waitcnt lgkmcnt(0)" : "=&v"(f) : "v"(kn) : "memory");
            known = (int)f - job;
            if (known > need) break;
            __builtin_amdgcn_s_sleep(4);
        }
    }
    return known;
}

__device__ __forceinline__ void gemm_wg(const Params& p, LAS char* lds, int gb) {
    const int tid = threadIdx.x, wid = __builtin_amdgcn_readfirstlane(tid >> 6), lane = tid & 63, wr = wid >> 2, wc = wid & 3;
    char* ws = p.myws;
    unsigned cv;
    asm volatile("global_load_dword %0, %1, %2 sc0 sc1" : "=v"(cv) : "v"(CALL_OFF), "s"(ws) : "memory");
    const int late = (int)(signed char)((p.late[gb >> 2] >> ((gb & 3) * 8)) & 255);
    const bool bulk = wid >= 4;
    LAS unsigned* ucnt = (LAS unsigned*)(lds + 131072);
    if (tid < 2) ucnt[tid] = 0u;
    touch_args(p, gb >> 4);
    WAIT_L(0); BAR;
    for (int i = bulk ? (late >= 0 ? 4 + wid : 0) : (gb < URG0 ? 0 : 2 + ((gb - URG0) >> 5)); i > 0; --i) __builtin_amdgcn_s_sleep(32);
    unsigned nbulk = 0;
    for (int r = !bulk ? gb * 4 + wid : late >= 0 ? 4 * NGEMM + (wid - 4) * NLATE + late : 2048; r < 2048; r = bulk ? r + 4 * NLATE : 2048) {
        const int other = (r >> 3) & 1, half = (r >> 2) & 1, pcs = r & 3;
        const int u = unit_at(p, r >> 4), panel = u & 31, cc = u >> 5;
        const int d0 = (((p.rank >> 2) ^ other) << 2) + (panel >= 16 ? half : half * 2), d1 = d0 + (panel >= 16 ? 2 : 1);
        const bool own0 = ((d0 ^ p.rank) & 3) == 0, own1 = ((d1 ^ p.rank) & 3) == 0, own = own0 || own1;
        const Piece pc = shard_piece(p, d0, panel, cc, pcs, panel < 16 && other ? panel ^ 8 : panel, other * 22 + cc);
        LAS char* buf = lds + wid * HT_B;
        piece_dma(pc, buf, lane);
        asm volatile("s_waitcnt vmcnt(0)" : "+v"(cv) :: "memory");
        const unsigned call = __builtin_amdgcn_readfirstlane(cv) + 1u;
        u32x4 v[16];
        if (pc.kind) piece_read_tr(v, buf, lane); else piece_read(v, true, buf, lane);
        const __amdgpu_buffer_rsrc_t rs = __builtin_amdgcn_make_buffer_rsrc(ws, 0, DATA_B, 0x00020000);
        const unsigned so = send_off(d0, panel, cc, pcs);
#pragma unroll
        for (int f = 0; f < 16; ++f) __builtin_amdgcn_raw_buffer_store_b128(v[f], rs, so + lane * 16 + f * 1024, 0, AUX_SYS);
        if (own) {
#pragma unroll
            for (int f = 0; f < 16; ++f) __builtin_amdgcn_raw_buffer_store_b128(v[f], rs, pc.dst + lane * 16 + f * 1024, 0, AUX_SYS);
        }
        WAIT_V(0);
        ++nbulk;
        if (lane == 0) {
            if (own) st8_sys(ws + pc.pub, call_tag(call));
            if (!bulk && __hip_atomic_fetch_add(ucnt, 1u, __ATOMIC_RELAXED, __HIP_MEMORY_SCOPE_WORKGROUP) == 3u) {
                if (!own0) st_sys(p.ws[d0] + IR_OFF + (p.rank ^ d0) * 1024 + (r >> 4) * 4, (call & 0xfffffu) << 12);
                if (!own1) st_sys(p.ws[d1] + IR_OFF + (p.rank ^ d1) * 1024 + (r >> 4) * 4, (call & 0xfffffu) << 12);
            }
        }
    }
    asm volatile("s_waitcnt vmcnt(0)" : "+v"(cv) :: "memory");
    const unsigned call = __builtin_amdgcn_readfirstlane(cv) + 1u;
    if (bulk && nbulk && lane == 0 &&
        __hip_atomic_fetch_add((GAS unsigned*)(ws + TAIL_OFF + 256), nbulk, __ATOMIC_RELAXED, __HIP_MEMORY_SCOPE_AGENT) + nbulk == call * (2048u - 4u * NGEMM)) {
#pragma unroll
        for (int q = 1; q < NR; ++q) if (q != 4) st_sys(p.ws[p.rank ^ q] + IR_OFF + q * 1024 + NURG * 4, (call & 0xfffffu) << 12);
    }
    BAR;
    if (tid == 0) add_sys(ws + DONE_OFF, 1u);
    const unsigned target = call_tag(call) * 0x01010101u;
    const unsigned voff = (unsigned)tid * 16;
    const unsigned ldsw = (unsigned)wid * 1024u;
    const int aoff = wr * 8192 + lane * 16, boff = wc * 4096 + lane * 16;
#define SA(b, h) (((b) * 2 + (h)) * HT_B)
#define SB(b, h) ((4 + (b) * 2 + (h)) * HT_B)
#define STAGE(bufoff, gbase) do { _Pragma("unroll") for (int _i = 0; _i < 2; ++_i) \
        __builtin_amdgcn_global_load_lds((const GAS unsigned*)((const char*)(gbase) + voff + _i * 8192), (LAS unsigned*)(lds + (bufoff) + ldsw + _i * 8192), 16, 0, AUX_DEV); } while (0)
#define LDA(dst, b, h) do { _Pragma("unroll") for (int m = 0; m < 4; ++m) _Pragma("unroll") for (int k = 0; k < 2; ++k) dst[m][k] = *(const LAS bf16x8*)(lds + SA(b, h) + aoff + m * 2048 + k * 1024); } while (0)
#define LDB(dst, b, h) do { _Pragma("unroll") for (int n = 0; n < 2; ++n) _Pragma("unroll") for (int k = 0; k < 2; ++k) dst[n][k] = *(const LAS bf16x8*)(lds + SB(b, h) + boff + n * 2048 + k * 1024); } while (0)
#define MMA(ai, bj, At, Bt) do { __builtin_amdgcn_s_setprio(1); _Pragma("unroll") for (int m = 0; m < 4; ++m) _Pragma("unroll") for (int n = 0; n < 2; ++n) _Pragma("unroll") for (int k = 0; k < 2; ++k) \
        acc[ai][bj][m][n] = __builtin_amdgcn_mfma_f32_16x16x32_bf16(Bt[n][k], At[m][k], acc[ai][bj][m][n], 0, 0, 0); __builtin_amdgcn_s_setprio(0); } while (0)
    for (int rnd = 0; rnd < 2; ++rnd) {
        const int code = (p.tiles[gb] >> (16 * rnd)) & 0xffff;
        if (code == 0xffff) break;
        const int pm = (code >> 4) & 15, pn = code & 15, part = (code >> 8) & 1, np = 1 + (code >> 9);
        const unsigned low = np == 1 ? 0u : 0xaaaaaaaau >> part;
        const char* Ab = ws + (size_t)pm * PANEL_B;
        const char* Bb = ws + (size_t)(16 + pn) * PANEL_B;
        const unsigned coff = CNT_OFF + (unsigned)(lane < 32 ? pm : 16 + pn) * 128 + (unsigned)(lane & 31) * 4;
        f32x4 acc[2][2][4][2];
#pragma unroll
        for (int a = 0; a < 2; ++a)
#pragma unroll
            for (int b = 0; b < 2; ++b)
#pragma unroll
                for (int m = 0; m < 4; ++m)
#pragma unroll
                    for (int n = 0; n < 2; ++n) acc[a][b][m][n] = (f32x4){0.f, 0.f, 0.f, 0.f};
        bf16x8 At[4][2], B0[2][2], B1[2][2];
        int known = __builtin_amdgcn_readfirstlane(wait_slot(ws + coff, ws + TMO_OFF, call, part, low, ucnt + 1, rnd * 64, wid == 0));
        {
            const char* a0 = Ab + (unsigned)part * SLOT_B; const char* b0 = Bb + (unsigned)part * SLOT_B;
            STAGE(SB(0, 0), b0); STAGE(SA(0, 0), a0); STAGE(SB(0, 1), b0 + HT_B); STAGE(SA(0, 1), a0 + HT_B);
            if (wr == 1) BAR;
            WAIT_V(4); BAR;
            STAGE(SB(1, 0), b0 + KT_B); STAGE(SA(1, 0), a0 + KT_B); STAGE(SB(1, 1), b0 + KT_B + HT_B);
            WAIT_V(6); BAR;
        }
        for (int T = part; T < 32; T += np) {
            if (T + np < 32 && known <= T + np) known = __builtin_amdgcn_readfirstlane(wait_slot(ws + coff, ws + TMO_OFF, call, T + np, low, ucnt + 1, rnd * 64, wid == 0));
            const unsigned o1 = (unsigned)T * SLOT_B, o2 = (unsigned)(T + np < 32 ? T + np : T) * SLOT_B;
            const char* a1 = Ab + o1 + KT_B; const char* a2 = Ab + o2; const char* b2 = Bb + o2;
            const char* a3 = a2 + KT_B; const char* b3 = b2 + KT_B;
            unsigned pv = 0;
            const bool ask = known < 32;
            LDB(B0, 0, 0); SCHED; LDA(At, 0, 0);
            if (ask) asm volatile("global_load_dword %0, %1, %2 sc0 sc1" : "+v"(pv) : "v"(coff), "s"(ws) : "memory");
            STAGE(SA(1, 1), a1 + HT_B);
            WAIT_L(8); BAR; WAIT_L(0); MMA(0, 0, At, B0); BAR; SCHED;
            LDB(B1, 0, 1); STAGE(SB(0, 0), b2);
            BAR; WAIT_L(0); MMA(0, 1, At, B1); BAR;
            LDA(At, 0, 1); STAGE(SA(0, 0), a2);
            BAR; WAIT_L(0); MMA(1, 0, At, B0); BAR; SCHED;
            STAGE(SB(0, 1), b2 + HT_B);
            WAIT_V(6);
            asm volatile("" : "+v"(pv));
            { const int k2 = ready_prefix(pv, target, low); if (k2 > known) { known = k2; if (wid == 0) kn_put(ucnt + 1, (unsigned)(rnd * 64 + k2)); } }
            BAR; MMA(1, 1, At, B1); BAR;
            LDB(B0, 1, 0); SCHED; LDA(At, 1, 0); STAGE(SA(0, 1), a2 + HT_B);
            WAIT_L(8); BAR; WAIT_L(0); MMA(0, 0, At, B0); BAR; SCHED;
            LDB(B1, 1, 1); STAGE(SB(1, 0), b3);
            BAR; WAIT_L(0); MMA(0, 1, At, B1); BAR;
            LDA(At, 1, 1); STAGE(SA(1, 0), a3);
            BAR; WAIT_L(0); MMA(1, 0, At, B0); BAR; SCHED;
            STAGE(SB(1, 1), b3 + HT_B);
            WAIT_V(6); BAR; MMA(1, 1, At, B1); BAR;
        }
        WAIT_V(0);
        if (wr == 0) BAR;
        BAR;
        int tz = threadIdx.x; asm volatile("" : "+v"(tz));
        const int ln = tz & 63;
        const int id = ((pm & 7) * 16 + pn) * 8 + wid;
        if (part + 1 < np) {
            const __amdgpu_buffer_rsrc_t rq = __builtin_amdgcn_make_buffer_rsrc(ws + SCR_OFF, 0, SCR_B, 0x00020000);
            const unsigned base = (unsigned)id * (2 * HT_B) + (unsigned)ln * 16;
#pragma unroll
            for (int f = 0; f < 32; ++f) __builtin_amdgcn_raw_buffer_store_b128(__builtin_bit_cast(u32x4, acc[f >> 4][(f >> 1) & 1][(f >> 2) & 3][f & 1]), rq, base + f * 1024, 0, AUX_SYS);
            WAIT_V(0);
            if (ln == 0) st_sys(ws + SF_OFF + id * 4, call);
            continue;
        }
        if (pm < 8) {
            unsigned pos = 0;
            if (tz == 0) pos = __hip_atomic_fetch_add((GAS unsigned*)(ws + TAIL_OFF), 8u, __ATOMIC_RELAXED, __HIP_MEMORY_SCOPE_AGENT);
            if (np > 1) {
                const unsigned long long t0 = __builtin_amdgcn_s_memrealtime();
                for (unsigned polls = 1;; ++polls) {
                    if ((int)(__builtin_amdgcn_readfirstlane(ld_sys(ws + SF_OFF + id * 4)) - call) >= 0) break;
                    if ((polls & 63u) == 0u && give_up(ws + TMO_OFF, call, t0)) break;
                    __builtin_amdgcn_s_sleep(4);
                }
                asm volatile("" ::: "memory");
            }
            const bool bad = __builtin_amdgcn_readfirstlane(__hip_atomic_load((const GAS unsigned*)(ws + TMO_OFF), __ATOMIC_RELAXED, __HIP_MEMORY_SCOPE_AGENT)) == call;
            const __amdgpu_buffer_rsrc_t rq = __builtin_amdgcn_make_buffer_rsrc(ws + SCR_OFF, 0, (unsigned)part * SCR_B, 0x00020000);
            const __amdgpu_buffer_rsrc_t rp = __builtin_amdgcn_make_buffer_rsrc(ws + POUT_OFF, 0, 128 * TILE_B, 0x00020000);
            const unsigned base = (unsigned)id * HT_B + (unsigned)ln * 16;
#pragma unroll
            for (int g4 = 0; g4 < 4; ++g4) {
                f32x4 pr[8];
#pragma unroll
                for (int f = 0; f < 8; ++f) pr[f] = __builtin_bit_cast(f32x4, __builtin_amdgcn_raw_buffer_load_b128(rq, base + (unsigned)id * HT_B + (g4 * 8 + f) * 1024, 0, AUX_SYS));
#pragma unroll
                for (int mm = 0; mm < 2; ++mm)
#pragma unroll
                    for (int bj = 0; bj < 2; ++bj) {
                        const int ai = g4 >> 1, m = (g4 & 1) * 2 + mm;
                        const f32x4 v0 = acc[ai][bj][m][0] + pr[mm * 4 + bj * 2], v1 = acc[ai][bj][m][1] + pr[mm * 4 + bj * 2 + 1];
                        u32x4 w; w.x = cvt_pk_bf16(v0[0], v0[1]); w.y = cvt_pk_bf16(v0[2], v0[3]); w.z = cvt_pk_bf16(v1[0], v1[1]); w.w = cvt_pk_bf16(v1[2], v1[3]);
                        if (bad) w = (u32x4){0x7fc07fc0u, 0x7fc07fc0u, 0x7fc07fc0u, 0x7fc07fc0u};
                        __builtin_amdgcn_raw_buffer_store_b128(w, rp, base + (unsigned)((ai * 4 + m) * 2 + bj) * 1024, 0, AUX_SYS);
                    }
                if (g4 & 1) asm volatile("" ::: "memory");
            }
            WAIT_V(0); BAR;
            pos = __builtin_amdgcn_readfirstlane(pos) & 1023u;
            if (tz < 8) st_sys(p.ws[p.rank ^ 4] + PL_OFF + (pos + tz) * 4, (call & 0xfffffu) << 12 | (unsigned)(id + tz));
            continue;
        }
        {
            const unsigned long long t0 = __builtin_amdgcn_s_memrealtime();
            for (unsigned polls = 1;; ++polls) {
                if ((int)(__builtin_amdgcn_readfirstlane(ld_sys(ws + PF_OFF + id * 4)) - call) >= 0) break;
                if ((polls & 63u) == 0u && give_up(ws + TMO_OFF, call, t0)) break;
                __builtin_amdgcn_s_sleep(127);
            }
            asm volatile("" ::: "memory");
            const bool bad = __builtin_amdgcn_readfirstlane(__hip_atomic_load((const GAS unsigned*)(ws + TMO_OFF), __ATOMIC_RELAXED, __HIP_MEMORY_SCOPE_AGENT)) == call;
            const __amdgpu_buffer_rsrc_t rp = __builtin_amdgcn_make_buffer_rsrc(ws + PIN_OFF, 0, 128 * TILE_B, 0x00020000);
            const unsigned base = (unsigned)id * HT_B + (unsigned)ln * 16;
            u32x4 pr[16];
#pragma unroll
            for (int f = 0; f < 16; ++f) pr[f] = __builtin_amdgcn_raw_buffer_load_b128(rp, base + f * 1024, 0, AUX_SYS);
            const int fr = ln & 15, fq = ln >> 4;
#pragma unroll
            for (int ai = 0; ai < 2; ++ai)
#pragma unroll
                for (int m = 0; m < 4; ++m) {
                    const int row = ai * 128 + wr * 64 + m * 16 + fr;
#pragma unroll
                    for (int bj = 0; bj < 2; ++bj) {
                        const u32x4 q = pr[(ai * 4 + m) * 2 + bj];
                        const f32x4 v0 = acc[ai][bj][m][0], v1 = acc[ai][bj][m][1];
                        u32x4 w;
                        w.x = cvt_pk_bf16(v0[0] + __uint_as_float(q.x << 16), v0[1] + __uint_as_float(q.x & 0xffff0000u));
                        w.y = cvt_pk_bf16(v0[2] + __uint_as_float(q.y << 16), v0[3] + __uint_as_float(q.y & 0xffff0000u));
                        w.z = cvt_pk_bf16(v1[0] + __uint_as_float(q.z << 16), v1[1] + __uint_as_float(q.z & 0xffff0000u));
                        w.w = cvt_pk_bf16(v1[2] + __uint_as_float(q.w << 16), v1[3] + __uint_as_float(q.w & 0xffff0000u));
                        if (bad) w = (u32x4){0x7fc07fc0u, 0x7fc07fc0u, 0x7fc07fc0u, 0x7fc07fc0u};
                        const int ch = (bj * 16 + wc * 4 + fq) ^ (fr & 7);
                        *(LAS u32x4*)(lds + row * 512 + ch * 16) = w;
                    }
                }
        }
        WAIT_L(0); BAR; asm volatile("" ::: "memory");
        {
            unsigned pos = 0;
            if (tz == 0) pos = __hip_atomic_fetch_add((GAS unsigned*)(ws + TAIL_OFF + 128), 8u, __ATOMIC_RELAXED, __HIP_MEMORY_SCOPE_AGENT);
            const __amdgpu_buffer_rsrc_t ro = __builtin_amdgcn_make_buffer_rsrc(p.out[p.rank], 0, OUT_B, 0x00020000);
            const unsigned vo = (unsigned)(panel_row(p.rank, pm) + wid * 32 + (ln >> 5)) * (DIM * 2) + (unsigned)panel_col(p.rank, pn) * 2 + (unsigned)(ln & 31) * 16;
            const int hi = ln >> 5, chn = ln & 31;
#pragma unroll
            for (int f = 0; f < 16; ++f) {
                const u32x4 v = *(const LAS u32x4*)(lds + (wid * 32 + 2 * f + hi) * 512 + ((chn ^ ((2 * f + hi) & 7)) * 16));
                __builtin_amdgcn_raw_buffer_store_b128(v, ro, vo + (unsigned)(2 * f) * (DIM * 2), 0, AUX_SYS);
            }
            WAIT_V(0); BAR;
            pos = __builtin_amdgcn_readfirstlane(pos) & 1023u;
            if (tz < 8) {
#pragma unroll
                for (int q = 1; q < NR; ++q) st_sys(p.ws[p.rank ^ q] + FL_OFF + (q * 1024 + pos + tz) * 4, (call & 0xfffffu) << 12 | (unsigned)(id + tz));
            }
        }
        break;
    }
#undef SA
#undef SB
#undef STAGE
#undef LDA
#undef LDB
#undef MMA
}

__global__ __launch_bounds__(512, 2) void gemm_ar_kernel(const Params p) {
    extern __shared__ __attribute__((aligned(16))) char smem[];
    LAS char* lds = (LAS char*)smem;
    const int b = blockIdx.x;
    if (b >= NLINK) { gemm_wg(p, lds, b - NLINK); return; }
    if ((b + (b >> 3)) & 7) { link_wg(p, lds, (b + (b >> 3)) & 7, b >> 3); return; }
    if (b == 0 && threadIdx.x < 64) {
        char* ws = p.myws;
        const unsigned call = __builtin_amdgcn_readfirstlane(ld_sys(ws + CALL_OFF)) + 1u;
        const unsigned long long t0 = __builtin_amdgcn_s_memrealtime();
        for (unsigned polls = 1;; ++polls) {
            if ((int)(__builtin_amdgcn_readfirstlane(ld_sys(ws + DONE_OFF)) - call * (unsigned)(7 * NE + NGEMM)) >= 0) break;
            if ((polls & 63u) == 0u && give_up(ws + TMO_OFF, call, t0)) break;
            __builtin_amdgcn_s_sleep(32);
        }
        if (threadIdx.x == 0) st_sys(ws + CALL_OFF, call);
    }
}

void build_schedule(Params& p) {
    static const int bt[] = {0, 16, 1, 17, 2, 18, 19, -1, 3, 4, 20, 21, -1, 8, 9, -1, 22, 23, -1, 5, 24, 25, -1, 10, 11, -1,
                             6, 26, 27, -1, 7, 28, 29, -1, 12, 13, -1, 30, 31, -1, 14, 15, -1};
    int pos[32], nu = 0, nb = 0;
    unsigned char* un = (unsigned char*)p.units;
    unsigned short* tl = (unsigned short*)p.tiles;
    for (int i = 0; i < (int)(sizeof(bt) / sizeof(bt[0])); ++nb) {
        int e = i; while (bt[e] >= 0) pos[bt[e++]] = nb;
        for (int c = 0; c < 4; ++c) for (int k = i; k < e; ++k) un[nu++] = (unsigned char)(bt[k] | c << 5);
        i = e + 1;
    }
    for (int i = 0; i < NGEMM; ++i) p.tiles[i] = 0xffffffffu;
    int next = 0, nf = 0;
    for (int own = 0; own < 2; ++own)
        for (int b = 0; b < nb; ++b) {
            int list[256], n = 0;
            for (int r = 0; r < 8; ++r) for (int c = 0; c < 16; ++c) {
                const int a = pos[own * 8 + r], bb = pos[16 + c], t = (own * 8 + r) << 4 | c;
                if ((a > bb ? a : bb) != b) continue;
                if (!own && nf++ < NSPLIT) { list[n++] = t | 2 << 8; list[n++] = t | 3 << 8; } else list[n++] = t;
            }
            int k = 0;
            for (int x = 0; x < 8; ++x) for (int s = next; s < next + n; ++s) if ((s & 7) == x)
                tl[s < NGEMM ? 2 * s : 2 * (s - NGEMM) + 1] = (unsigned short)list[k++];
            next += n;
        }
    signed char* lt = (signed char*)p.late;
    int nl = 0;
    for (int i = 0; i < NGEMM; ++i) lt[i] = -1;
    for (int b = nb - 1; b >= 0; --b) for (int i = NGEMM - 1; i >= 0 && nl < NLATE; --i) {
        const int a = pos[(tl[2 * i] >> 4) & 15], bb = pos[16 + (tl[2 * i] & 15)];
        if ((a > bb ? a : bb) == b) lt[i] = (signed char)nl++;
    }
}
static_assert(128 + NSPLIT <= NGEMM && NGEMM % 8 == 0, "foreign jobs are first jobs");
}

extern "C" void kernel_launch(int rank, int nranks, ncclComm_t comm,
                              hipStream_t stream,
                              void* const* d_in, const int* in_sizes, int n_in,
                              void* d_out, int out_size,
                              const int* in_dtypes, int out_dtype,
                              void* const* d_out_all,
                              void* const* d_ws_all, size_t ws_bytes) {
    (void)comm; (void)in_dtypes; (void)out_dtype; (void)d_out;
    if (nranks != NR || (unsigned)rank >= (unsigned)NR || n_in < 2 || in_sizes[0] != DIM * KL || in_sizes[1] != KL * DIM ||
        out_size != DIM * DIM || ws_bytes < WS_NEED) {
        fprintf(stderr, "kernel_launch: unexpected problem shape\n");
        return;
    }
    Params p;
    p.x = (const char*)d_in[0]; p.w = (const char*)d_in[1]; p.rank = rank;
    for (int r = 0; r < NR; ++r) { p.out[r] = (char*)d_out_all[r]; p.ws[r] = (char*)d_ws_all[r]; }
    p.myws = p.ws[rank];
    build_schedule(p);
    HIP_CHECK(hipFuncSetAttribute((const void*)gemm_ar_kernel, hipFuncAttributeMaxDynamicSharedMemorySize, LDS_B));
    hipLaunchKernelGGL(gemm_ar_kernel, dim3(NLINK + NGEMM), dim3(512), LDS_B, stream, p);
    HIP_CHECK(hipGetLastError());
}
```

```cpp
#include <hip/hip_runtime.h>
#include <rccl/rccl.h>
#include <cstdio>

#define HIP_CHECK(expr) do { hipError_t _e = (expr); if (_e != hipSuccess) { \
    fprintf(stderr, "HIP error %s at %s:%d\n", hipGetErrorString(_e), __FILE__, __LINE__); \
    return; } } while (0)

#define LAS __attribute__((address_space(3)))
#define GAS __attribute__((address_space(1)))
typedef short bf16x8 __attribute__((ext_vector_type(8)));
typedef short s16x4 __attribute__((ext_vector_type(4)));
typedef float f32x4 __attribute__((ext_vector_type(4)));
typedef unsigned u32x4 __attribute__((ext_vector_type(4)));
typedef unsigned u32x2 __attribute__((ext_vector_type(2)));

namespace {
constexpr int NR = 8;
constexpr int DIM = 8192, KL = 1024;
constexpr unsigned KT_B = 32768, HT_B = 16384;
constexpr unsigned SLOT_B = 2 * KT_B;
constexpr unsigned PANEL_B = 32 * SLOT_B;
constexpr unsigned STAGE_B = 32 * PANEL_B;
constexpr unsigned TILE_B = 131072;
constexpr unsigned PIN_OFF = STAGE_B;
constexpr unsigned POUT_OFF = PIN_OFF + 128 * TILE_B;
constexpr unsigned SEND_OFF = POUT_OFF + 128 * TILE_B;
constexpr unsigned SCR_OFF = SEND_OFF + 2048 * HT_B;
constexpr unsigned SCR_B = 256 * TILE_B;
constexpr unsigned DATA_B = SCR_OFF + SCR_B;
constexpr unsigned CNT_OFF = DATA_B;
constexpr unsigned PF_OFF = DATA_B + 4096;
constexpr unsigned IR_OFF = DATA_B + 8192;
constexpr unsigned PL_OFF = DATA_B + 16384;
constexpr unsigned SF_OFF = DATA_B + 20480;
constexpr unsigned FL_OFF = DATA_B + 32768;
constexpr unsigned HV_OFF = DATA_B + 65536;
constexpr unsigned DONE_OFF = DATA_B + 98304;
constexpr unsigned TAIL_OFF = DONE_OFF + 1024;
constexpr unsigned CALL_OFF = DONE_OFF + 2048;
constexpr unsigned TMO_OFF = CALL_OFF + 128;
constexpr unsigned HEAD_OFF = DONE_OFF + 4096;
constexpr size_t WS_NEED = (size_t)DONE_OFF + 8192;
constexpr unsigned OUT_B = (unsigned)DIM * DIM * 2;
constexpr int NE = 8;
constexpr int NS = 2;
constexpr int NLINK = 8 * NE, NGEMM = 256 - NLINK;
constexpr int NURG = NGEMM / 4;
constexpr int LDS_B = 131072 + 256;
constexpr unsigned long long DEADLINE = 100000000ull;
constexpr int AUX_SYS = 17;
constexpr int AUX_DEV = 16;
#define WIN 3
#define TAILWIN 6
constexpr int TAILN = 64;
constexpr int CLAIM_AT = 10;
constexpr int FLEX_AT = 32;
constexpr int KSL = 3, KS3 = 2;
constexpr int NSPLIT = 40;
constexpr int NLATE = 96;
constexpr int URG0 = 48;

struct Params {
    const char* x; const char* w;
    char* myws;
    int rank;
    char* out[NR]; char* ws[NR];
    unsigned units[32];
    unsigned late[NGEMM / 4];
    unsigned tiles[NGEMM];
};

#define STR2(x) #x
#define STR(x) STR2(x)
#define WAIT_V(n) asm volatile("s_waitcnt vmcnt(" STR(n) ")" ::: "memory")
#define WAIT_L(n) asm volatile("s_waitcnt lgkmcnt(" STR(n) ")" ::: "memory")
#define BAR __builtin_amdgcn_s_barrier()
#define SCHED __builtin_amdgcn_sched_barrier(0)


__device__ __forceinline__ void touch_args(const Params& p, int u) {
    const int a = p.rank; const unsigned b = p.units[u]; char* const c = p.out[4]; char* const d = p.ws[4];
    asm volatile("" :: "s"(a), "s"(b), "s"(c), "s"(d));
}
__device__ __forceinline__ unsigned ld_sys(const char* p) {
    return __hip_atomic_load((const GAS unsigned*)p, __ATOMIC_RELAXED, __HIP_MEMORY_SCOPE_SYSTEM);
}
__device__ __forceinline__ void st_sys(char* p, unsigned v) {
    __hip_atomic_store((GAS unsigned*)p, v, __ATOMIC_RELAXED, __HIP_MEMORY_SCOPE_SYSTEM);
}
__device__ __forceinline__ void st8_sys(char* p, unsigned v) {
    __hip_atomic_store((GAS unsigned char*)p, (unsigned char)v, __ATOMIC_RELAXED, __HIP_MEMORY_SCOPE_SYSTEM);
}
__device__ __forceinline__ unsigned call_tag(unsigned call) { return call % 255u + 1u; }
__device__ __forceinline__ void add_sys(char* p, unsigned v) {
    __hip_atomic_fetch_add((GAS unsigned*)p, v, __ATOMIC_RELAXED, __HIP_MEMORY_SCOPE_SYSTEM);
}
__device__ __forceinline__ void draw8(unsigned& pos, char* ws, unsigned off) {
    asm volatile("global_atomic_add %0, %1, %2, %3 sc0" : "+v"(pos) : "v"(off), "v"(8u), "s"(ws) : "memory");
}
__device__ __forceinline__ unsigned cvt_pk_bf16(float lo, float hi) {
    unsigned r; asm volatile("v_cvt_pk_bf16_f32 %0, %1, %2" : "=v"(r) : "v"(lo), "v"(hi)); return r;
}
__device__ __forceinline__ int panel_row(int d, int P) { return ((d >> 1) & 1) * 4096 + ((((P >> 3) ^ (d >> 2) ^ 1) & 1) * 8 + (P & 7)) * 256; }
__device__ __forceinline__ int panel_col(int d, int k) { return (d & 1) * 4096 + k * 256; }
__device__ __forceinline__ int link_slot(int dsel, int cc) { return (cc == 3 ? 8 : 4) + cc * 6 + (dsel < 4 ? dsel - 1 : dsel - 2); }
__device__ __forceinline__ bool give_up(char* tmo, unsigned call, unsigned long long t0) {
    if (__builtin_amdgcn_readfirstlane(__hip_atomic_load((const GAS unsigned*)tmo, __ATOMIC_RELAXED, __HIP_MEMORY_SCOPE_AGENT)) == call
        || __builtin_amdgcn_s_memrealtime() - t0 > DEADLINE) {
        __hip_atomic_store((GAS unsigned*)tmo, call, __ATOMIC_RELAXED, __HIP_MEMORY_SCOPE_AGENT);
        return true;
    }
    return false;
}

struct Piece { const char* src; unsigned stride, dst, pub; int kind; };

__device__ __forceinline__ Piece shard_piece(const Params& p, int d, int panel, int cc, int pc, int dp, int slot) {
    const int k0 = ((d >> 2) * 4 + cc) * 128 + (pc >> 1) * 64, h = pc & 1;
    Piece o; o.kind = panel >= 16;
    o.dst = (unsigned)dp * PANEL_B + (unsigned)slot * SLOT_B + (unsigned)pc * HT_B;
    o.pub = CNT_OFF + (unsigned)dp * 128 + (unsigned)slot * 4 + (unsigned)pc;
    if (o.kind) {
        o.src = p.w + ((size_t)k0 * DIM + panel_col(d, panel - 16) + h * 128) * 2;
        o.stride = 4u * DIM * 2;
    } else {
        o.src = p.x + ((size_t)(panel_row(d, panel) + h * 128) * KL + k0) * 2;
        o.stride = 8u * KL * 2;
    }
    return o;
}
__device__ __forceinline__ int unit_at(const Params& p, int n) { return (p.units[n >> 2] >> ((n & 3) * 8)) & 255; }
__device__ __forceinline__ unsigned send_off(int d, int panel, int cc, int pc) {
    const int gp = panel >= 16 ? 32 + (panel_col(d, panel - 16) >> 8) : panel_row(d, panel) >> 8;
    return SEND_OFF + (unsigned)((gp * 8 + (d >> 2) * 4 + cc) * 4 + pc) * HT_B;
}
__device__ __forceinline__ Piece pull_piece(const Params& p, int code, int dsel) {
    const int cls = code >> 21, pos = (code >> 10) & 2047, id = code & 1023, s = p.rank ^ dsel;
    Piece o; o.kind = 2; o.stride = 1024;
    if (cls == 1) {
        const int u = unit_at(p, (pos >> 2) & 127), panel = u & 31, cc = u >> 5, pc = pos & 3, slot = link_slot(dsel, cc);
        o.src = p.ws[s] + send_off(p.rank, panel, cc, pc);
        o.dst = (unsigned)panel * PANEL_B + (unsigned)slot * SLOT_B + (unsigned)pc * HT_B;
        o.pub = CNT_OFF + (unsigned)panel * 128 + (unsigned)slot * 4 + (unsigned)pc;
    } else if (cls == 0) {
        o.src = p.ws[s] + POUT_OFF + (size_t)id * HT_B;
        o.dst = PIN_OFF + (unsigned)id * HT_B;
        o.pub = PF_OFF + (unsigned)id * 4;
    } else {
        const int ow = cls == 2 ? s : p.rank ^ 4;
        o.kind = 3; o.stride = 2u * DIM * 2;
        o.dst = (unsigned)(panel_row(ow, 8 + (id >> 7)) + (id & 7) * 32) * (DIM * 2) + (unsigned)panel_col(ow, (id >> 3) & 15) * 2;
        o.src = p.out[s] + o.dst;
        o.pub = cls == 2 ? (unsigned)(code & 0xfffff) + 1u : 0u;
    }
    return o;
}
__device__ __forceinline__ const char* piece_src(const Piece& pc, int lane) {
    const unsigned xl = (unsigned)(lane >> 3) * (KL * 2) + (unsigned)((lane & 7) ^ (lane >> 3)) * 16;
    const unsigned wl = (unsigned)(lane >> 4) * (DIM * 2) + (unsigned)((lane & 15) ^ ((lane >> 4) << 2)) * 16;
    const unsigned ol = (unsigned)(lane >> 5) * (DIM * 2) + (unsigned)(lane & 31) * 16;
    return pc.src + (pc.kind == 0 ? xl : pc.kind == 1 ? wl : pc.kind == 2 ? (unsigned)lane * 16 : ol);
}
__device__ __forceinline__ void piece_dma(const Piece& pc, LAS char* buf, int lane) {
    const char* s = piece_src(pc, lane);
#pragma unroll
    for (int f = 0; f < 16; ++f) __builtin_amdgcn_global_load_lds((const GAS unsigned*)(s + (size_t)f * pc.stride), (LAS unsigned*)(buf + f * 1024), 16, 0, AUX_DEV);
}
__device__ __forceinline__ void piece_read_tr(u32x4 (&v)[16], LAS char* buf, int lane) {
    const int q = (lane >> 2) & 3;
    const unsigned trl = (unsigned)(8 * (lane >> 4) + q) * 256 + (unsigned)(lane & 3) * 16;
#pragma unroll
    for (int f = 0; f < 16; ++f) {
        const int nb = f >> 1, kb = f & 1;
        LAS char* a = buf + trl + (((nb >> 1) ^ q) * 64) + kb * 8192 + (nb & 1) * 8;
        const u32x2 lo = __builtin_bit_cast(u32x2, __builtin_amdgcn_ds_read_tr16_b64_v4i16((LAS s16x4*)a));
        const u32x2 hi = __builtin_bit_cast(u32x2, __builtin_amdgcn_ds_read_tr16_b64_v4i16((LAS s16x4*)(a + 1024)));
        v[f] = (u32x4){lo.x, lo.y, hi.x, hi.y};
    }
}
__device__ __forceinline__ void piece_read(u32x4 (&v)[16], bool isx, LAS char* buf, int lane) {
    const unsigned xb = (unsigned)(lane & 15) * 128 + (unsigned)((lane >> 4) ^ (lane & 7)) * 16;
    LAS char* b0 = buf + (isx ? xb : (unsigned)lane * 16);
    LAS char* b1 = buf + (isx ? xb ^ 64u : (unsigned)lane * 16 + 1024u);
#pragma unroll
    for (int f = 0; f < 16; ++f) asm volatile("ds_read_b128 %0, %1 offset:%2" : "=v"(v[f]) : "v"((f & 1) ? b1 : b0), "i"((f >> 1) * 2048) : "memory");
    asm volatile("s_waitcnt lgkmcnt(0)" : "+v"(v[0]), "+v"(v[1]), "+v"(v[2]), "+v"(v[3]), "+v"(v[4]), "+v"(v[5]), "+v"(v[6]), "+v"(v[7]),
                 "+v"(v[8]), "+v"(v[9]), "+v"(v[10]), "+v"(v[11]), "+v"(v[12]), "+v"(v[13]), "+v"(v[14]), "+v"(v[15]) :: "memory");
}

constexpr int NMB = 2 * KSL + KS3;

__device__ __forceinline__ int next_task(LAS unsigned* mb, int lane) {
    for (;;) {
        unsigned v;
        asm volatile("ds_read_b32 %0, %1\n\ts_waitcnt lgkmcnt(0)" : "=&v"(v) : "v"(mb + (lane & 15)) : "memory");
        unsigned best = 0xffffffffu; int bl = 0;
#pragma unroll
        for (int l = 0; l < NMB; ++l) { const unsigned k = (unsigned)__builtin_amdgcn_readlane(v, l) - 1u; if (k < best) { best = k; bl = l; } }
        if (best == 0xffffffffu) return __builtin_amdgcn_readlane(v, NMB) ? -2 : -1;
        unsigned old = 0;
        if (lane == 0) asm volatile("ds_cmpst_rtn_b32 %0, %1, %2, %3\n\ts_waitcnt lgkmcnt(0)" : "=&v"(old) : "v"(mb + bl), "v"(best + 1u), "v"(0u) : "memory");
        if ((unsigned)__builtin_amdgcn_readfirstlane(old) == best + 1u) return (int)(best & 0x7fffffu);
    }
}
static_assert(WIN >= 2 && WIN <= 8 && NS >= 1 && NS <= 3, "window / pullers");
__device__ __forceinline__ void lds_wait_zero(LAS unsigned* a) {
    for (;;) {
        unsigned f;
        asm volatile("ds_read_b32 %0, %1\n\ts_waitcnt lgkmcnt(0)" : "=&v"(f) : "v"(a) : "memory");
        if (__builtin_amdgcn_readfirstlane(f) == 0) break;
        __builtin_amdgcn_s_sleep(1);
    }
}

__device__ __forceinline__ void link_wg(const Params& p, LAS char* lds, int dsel, int e) {
    const int wid = __builtin_amdgcn_readfirstlane(threadIdx.x >> 6), lane = threadIdx.x & 63;
    char* ws = p.myws;
    LAS unsigned* mb = (LAS unsigned*)(lds + 7 * HT_B);
    const int sn = wid < NS ? wid : wid - NS;
    LAS char* buf = lds + sn * 2 * HT_B;
    LAS unsigned* slot = mb + 16 + sn * 8;
    touch_args(p, 0);
    Piece first;
    if (wid >= NS && wid < 2 * NS) {
        if (dsel != 4) {
            const int q = sn * NE + e, u = unit_at(p, q >> 2);
            first = shard_piece(p, p.rank ^ dsel, u & 31, u >> 5, q & 3, u & 31, link_slot(dsel, u >> 5));
            piece_dma(first, buf + HT_B, lane);
        }
    } else if (wid == 7) {
        const unsigned c = ld_sys(ws + CALL_OFF) + 1u;
        mb[lane] = lane == 15 ? c : (lane >= 16 && (lane & 7) == 4 && dsel != 4) ? 14u : 0u; WAIT_L(0);
        if (lane == 0) add_sys(ws + DONE_OFF, 1u);
    }
    BAR; asm volatile("" ::: "memory");
    unsigned call;
    asm volatile("ds_read_b32 %0, %1 offset:60\n\ts_waitcnt lgkmcnt(0)" : "=&v"(call) : "v"(mb) : "memory");
    call = __builtin_amdgcn_readfirstlane(call);
    if (wid >= 2 * NS) {
        if (wid != 7) return;
        const int cls = lane < KSL ? (dsel == 4 ? 0 : 1) : lane < 2 * KSL ? 2 : 3;
        const unsigned nfirst = cls == 1 ? (NS + KSL) * NE : 0u;
        const unsigned n = cls == 1 ? 512u : 1024u;
        const unsigned base = (call - 1u) * (n - nfirst + NE * (cls == 3 ? 7 * KS3 : KSL)) - nfirst;
        char* head = ws + HEAD_OFF + (cls == 3 ? 0 : dsel * 2 + (cls >> 1)) * 128;
        const char* list = (cls == 3 && dsel != 4) ? p.ws[p.rank ^ dsel] + HV_OFF + (dsel ^ 4) * 4096
                                                     : ws + (cls == 0 ? PL_OFF : cls == 1 ? IR_OFF + dsel * 1024 : FL_OFF + dsel * 4096);
        const unsigned tag = call & 0xfffffu;
        int st = (lane >= NMB || (cls == 2 && dsel == 4)) ? 3 : (cls == 3 && dsel != 4) ? 4 : 0;
        unsigned pos = (NS + lane) * NE + e, lo = pos & ~3u;
        if (cls == 1) st = 1;
        const unsigned long long t0 = __builtin_amdgcn_s_memrealtime();
        for (unsigned polls = 1;; ++polls) {
            if (st == 2 && __hip_atomic_load(mb + lane, __ATOMIC_RELAXED, __HIP_MEMORY_SCOPE_WORKGROUP) == 0u) st = 0;
            unsigned t = 0, en = 0;
            if (st == 0) t = __hip_atomic_fetch_add((GAS unsigned*)head, 1u, __ATOMIC_RELAXED, __HIP_MEMORY_SCOPE_AGENT);
            if (st == 1) en = ld_sys(list + lo);
            if (st == 0) {
                pos = t - base; st = pos >= n ? 3 : 1;
                lo = cls != 1 ? pos * 4 : (pos < 4 * NURG ? pos & ~3u : 4u * NURG);
            } else if (st == 1 && (en >> 12) == tag) {
                __hip_atomic_store(mb + lane, ((((unsigned)cls << 11 | pos) << 10) | (en & 1023u)) + 1u, __ATOMIC_RELAXED, __HIP_MEMORY_SCOPE_WORKGROUP);
                st = 2;
            }
            if (__any(cls == 2 && st != 0 && pos >= 1024u - FLEX_AT) && st == 4) st = 0;
            if (__all(st == 3)) break;
            if ((polls & 63u) == 0u && give_up(ws + TMO_OFF, call, t0)) {
                if (lane == 0) st_sys(p.out[p.rank] + ((size_t)panel_row(p.rank, 8) * DIM + (size_t)panel_col(p.rank, 0)) * 2, 0x7fc07fc0u);
                break;
            }
            __builtin_amdgcn_s_sleep(4);
        }
        if (lane == 0) __hip_atomic_store(mb + NMB, 1u, __ATOMIC_RELAXED, __HIP_MEMORY_SCOPE_WORKGROUP);
        return;
    }
    if (wid < NS) {
        int k = 0, code = -1;
        bool pend = false;
        u32x4 desc = {0u, 0u, 0u, 0u};
        Piece cur;
        for (;;) {
            if (code < 0) {
                if (code == -1) code = next_task(mb, lane);
                if (code < 0) {
                    if (pend) { WAIT_V(0); if (lane == 0) asm volatile("ds_write_b128 %0, %1" :: "v"(slot + ((k - 1) & 1) * 4), "v"(desc) : "memory"); pend = false; }
                    while (code == -1) { __builtin_amdgcn_s_sleep(2); code = next_task(mb, lane); }
                    if (code < 0) break;
                }
                cur = pull_piece(p, code, dsel);
            }
            lds_wait_zero(slot + (k & 1) * 4);
            LAS char* b = buf + (k & 1) * HT_B;
            const char* sp = piece_src(cur, lane);
            const unsigned stride = cur.stride;
            const u32x4 ndesc = {(unsigned)cur.kind + 1u, cur.dst, cur.pub, 0u};
            const bool fast = (code >> 10) >= (3 << 11 | (1024 - TAILN));
            const bool wide = sn == 0 && (code >> 22) == 0;
#pragma unroll
            for (int f = 0; f < 16; ++f) {
                __builtin_amdgcn_global_load_lds((const GAS unsigned*)sp, (LAS unsigned*)(b + f * 1024), 16, 0, AUX_SYS);
                sp += stride;
                asm volatile("" ::: "memory");
                if (f == CLAIM_AT) { code = next_task(mb, lane); if (code >= 0) cur = pull_piece(p, code, dsel); }
                if (f > WIN - 2 && fast) WAIT_V(TAILWIN - 1); else if (f != WIN - 2 && wide) WAIT_V(WIN); else WAIT_V(WIN - 1);
                if (f == WIN - 2 && pend && lane == 0)
                    asm volatile("ds_write_b128 %0, %1" :: "v"(slot + ((k - 1) & 1) * 4), "v"(desc) : "memory");
            }
            desc = ndesc; pend = true; ++k;
        }
        lds_wait_zero(slot + (k & 1) * 4);
        if (lane == 0) asm volatile("ds_write_b32 %0, %1" :: "v"(slot + (k & 1) * 4), "v"(15u) : "memory");
        return;
    }
    if (dsel != 4) {
        char* dws = p.ws[p.rank ^ dsel];
        const __amdgpu_buffer_rsrc_t rs = __builtin_amdgcn_make_buffer_rsrc(dws, 0, STAGE_B, 0x00020000);
        WAIT_V(0);
        u32x4 v[16];
        if (first.kind) { piece_read_tr(v, buf + HT_B, lane); WAIT_L(0);
#pragma unroll
            for (int f = 0; f < 16; ++f) asm volatile("" : "+v"(v[f]));
        } else piece_read(v, true, buf + HT_B, lane);
        if (lane == 0) asm volatile("ds_write_b32 %0, %1" :: "v"(slot + 4), "v"(0u) : "memory");
#pragma unroll
        for (int f = 0; f < 16; ++f) { __builtin_amdgcn_raw_buffer_store_b128(v[f], rs, first.dst + lane * 16 + f * 1024, 0, AUX_SYS); WAIT_V(WIN - 1); }
        WAIT_V(0);
        if (lane == 0) st8_sys(dws + first.pub, call_tag(call));
    }
    const __amdgpu_buffer_rsrc_t rsw = __builtin_amdgcn_make_buffer_rsrc(ws, 0, POUT_OFF, 0x00020000);
    const __amdgpu_buffer_rsrc_t rso = __builtin_amdgcn_make_buffer_rsrc(p.out[p.rank], 0, OUT_B, 0x00020000);
    const unsigned ol = (unsigned)(lane >> 5) * (DIM * 2) + (unsigned)(lane & 31) * 16;
    char* fwd = ws + HV_OFF + dsel * 4096;
    for (int k = 0;;) {
        u32x4 ds4;
        asm volatile("ds_read_b128 %0, %1\n\ts_waitcnt lgkmcnt(0)" : "=&v"(ds4) : "v"(slot + (k & 1) * 4) : "memory");
        const int kind = __builtin_amdgcn_readfirstlane(ds4.x) - 1;
        if (kind > 3) break;
        if (kind < 0) { __builtin_amdgcn_s_sleep(1); continue; }
        const unsigned dst = __builtin_amdgcn_readfirstlane(ds4.y), pub = __builtin_amdgcn_readfirstlane(ds4.z);
        u32x4 v[16];
        piece_read(v, false, buf + (k & 1) * HT_B, lane);
        if (lane == 0) asm volatile("ds_write_b32 %0, %1" :: "v"(slot + (k & 1) * 4), "v"(0u) : "memory");
        const bool o = kind == 3;
        const __amdgpu_buffer_rsrc_t rs = o ? rso : rsw;
        const unsigned ds = o ? 2u * DIM * 2 : 1024u;
        unsigned vo = dst + (o ? ol : (unsigned)lane * 16);
#pragma unroll
        for (int f = 0; f < 16; ++f) {
            __builtin_amdgcn_raw_buffer_store_b128(v[f], rs, vo, 0, AUX_SYS);
            vo += ds;
        }
        if (pub) {
            WAIT_V(0);
            if (lane == 0) {
                if (o) st_sys(fwd + ((pub - 1u) >> 10) * 4, (call & 0xfffffu) << 12 | ((pub - 1u) & 1023u));
                else if (pub >= PF_OFF) st_sys(ws + pub, call);
                else st8_sys(ws + pub, call_tag(call));
            }
        }
        ++k;
    }
}

__device__ __forceinline__ int ready_prefix(unsigned v, unsigned target, unsigned low) {
    const unsigned long long m = __ballot(v == target);
    const unsigned both = ((unsigned)m & (unsigned)(m >> 32)) | low;
    return both == 0xffffffffu ? 32 : __builtin_ctz(~both);
}
__device__ __forceinline__ void kn_put(LAS unsigned* kn, unsigned v) { asm volatile("ds_write_b32 %0, %1" :: "v"(kn), "v"(v) : "memory"); }
__device__ __noinline__ int wait_slot(const char* cptr, char* tmo, unsigned call, int need, unsigned low, LAS unsigned* kn, int job, int duty) {
    int known;
    if (__builtin_amdgcn_readfirstlane(duty)) {
        const unsigned long long t0 = __builtin_amdgcn_s_memrealtime();
        for (unsigned polls = 1;; ++polls) {
            known = ready_prefix(ld_sys(cptr), call_tag(call) * 0x01010101u, low);
            if (known > need) break;
            if ((polls & 63u) == 0u && give_up(tmo, call, t0)) { known = 32; break; }
            __builtin_amdgcn_s_sleep(32);
        }
        kn_put(kn, (unsigned)(job + known));
    } else {
        for (;;) {
            unsigned f;
            asm volatile("ds_read_b32 %0, %1\n\ts_waitcnt lgkmcnt(0)" : "=&v"(f) : "v"(kn) : "memory");
            known = (int)f - job;
            if (known > need) break;
            __builtin_amdgcn_s_sleep(4);
        }
    }
    return known;
}

__device__ __forceinline__ void gemm_wg(const Params& p, LAS char* lds, int gb) {
    const int tid = threadIdx.x, wid = __builtin_amdgcn_readfirstlane(tid >> 6), lane = tid & 63, wr = wid >> 2, wc = wid & 3;
    char* ws = p.myws;
    unsigned cv;
    asm volatile("global_load_dword %0, %1, %2 sc0 sc1" : "=v"(cv) : "v"(CALL_OFF), "s"(ws) : "memory");
    const int late = (int)(signed char)((p.late[gb >> 2] >> ((gb & 3) * 8)) & 255);
    const bool bulk = wid >= 4;
    LAS unsigned* ucnt = (LAS unsigned*)(lds + 131072);
    if (tid < 2) ucnt[tid] = 0u;
    touch_args(p, gb >> 4);
    WAIT_L(0); BAR; asm volatile("" ::: "memory");
    for (int i = bulk ? (late >= 0 ? 4 + wid : 0) : (gb < URG0 ? 0 : 2 + ((gb - URG0) >> 5)); i > 0; --i) __builtin_amdgcn_s_sleep(32);
    unsigned nbulk = 0;
    for (int r = !bulk ? gb * 4 + wid : late >= 0 ? 4 * NGEMM + (wid - 4) * NLATE + late : 2048; r < 2048; r = bulk ? r + 4 * NLATE : 2048) {
        const int other = (r >> 3) & 1, half = (r >> 2) & 1, pcs = r & 3;
        const int u = unit_at(p, r >> 4), panel = u & 31, cc = u >> 5;
        const int d0 = (((p.rank >> 2) ^ other) << 2) + (panel >= 16 ? half : half * 2), d1 = d0 + (panel >= 16 ? 2 : 1);
        const bool own0 = ((d0 ^ p.rank) & 3) == 0, own1 = ((d1 ^ p.rank) & 3) == 0, own = own0 || own1;
        const Piece pc = shard_piece(p, d0, panel, cc, pcs, panel < 16 && other ? panel ^ 8 : panel, other * 22 + cc);
        LAS char* buf = lds + wid * HT_B;
        piece_dma(pc, buf, lane);
        asm volatile("s_waitcnt vmcnt(0)" : "+v"(cv) :: "memory");
        const unsigned call = __builtin_amdgcn_readfirstlane(cv) + 1u;
        u32x4 v[16];
        if (pc.kind) piece_read_tr(v, buf, lane); else piece_read(v, true, buf, lane);
        const __amdgpu_buffer_rsrc_t rs = __builtin_amdgcn_make_buffer_rsrc(ws, 0, DATA_B, 0x00020000);
        const unsigned so = send_off(d0, panel, cc, pcs);
#pragma unroll
        for (int f = 0; f < 16; ++f) __builtin_amdgcn_raw_buffer_store_b128(v[f], rs, so + lane * 16 + f * 1024, 0, AUX_SYS);
        if (own) {
#pragma unroll
            for (int f = 0; f < 16; ++f) __builtin_amdgcn_raw_buffer_store_b128(v[f], rs, pc.dst + lane * 16 + f * 1024, 0, AUX_SYS);
        }
        WAIT_V(0);
        ++nbulk;
        if (lane == 0) {
            if (own) st8_sys(ws + pc.pub, call_tag(call));
            if (!bulk && __hip_atomic_fetch_add(ucnt, 1u, __ATOMIC_RELAXED, __HIP_MEMORY_SCOPE_WORKGROUP) == 3u) {
                if (!own0) st_sys(p.ws[d0] + IR_OFF + (p.rank ^ d0) * 1024 + (r >> 4) * 4, (call & 0xfffffu) << 12);
                if (!own1) st_sys(p.ws[d1] + IR_OFF + (p.rank ^ d1) * 1024 + (r >> 4) * 4, (call & 0xfffffu) << 12);
            }
        }
    }
    asm volatile("s_waitcnt vmcnt(0)" : "+v"(cv) :: "memory");
    const unsigned call = __builtin_amdgcn_readfirstlane(cv) + 1u;
    if (bulk && nbulk && lane == 0 &&
        __hip_atomic_fetch_add((GAS unsigned*)(ws + TAIL_OFF + 256), nbulk, __ATOMIC_RELAXED, __HIP_MEMORY_SCOPE_AGENT) + nbulk == call * (2048u - 4u * NGEMM)) {
#pragma unroll
        for (int q = 1; q < NR; ++q) if (q != 4) st_sys(p.ws[p.rank ^ q] + IR_OFF + q * 1024 + NURG * 4, (call & 0xfffffu) << 12);
    }
    BAR; asm volatile("" ::: "memory");
    if (tid == 0) add_sys(ws + DONE_OFF, 1u);
    const unsigned target = call_tag(call) * 0x01010101u;
    const unsigned voff = (unsigned)tid * 16;
    const unsigned ldsw = (unsigned)wid * 1024u;
    const int aoff = wr * 8192 + lane * 16, boff = wc * 4096 + lane * 16;
#define SA(b, h) (((b) * 2 + (h)) * HT_B)
#define SB(b, h) ((4 + (b) * 2 + (h)) * HT_B)
#define STAGE(bufoff, gbase) do { _Pragma("unroll") for (int _i = 0; _i < 2; ++_i) \
        __builtin_amdgcn_global_load_lds((const GAS unsigned*)((const char*)(gbase) + voff + _i * 8192), (LAS unsigned*)(lds + (bufoff) + ldsw + _i * 8192), 16, 0, AUX_DEV); } while (0)
#define LDA(dst, b, h) do { _Pragma("unroll") for (int m = 0; m < 4; ++m) _Pragma("unroll") for (int k = 0; k < 2; ++k) dst[m][k] = *(const LAS bf16x8*)(lds + SA(b, h) + aoff + m * 2048 + k * 1024); } while (0)
#define LDB(dst, b, h) do { _Pragma("unroll") for (int n = 0; n < 2; ++n) _Pragma("unroll") for (int k = 0; k < 2; ++k) dst[n][k] = *(const LAS bf16x8*)(lds + SB(b, h) + boff + n * 2048 + k * 1024); } while (0)
#define MMA(ai, bj, At, Bt) do { __builtin_amdgcn_s_setprio(1); _Pragma("unroll") for (int m = 0; m < 4; ++m) _Pragma("unroll") for (int n = 0; n < 2; ++n) _Pragma("unroll") for (int k = 0; k < 2; ++k) \
        acc[ai][bj][m][n] = __builtin_amdgcn_mfma_f32_16x16x32_bf16(Bt[n][k], At[m][k], acc[ai][bj][m][n], 0, 0, 0); __builtin_amdgcn_s_setprio(0); } while (0)
    for (int rnd = 0; rnd < 2; ++rnd) {
        const int code = (p.tiles[gb] >> (16 * rnd)) & 0xffff;
        if (code == 0xffff) break;
        const int pm = (code >> 4) & 15, pn = code & 15, part = (code >> 8) & 1, np = 1 + (code >> 9);
        const unsigned low = np == 1 ? 0u : 0xaaaaaaaau >> part;
        const char* Ab = ws + (size_t)pm * PANEL_B;
        const char* Bb = ws + (size_t)(16 + pn) * PANEL_B;
        const unsigned coff = CNT_OFF + (unsigned)(lane < 32 ? pm : 16 + pn) * 128 + (unsigned)(lane & 31) * 4;
        f32x4 acc[2][2][4][2];
#pragma unroll
        for (int a = 0; a < 2; ++a)
#pragma unroll
            for (int b = 0; b < 2; ++b)
#pragma unroll
                for (int m = 0; m < 4; ++m)
#pragma unroll
                    for (int n = 0; n < 2; ++n) acc[a][b][m][n] = (f32x4){0.f, 0.f, 0.f, 0.f};
        bf16x8 At[4][2], B0[2][2], B1[2][2];
        int known = __builtin_amdgcn_readfirstlane(wait_slot(ws + coff, ws + TMO_OFF, call, part, low, ucnt + 1, rnd * 64, wid == 0));
        {
            const char* a0 = Ab + (unsigned)part * SLOT_B; const char* b0 = Bb + (unsigned)part * SLOT_B;
            STAGE(SB(0, 0), b0); STAGE(SA(0, 0), a0); STAGE(SB(0, 1), b0 + HT_B); STAGE(SA(0, 1), a0 + HT_B);
            if (wr == 1) BAR;
            WAIT_V(4); BAR;
            STAGE(SB(1, 0), b0 + KT_B); STAGE(SA(1, 0), a0 + KT_B); STAGE(SB(1, 1), b0 + KT_B + HT_B);
            WAIT_V(6); BAR;
        }
        for (int T = part; T < 32; T += np) {
            if (T + np < 32 && known <= T + np) known = __builtin_amdgcn_readfirstlane(wait_slot(ws + coff, ws + TMO_OFF, call, T + np, low, ucnt + 1, rnd * 64, wid == 0));
            const unsigned o1 = (unsigned)T * SLOT_B, o2 = (unsigned)(T + np < 32 ? T + np : T) * SLOT_B;
            const char* a1 = Ab + o1 + KT_B; const char* a2 = Ab + o2; const char* b2 = Bb + o2;
            const char* a3 = a2 + KT_B; const char* b3 = b2 + KT_B;
            unsigned pv = 0;
            const bool ask = known < 32;
            LDB(B0, 0, 0); SCHED; LDA(At, 0, 0);
            if (ask) asm volatile("global_load_dword %0, %1, %2 sc0 sc1" : "+v"(pv) : "v"(coff), "s"(ws) : "memory");
            STAGE(SA(1, 1), a1 + HT_B);
            WAIT_L(8); BAR; WAIT_L(0); MMA(0, 0, At, B0); BAR; SCHED;
            LDB(B1, 0, 1); STAGE(SB(0, 0), b2);
            BAR; WAIT_L(0); MMA(0, 1, At, B1); BAR;
            LDA(At, 0, 1); STAGE(SA(0, 0), a2);
            BAR; WAIT_L(0); MMA(1, 0, At, B0); BAR; SCHED;
            STAGE(SB(0, 1), b2 + HT_B);
            WAIT_V(6);
            asm volatile("" : "+v"(pv));
            { const int k2 = ready_prefix(pv, target, low); if (k2 > known) { known = k2; if (wid == 0) kn_put(ucnt + 1, (unsigned)(rnd * 64 + k2)); } }
            BAR; MMA(1, 1, At, B1); BAR;
            LDB(B0, 1, 0); SCHED; LDA(At, 1, 0); STAGE(SA(0, 1), a2 + HT_B);
            WAIT_L(8); BAR; WAIT_L(0); MMA(0, 0, At, B0); BAR; SCHED;
            LDB(B1, 1, 1); STAGE(SB(1, 0), b3);
            BAR; WAIT_L(0); MMA(0, 1, At, B1); BAR;
            LDA(At, 1, 1); STAGE(SA(1, 0), a3);
            BAR; WAIT_L(0); MMA(1, 0, At, B0); BAR; SCHED;
            STAGE(SB(1, 1), b3 + HT_B);
            WAIT_V(6); BAR; MMA(1, 1, At, B1); BAR;
        }
        WAIT_V(0);
        if (wr == 0) BAR;
        BAR; asm volatile("" ::: "memory");
        int tz = threadIdx.x; asm volatile("" : "+v"(tz));
        const int ln = tz & 63;
        const int id = ((pm & 7) * 16 + pn) * 8 + wid;
        if (part + 1 < np) {
            const __amdgpu_buffer_rsrc_t rq = __builtin_amdgcn_make_buffer_rsrc(ws + SCR_OFF, 0, SCR_B, 0x00020000);
            const unsigned base = (unsigned)id * (2 * HT_B) + (unsigned)ln * 16;
#pragma unroll
            for (int f = 0; f < 32; ++f) __builtin_amdgcn_raw_buffer_store_b128(__builtin_bit_cast(u32x4, acc[f >> 4][(f >> 1) & 1][(f >> 2) & 3][f & 1]), rq, base + f * 1024, 0, AUX_SYS);
            WAIT_V(0);
            if (ln == 0) st_sys(ws + SF_OFF + id * 4, call);
            continue;
        }
        if (pm < 8) {
            unsigned pos = 0;
            if (tz == 0) draw8(pos, ws, TAIL_OFF);
            if (np > 1) {
                const unsigned long long t0 = __builtin_amdgcn_s_memrealtime();
                for (unsigned polls = 1;; ++polls) {
                    if ((int)(__builtin_amdgcn_readfirstlane(ld_sys(ws + SF_OFF + id * 4)) - call) >= 0) break;
                    if ((polls & 63u) == 0u && give_up(ws + TMO_OFF, call, t0)) break;
                    __builtin_amdgcn_s_sleep(4);
                }
                asm volatile("" ::: "memory");
            }
            const bool bad = __hip_atomic_load((const GAS unsigned*)(ws + TMO_OFF), __ATOMIC_RELAXED, __HIP_MEMORY_SCOPE_AGENT) == call;
            const __amdgpu_buffer_rsrc_t rq = __builtin_amdgcn_make_buffer_rsrc(ws + SCR_OFF, 0, (unsigned)part * SCR_B, 0x00020000);
            const __amdgpu_buffer_rsrc_t rp = __builtin_amdgcn_make_buffer_rsrc(ws + POUT_OFF, 0, 128 * TILE_B, 0x00020000);
            const unsigned base = (unsigned)id * HT_B + (unsigned)ln * 16;
#pragma unroll
            for (int ai = 0; ai < 2; ++ai) {
                f32x4 pr[16];
#pragma unroll
                for (int f = 0; f < 16; ++f) pr[f] = __builtin_bit_cast(f32x4, __builtin_amdgcn_raw_buffer_load_b128(rq, base + (unsigned)id * HT_B + (ai * 16 + f) * 1024, 0, AUX_SYS));
#pragma unroll
                for (int m = 0; m < 4; ++m)
#pragma unroll
                    for (int bj = 0; bj < 2; ++bj) {
                        const f32x4 v0 = acc[ai][bj][m][0] + pr[m * 4 + bj * 2], v1 = acc[ai][bj][m][1] + pr[m * 4 + bj * 2 + 1];
                        u32x4 w; w.x = cvt_pk_bf16(v0[0], v0[1]); w.y = cvt_pk_bf16(v0[2], v0[3]); w.z = cvt_pk_bf16(v1[0], v1[1]); w.w = cvt_pk_bf16(v1[2], v1[3]);
                        if (bad) w = (u32x4){0x7fc07fc0u, 0x7fc07fc0u, 0x7fc07fc0u, 0x7fc07fc0u};
                        __builtin_amdgcn_raw_buffer_store_b128(w, rp, base + (unsigned)((ai * 4 + m) * 2 + bj) * 1024, 0, AUX_SYS);
                    }
                asm volatile("" ::: "memory");
            }
            asm volatile("s_waitcnt vmcnt(0)" : "+v"(pos) :: "memory");
            BAR; asm volatile("" ::: "memory");
            pos = __builtin_amdgcn_readfirstlane(pos) & 1023u;
            if (tz < 8) st_sys(p.ws[p.rank ^ 4] + PL_OFF + (pos + tz) * 4, (call & 0xfffffu) << 12 | (unsigned)(id + tz));
            continue;
        }
        {
            const unsigned long long t0 = __builtin_amdgcn_s_memrealtime();
            for (unsigned polls = 1;; ++polls) {
                if ((int)(__builtin_amdgcn_readfirstlane(ld_sys(ws + PF_OFF + id * 4)) - call) >= 0) break;
                if ((polls & 63u) == 0u && give_up(ws + TMO_OFF, call, t0)) break;
                __builtin_amdgcn_s_sleep(127);
            }
            asm volatile("" ::: "memory");
            const bool bad = __builtin_amdgcn_readfirstlane(__hip_atomic_load((const GAS unsigned*)(ws + TMO_OFF), __ATOMIC_RELAXED, __HIP_MEMORY_SCOPE_AGENT)) == call;
            const __amdgpu_buffer_rsrc_t rp = __builtin_amdgcn_make_buffer_rsrc(ws + PIN_OFF, 0, 128 * TILE_B, 0x00020000);
            const unsigned base = (unsigned)id * HT_B + (unsigned)ln * 16;
            u32x4 pr[16];
#pragma unroll
            for (int f = 0; f < 16; ++f) pr[f] = __builtin_amdgcn_raw_buffer_load_b128(rp, base + f * 1024, 0, AUX_SYS);
            const int fr = ln & 15, fq = ln >> 4;
#pragma unroll
            for (int ai = 0; ai < 2; ++ai)
#pragma unroll
                for (int m = 0; m < 4; ++m) {
                    const int row = ai * 128 + wr * 64 + m * 16 + fr;
#pragma unroll
                    for (int bj = 0; bj < 2; ++bj) {
                        const u32x4 q = pr[(ai * 4 + m) * 2 + bj];
                        const f32x4 v0 = acc[ai][bj][m][0], v1 = acc[ai][bj][m][1];
                        u32x4 w;
                        w.x = cvt_pk_bf16(v0[0] + __uint_as_float(q.x << 16), v0[1] + __uint_as_float(q.x & 0xffff0000u));
                        w.y = cvt_pk_bf16(v0[2] + __uint_as_float(q.y << 16), v0[3] + __uint_as_float(q.y & 0xffff0000u));
                        w.z = cvt_pk_bf16(v1[0] + __uint_as_float(q.z << 16), v1[1] + __uint_as_float(q.z & 0xffff0000u));
                        w.w = cvt_pk_bf16(v1[2] + __uint_as_float(q.w << 16), v1[3] + __uint_as_float(q.w & 0xffff0000u));
                        if (bad) w = (u32x4){0x7fc07fc0u, 0x7fc07fc0u, 0x7fc07fc0u, 0x7fc07fc0u};
                        const int ch = (bj * 16 + wc * 4 + fq) ^ (fr & 7);
                        *(LAS u32x4*)(lds + row * 512 + ch * 16) = w;
                    }
                }
        }
        WAIT_L(0); BAR; asm volatile("" ::: "memory");
        {
            unsigned pos = 0;
            if (tz == 0) draw8(pos, ws, TAIL_OFF + 128);
            const __amdgpu_buffer_rsrc_t ro = __builtin_amdgcn_make_buffer_rsrc(p.out[p.rank], 0, OUT_B, 0x00020000);
            const unsigned vo = (unsigned)(panel_row(p.rank, pm) + wid * 32 + (ln >> 5)) * (DIM * 2) + (unsigned)panel_col(p.rank, pn) * 2 + (unsigned)(ln & 31) * 16;
            const int hi = ln >> 5, chn = ln & 31;
#pragma unroll
            for (int f = 0; f < 16; ++f) {
                const u32x4 v = *(const LAS u32x4*)(lds + (wid * 32 + 2 * f + hi) * 512 + ((chn ^ ((2 * f + hi) & 7)) * 16));
                __builtin_amdgcn_raw_buffer_store_b128(v, ro, vo + (unsigned)(2 * f) * (DIM * 2), 0, AUX_SYS);
            }
            asm volatile("s_waitcnt vmcnt(0)" : "+v"(pos) :: "memory");
            BAR; asm volatile("" ::: "memory");
            pos = __builtin_amdgcn_readfirstlane(pos) & 1023u;
            if (tz < 8) {
#pragma unroll
                for (int q = 1; q < NR; ++q) st_sys(p.ws[p.rank ^ q] + FL_OFF + (q * 1024 + pos + tz) * 4, (call & 0xfffffu) << 12 | (unsigned)(id + tz));
            }
        }
        break;
    }
#undef SA
#undef SB
#undef STAGE
#undef LDA
#undef LDB
#undef MMA
}

__global__ __launch_bounds__(512, 2) void gemm_ar_kernel(const Params p) {
    extern __shared__ __attribute__((aligned(16))) char smem[];
    LAS char* lds = (LAS char*)smem;
    const int b = blockIdx.x;
    if (b >= NLINK) { gemm_wg(p, lds, b - NLINK); return; }
    if ((b + (b >> 3)) & 7) { link_wg(p, lds, (b + (b >> 3)) & 7, b >> 3); return; }
    if (b == 0 && threadIdx.x < 64) {
        char* ws = p.myws;
        const unsigned call = __builtin_amdgcn_readfirstlane(ld_sys(ws + CALL_OFF)) + 1u;
        const unsigned long long t0 = __builtin_amdgcn_s_memrealtime();
        for (unsigned polls = 1;; ++polls) {
            if ((int)(__builtin_amdgcn_readfirstlane(ld_sys(ws + DONE_OFF)) - call * (unsigned)(7 * NE + NGEMM)) >= 0) break;
            if ((polls & 63u) == 0u && give_up(ws + TMO_OFF, call, t0)) break;
            __builtin_amdgcn_s_sleep(32);
        }
        if (threadIdx.x == 0) st_sys(ws + CALL_OFF, call);
    }
}

void build_schedule(Params& p) {
    static const int bt[] = {0, 16, 1, 17, 2, 18, 19, -1, 3, 4, 20, 21, -1, 8, 9, -1, 22, 23, -1, 5, 24, 25, -1, 10, 11, -1,
                             6, 26, 27, -1, 7, 28, 29, -1, 12, 13, -1, 30, 31, -1, 14, 15, -1};
    int pos[32], nu = 0, nb = 0;
    unsigned char* un = (unsigned char*)p.units;
    unsigned short* tl = (unsigned short*)p.tiles;
    for (int i = 0; i < (int)(sizeof(bt) / sizeof(bt[0])); ++nb) {
        int e = i; while (bt[e] >= 0) pos[bt[e++]] = nb;
        for (int c = 0; c < 4; ++c) for (int k = i; k < e; ++k) un[nu++] = (unsigned char)(bt[k] | c << 5);
        i = e + 1;
    }
    for (int i = 0; i < NGEMM; ++i) p.tiles[i] = 0xffffffffu;
    int next = 0, nf = 0;
    for (int own = 0; own < 2; ++own)
        for (int b = 0; b < nb; ++b) {
            int list[256], n = 0;
            for (int r = 0; r < 8; ++r) for (int c = 0; c < 16; ++c) {
                const int a = pos[own * 8 + r], bb = pos[16 + c], t = (own * 8 + r) << 4 | c;
                if ((a > bb ? a : bb) != b) continue;
                if (!own && nf++ < NSPLIT) { list[n++] = t | 2 << 8; list[n++] = t | 3 << 8; } else list[n++] = t;
            }
            int k = 0;
            for (int x = 0; x < 8; ++x) for (int s = next; s < next + n; ++s) if ((s & 7) == x)
                tl[s < NGEMM ? 2 * s : 2 * (s - NGEMM) + 1] = (unsigned short)list[k++];
            next += n;
        }
    signed char* lt = (signed char*)p.late;
    int nl = 0;
    for (int i = 0; i < NGEMM; ++i) lt[i] = -1;
    for (int b = nb - 1; b >= 0; --b) for (int i = NGEMM - 1; i >= 0 && nl < NLATE; --i) {
        const int a = pos[(tl[2 * i] >> 4) & 15], bb = pos[16 + (tl[2 * i] & 15)];
        if ((a > bb ? a : bb) == b) lt[i] = (signed char)nl++;
    }
}
static_assert(128 + NSPLIT <= NGEMM && NGEMM % 8 == 0, "foreign jobs are first jobs");
}

extern "C" void kernel_launch(int rank, int nranks, ncclComm_t comm,
                              hipStream_t stream,
                              void* const* d_in, const int* in_sizes, int n_in,
                              void* d_out, int out_size,
                              const int* in_dtypes, int out_dtype,
                              void* const* d_out_all,
                              void* const* d_ws_all, size_t ws_bytes) {
    (void)comm; (void)in_dtypes; (void)out_dtype; (void)d_out;
    if (nranks != NR || (unsigned)rank >= (unsigned)NR || n_in < 2 || in_sizes[0] != DIM * KL || in_sizes[1] != KL * DIM ||
        out_size != DIM * DIM || ws_bytes < WS_NEED) {
        fprintf(stderr, "kernel_launch: unexpected problem shape\n");
        return;
    }
    Params p;
    p.x = (const char*)d_in[0]; p.w = (const char*)d_in[1]; p.rank = rank;
    for (int r = 0; r < NR; ++r) { p.out[r] = (char*)d_out_all[r]; p.ws[r] = (char*)d_ws_all[r]; }
    p.myws = p.ws[rank];
    build_schedule(p);
    HIP_CHECK(hipFuncSetAttribute((const void*)gemm_ar_kernel, hipFuncAttributeMaxDynamicSharedMemorySize, LDS_B));
    hipLaunchKernelGGL(gemm_ar_kernel, dim3(NLINK + NGEMM), dim3(512), LDS_B, stream, p);
    HIP_CHECK(hipGetLastError());
}
```

```cpp
#include <hip/hip_runtime.h>
#include <rccl/rccl.h>
#include <cstdio>

#define HIP_CHECK(expr) do { hipError_t _e = (expr); if (_e != hipSuccess) { \
    fprintf(stderr, "HIP error %s at %s:%d\n", hipGetErrorString(_e), __FILE__, __LINE__); \
    return; } } while (0)

#define LAS __attribute__((address_space(3)))
#define GAS __attribute__((address_space(1)))
typedef short bf16x8 __attribute__((ext_vector_type(8)));
typedef short s16x4 __attribute__((ext_vector_type(4)));
typedef float f32x4 __attribute__((ext_vector_type(4)));
typedef unsigned u32x4 __attribute__((ext_vector_type(4)));
typedef unsigned u32x2 __attribute__((ext_vector_type(2)));

namespace {
constexpr int NR = 8;
constexpr int DIM = 8192, KL = 1024;
constexpr unsigned KT_B = 32768, HT_B = 16384;
constexpr unsigned SLOT_B = 2 * KT_B;
constexpr unsigned PANEL_B = 32 * SLOT_B;
constexpr unsigned STAGE_B = 32 * PANEL_B;
constexpr unsigned TILE_B = 131072;
constexpr unsigned PIN_OFF = STAGE_B;
constexpr unsigned POUT_OFF = PIN_OFF + 128 * TILE_B;
constexpr unsigned SEND_OFF = POUT_OFF + 128 * TILE_B;
constexpr unsigned DATA_B = SEND_OFF + 2048 * HT_B;
constexpr unsigned CNT_OFF = DATA_B;
constexpr unsigned PF_OFF = DATA_B + 4096;
constexpr unsigned IR_OFF = DATA_B + 8192;
constexpr unsigned PL_OFF = DATA_B + 16384;
constexpr unsigned QC_OFF = DATA_B + 20480;
constexpr unsigned FL_OFF = DATA_B + 32768;
constexpr unsigned HV_OFF = DATA_B + 65536;
constexpr unsigned DONE_OFF = DATA_B + 98304;
constexpr unsigned TAIL_OFF = DONE_OFF + 1024;
constexpr unsigned CALL_OFF = DONE_OFF + 2048;
constexpr unsigned TMO_OFF = CALL_OFF + 128;
constexpr unsigned HEAD_OFF = DONE_OFF + 4096;
constexpr size_t WS_NEED = (size_t)DONE_OFF + 8192;
constexpr unsigned OUT_B = (unsigned)DIM * DIM * 2;
constexpr int NE = 8;
constexpr int NS = 2;
constexpr int NLINK = 8 * NE, NGEMM = 256 - NLINK;
constexpr int NURG = NGEMM / 4;
constexpr int LDS_B = 131072 + 256;
constexpr unsigned long long DEADLINE = 100000000ull;
constexpr int AUX_SYS = 17;
constexpr int AUX_DEV = 16;
#define WIN 3
#define TAILWIN 6
constexpr int TAILN = 64;
constexpr int CLAIM_AT = 10;
constexpr int FLEX_AT = 32;
constexpr int KSL = 3, KS3 = 2;
constexpr int NQUAD = 40;
constexpr int EARLYO = 32;
constexpr int NLATE = 96;
constexpr int URG0 = 48;

struct Params {
    const char* x; const char* w;
    char* myws;
    int rank;
    char* out[NR]; char* ws[NR];
    unsigned units[32];
    unsigned late[NGEMM / 4];
    unsigned tiles[NGEMM];
};

#define STR2(x) #x
#define STR(x) STR2(x)
#define WAIT_V(n) asm volatile("s_waitcnt vmcnt(" STR(n) ")" ::: "memory")
#define WAIT_L(n) asm volatile("s_waitcnt lgkmcnt(" STR(n) ")" ::: "memory")
#define BAR __builtin_amdgcn_s_barrier()
#define SCHED __builtin_amdgcn_sched_barrier(0)


__device__ __forceinline__ void touch_args(const Params& p, int u) {
    const int a = p.rank; const unsigned b = p.units[u]; char* const c = p.out[4]; char* const d = p.ws[4];
    asm volatile("" :: "s"(a), "s"(b), "s"(c), "s"(d));
}
__device__ __forceinline__ unsigned ld_sys(const char* p) {
    return __hip_atomic_load((const GAS unsigned*)p, __ATOMIC_RELAXED, __HIP_MEMORY_SCOPE_SYSTEM);
}
__device__ __forceinline__ void st_sys(char* p, unsigned v) {
    __hip_atomic_store((GAS unsigned*)p, v, __ATOMIC_RELAXED, __HIP_MEMORY_SCOPE_SYSTEM);
}
__device__ __forceinline__ void st8_sys(char* p, unsigned v) {
    __hip_atomic_store((GAS unsigned char*)p, (unsigned char)v, __ATOMIC_RELAXED, __HIP_MEMORY_SCOPE_SYSTEM);
}
__device__ __forceinline__ unsigned call_tag(unsigned call) { return call % 255u + 1u; }
__device__ __forceinline__ void add_sys(char* p, unsigned v) {
    __hip_atomic_fetch_add((GAS unsigned*)p, v, __ATOMIC_RELAXED, __HIP_MEMORY_SCOPE_SYSTEM);
}
__device__ __forceinline__ void draw8(unsigned& pos, char* ws, unsigned off) {
    asm volatile("global_atomic_add %0, %1, %2, %3 sc0" : "+v"(pos) : "v"(off), "v"(8u), "s"(ws) : "memory");
}
__device__ __forceinline__ unsigned cvt_pk_bf16(float lo, float hi) {
    unsigned r; asm volatile("v_cvt_pk_bf16_f32 %0, %1, %2" : "=v"(r) : "v"(lo), "v"(hi)); return r;
}
__device__ __forceinline__ int panel_row(int d, int P) { return ((d >> 1) & 1) * 4096 + ((((P >> 3) ^ (d >> 2) ^ 1) & 1) * 8 + (P & 7)) * 256; }
__device__ __forceinline__ int panel_col(int d, int k) { return (d & 1) * 4096 + k * 256; }
__device__ __forceinline__ int link_slot(int dsel, int cc) { return (cc == 3 ? 8 : 4) + cc * 6 + (dsel < 4 ? dsel - 1 : dsel - 2); }
__device__ __forceinline__ bool give_up(char* tmo, unsigned call, unsigned long long t0) {
    if (__builtin_amdgcn_readfirstlane(__hip_atomic_load((const GAS unsigned*)tmo, __ATOMIC_RELAXED, __HIP_MEMORY_SCOPE_AGENT)) == call
        || __builtin_amdgcn_s_memrealtime() - t0 > DEADLINE) {
        __hip_atomic_store((GAS unsigned*)tmo, call, __ATOMIC_RELAXED, __HIP_MEMORY_SCOPE_AGENT);
        return true;
    }
    return false;
}

struct Piece { const char* src; unsigned stride, dst, pub; int kind; };

__device__ __forceinline__ Piece shard_piece(const Params& p, int d, int panel, int cc, int pc, int dp, int slot) {
    const int k0 = ((d >> 2) * 4 + cc) * 128 + (pc >> 1) * 64, h = pc & 1;
    Piece o; o.kind = panel >= 16;
    o.dst = (unsigned)dp * PANEL_B + (unsigned)slot * SLOT_B + (unsigned)pc * HT_B;
    o.pub = CNT_OFF + (unsigned)dp * 128 + (unsigned)slot * 4 + (unsigned)pc;
    if (o.kind) {
        o.src = p.w + ((size_t)k0 * DIM + panel_col(d, panel - 16) + h * 128) * 2;
        o.stride = 4u * DIM * 2;
    } else {
        o.src = p.x + ((size_t)(panel_row(d, panel) + h * 128) * KL + k0) * 2;
        o.stride = 8u * KL * 2;
    }
    return o;
}
__device__ __forceinline__ int unit_at(const Params& p, int n) { return (p.units[n >> 2] >> ((n & 3) * 8)) & 255; }
__device__ __forceinline__ unsigned send_off(int d, int panel, int cc, int pc) {
    const int gp = panel >= 16 ? 32 + (panel_col(d, panel - 16) >> 8) : panel_row(d, panel) >> 8;
    return SEND_OFF + (unsigned)((gp * 8 + (d >> 2) * 4 + cc) * 4 + pc) * HT_B;
}
__device__ __forceinline__ Piece pull_piece(const Params& p, int code, int dsel) {
    const int cls = code >> 21, pos = (code >> 10) & 2047, id = code & 1023, s = p.rank ^ dsel;
    Piece o; o.kind = 2; o.stride = 1024;
    if (cls == 1) {
        const int u = unit_at(p, (pos >> 2) & 127), panel = u & 31, cc = u >> 5, pc = pos & 3, slot = link_slot(dsel, cc);
        o.src = p.ws[s] + send_off(p.rank, panel, cc, pc);
        o.dst = (unsigned)panel * PANEL_B + (unsigned)slot * SLOT_B + (unsigned)pc * HT_B;
        o.pub = CNT_OFF + (unsigned)panel * 128 + (unsigned)slot * 4 + (unsigned)pc;
    } else if (cls == 0) {
        o.src = p.ws[s] + POUT_OFF + (size_t)id * HT_B;
        o.dst = PIN_OFF + (unsigned)id * HT_B;
        o.pub = PF_OFF + (unsigned)id * 4;
    } else {
        const int ow = cls == 2 ? s : p.rank ^ 4;
        o.kind = 3; o.stride = 2u * DIM * 2;
        o.dst = (unsigned)(panel_row(ow, 8 + (id >> 7)) + (id & 7) * 32) * (DIM * 2) + (unsigned)panel_col(ow, (id >> 3) & 15) * 2;
        o.src = p.out[s] + o.dst;
        o.pub = cls == 2 ? (unsigned)(code & 0xfffff) + 1u : 0u;
    }
    return o;
}
__device__ __forceinline__ const char* piece_src(const Piece& pc, int lane) {
    const unsigned xl = (unsigned)(lane >> 3) * (KL * 2) + (unsigned)((lane & 7) ^ (lane >> 3)) * 16;
    const unsigned wl = (unsigned)(lane >> 4) * (DIM * 2) + (unsigned)((lane & 15) ^ ((lane >> 4) << 2)) * 16;
    const unsigned ol = (unsigned)(lane >> 5) * (DIM * 2) + (unsigned)(lane & 31) * 16;
    return pc.src + (pc.kind == 0 ? xl : pc.kind == 1 ? wl : pc.kind == 2 ? (unsigned)lane * 16 : ol);
}
__device__ __forceinline__ void piece_dma(const Piece& pc, LAS char* buf, int lane) {
    const char* s = piece_src(pc, lane);
#pragma unroll
    for (int f = 0; f < 16; ++f) __builtin_amdgcn_global_load_lds((const GAS unsigned*)(s + (size_t)f * pc.stride), (LAS unsigned*)(buf + f * 1024), 16, 0, AUX_DEV);
}
__device__ __forceinline__ void piece_read_tr(u32x4 (&v)[16], LAS char* buf, int lane) {
    const int q = (lane >> 2) & 3;
    const unsigned trl = (unsigned)(8 * (lane >> 4) + q) * 256 + (unsigned)(lane & 3) * 16;
#pragma unroll
    for (int f = 0; f < 16; ++f) {
        const int nb = f >> 1, kb = f & 1;
        LAS char* a = buf + trl + (((nb >> 1) ^ q) * 64) + kb * 8192 + (nb & 1) * 8;
        const u32x2 lo = __builtin_bit_cast(u32x2, __builtin_amdgcn_ds_read_tr16_b64_v4i16((LAS s16x4*)a));
        const u32x2 hi = __builtin_bit_cast(u32x2, __builtin_amdgcn_ds_read_tr16_b64_v4i16((LAS s16x4*)(a + 1024)));
        v[f] = (u32x4){lo.x, lo.y, hi.x, hi.y};
    }
}
__device__ __forceinline__ void piece_read(u32x4 (&v)[16], bool isx, LAS char* buf, int lane) {
    const unsigned xb = (unsigned)(lane & 15) * 128 + (unsigned)((lane >> 4) ^ (lane & 7)) * 16;
    LAS char* b0 = buf + (isx ? xb : (unsigned)lane * 16);
    LAS char* b1 = buf + (isx ? xb ^ 64u : (unsigned)lane * 16 + 1024u);
#pragma unroll
    for (int f = 0; f < 16; ++f) asm volatile("ds_read_b128 %0, %1 offset:%2" : "=v"(v[f]) : "v"((f & 1) ? b1 : b0), "i"((f >> 1) * 2048) : "memory");
    asm volatile("s_waitcnt lgkmcnt(0)" : "+v"(v[0]), "+v"(v[1]), "+v"(v[2]), "+v"(v[3]), "+v"(v[4]), "+v"(v[5]), "+v"(v[6]), "+v"(v[7]),
                 "+v"(v[8]), "+v"(v[9]), "+v"(v[10]), "+v"(v[11]), "+v"(v[12]), "+v"(v[13]), "+v"(v[14]), "+v"(v[15]) :: "memory");
}

constexpr int NMB = 2 * KSL + KS3;

__device__ __forceinline__ int next_task(LAS unsigned* mb, int lane) {
    for (;;) {
        unsigned v;
        asm volatile("ds_read_b32 %0, %1\n\ts_waitcnt lgkmcnt(0)" : "=&v"(v) : "v"(mb + (lane & 15)) : "memory");
        unsigned best = 0xffffffffu; int bl = 0;
#pragma unroll
        for (int l = 0; l < NMB; ++l) { const unsigned k = (unsigned)__builtin_amdgcn_readlane(v, l) - 1u; if (k < best) { best = k; bl = l; } }
        if (best == 0xffffffffu) return __builtin_amdgcn_readlane(v, NMB) ? -2 : -1;
        unsigned old = 0;
        if (lane == 0) asm volatile("ds_cmpst_rtn_b32 %0, %1, %2, %3\n\ts_waitcnt lgkmcnt(0)" : "=&v"(old) : "v"(mb + bl), "v"(best + 1u), "v"(0u) : "memory");
        if ((unsigned)__builtin_amdgcn_readfirstlane(old) == best + 1u) return (int)(best & 0x7fffffu);
    }
}
static_assert(WIN >= 2 && WIN <= 8 && NS >= 1 && NS <= 3, "window / pullers");
__device__ __forceinline__ void lds_wait_zero(LAS unsigned* a) {
    for (;;) {
        unsigned f;
        asm volatile("ds_read_b32 %0, %1\n\ts_waitcnt lgkmcnt(0)" : "=&v"(f) : "v"(a) : "memory");
        if (__builtin_amdgcn_readfirstlane(f) == 0) break;
        __builtin_amdgcn_s_sleep(1);
    }
}

__device__ __forceinline__ void link_wg(const Params& p, LAS char* lds, int dsel, int e) {
    const int wid = __builtin_amdgcn_readfirstlane(threadIdx.x >> 6), lane = threadIdx.x & 63;
    char* ws = p.myws;
    LAS unsigned* mb = (LAS unsigned*)(lds + 7 * HT_B);
    const int sn = wid < NS ? wid : wid - NS;
    LAS char* buf = lds + sn * 2 * HT_B;
    LAS unsigned* slot = mb + 16 + sn * 8;
    touch_args(p, 0);
    Piece first;
    if (wid >= NS && wid < 2 * NS) {
        if (dsel != 4) {
            const int q = sn * NE + e, u = unit_at(p, q >> 2);
            first = shard_piece(p, p.rank ^ dsel, u & 31, u >> 5, q & 3, u & 31, link_slot(dsel, u >> 5));
            piece_dma(first, buf + HT_B, lane);
        }
    } else if (wid == 7) {
        const unsigned c = ld_sys(ws + CALL_OFF) + 1u;
        mb[lane] = lane == 15 ? c : (lane >= 16 && (lane & 7) == 4 && dsel != 4) ? 14u : 0u; WAIT_L(0);
        if (lane == 0) add_sys(ws + DONE_OFF, 1u);
    }
    BAR; asm volatile("" ::: "memory");
    unsigned call;
    asm volatile("ds_read_b32 %0, %1 offset:60\n\ts_waitcnt lgkmcnt(0)" : "=&v"(call) : "v"(mb) : "memory");
    call = __builtin_amdgcn_readfirstlane(call);
    if (wid >= 2 * NS) {
        if (wid != 7) return;
        const int cls = lane < KSL ? (dsel == 4 ? 0 : 1) : lane < 2 * KSL ? 2 : 3;
        const unsigned nfirst = cls == 1 ? (NS + KSL) * NE : 0u;
        const unsigned n = cls == 1 ? 512u : 1024u;
        const unsigned base = (call - 1u) * (n - nfirst + NE * (cls == 3 ? 7 * KS3 : KSL)) - nfirst;
        char* head = ws + HEAD_OFF + (cls == 3 ? 0 : dsel * 2 + (cls >> 1)) * 128;
        const char* list = (cls == 3 && dsel != 4) ? p.ws[p.rank ^ dsel] + HV_OFF + (dsel ^ 4) * 4096
                                                     : ws + (cls == 0 ? PL_OFF : cls == 1 ? IR_OFF + dsel * 1024 : FL_OFF + dsel * 4096);
        const unsigned tag = call & 0xfffffu;
        int st = (lane >= NMB || (cls == 2 && dsel == 4)) ? 3 : (cls == 3 && dsel != 4) ? 4 : 0;
        unsigned pos = (NS + lane) * NE + e, lo = pos & ~3u;
        if (cls == 1) st = 1;
        const unsigned long long t0 = __builtin_amdgcn_s_memrealtime();
        for (unsigned polls = 1;; ++polls) {
            if (st == 2 && __hip_atomic_load(mb + lane, __ATOMIC_RELAXED, __HIP_MEMORY_SCOPE_WORKGROUP) == 0u) st = 0;
            unsigned t = 0, en = 0;
            if (st == 0) t = __hip_atomic_fetch_add((GAS unsigned*)head, 1u, __ATOMIC_RELAXED, __HIP_MEMORY_SCOPE_AGENT);
            if (st == 1) en = ld_sys(list + lo);
            if (st == 0) {
                pos = t - base; st = pos >= n ? 3 : 1;
                lo = cls != 1 ? pos * 4 : (pos < 4 * NURG ? pos & ~3u : 4u * NURG);
            } else if (st == 1 && (en >> 12) == tag) {
                __hip_atomic_store(mb + lane, ((((unsigned)cls << 11 | pos) << 10) | (en & 1023u)) + 1u, __ATOMIC_RELAXED, __HIP_MEMORY_SCOPE_WORKGROUP);
                st = 2;
            }
            if (__any(cls == 2 && st != 0 && pos >= 1024u - FLEX_AT) && st == 4) st = 0;
            if (__all(st == 3)) break;
            if ((polls & 63u) == 0u && give_up(ws + TMO_OFF, call, t0)) {
                if (lane == 0) st_sys(p.out[p.rank] + ((size_t)panel_row(p.rank, 8) * DIM + (size_t)panel_col(p.rank, 0)) * 2, 0x7fc07fc0u);
                break;
            }
            __builtin_amdgcn_s_sleep(4);
        }
        if (lane == 0) __hip_atomic_store(mb + NMB, 1u, __ATOMIC_RELAXED, __HIP_MEMORY_SCOPE_WORKGROUP);
        return;
    }
    if (wid < NS) {
        int k = 0, code = -1;
        bool pend = false;
        u32x4 desc = {0u, 0u, 0u, 0u};
        Piece cur;
        for (;;) {
            if (code < 0) {
                if (code == -1) code = next_task(mb, lane);
                if (code < 0) {
                    if (pend) { WAIT_V(0); if (lane == 0) asm volatile("ds_write_b128 %0, %1" :: "v"(slot + ((k - 1) & 1) * 4), "v"(desc) : "memory"); pend = false; }
                    while (code == -1) { __builtin_amdgcn_s_sleep(2); code = next_task(mb, lane); }
                    if (code < 0) break;
                }
                cur = pull_piece(p, code, dsel);
            }
            lds_wait_zero(slot + (k & 1) * 4);
            LAS char* b = buf + (k & 1) * HT_B;
            const char* sp = piece_src(cur, lane);
            const unsigned stride = cur.stride;
            const u32x4 ndesc = {(unsigned)cur.kind + 1u, cur.dst, cur.pub, 0u};
            const bool fast = (code >> 10) >= (3 << 11 | (1024 - TAILN));
            const bool wide = sn == 0 && (code >> 22) == 0;
#pragma unroll
            for (int f = 0; f < 16; ++f) {
                __builtin_amdgcn_global_load_lds((const GAS unsigned*)sp, (LAS unsigned*)(b + f * 1024), 16, 0, AUX_SYS);
                sp += stride;
                asm volatile("" ::: "memory");
                if (f == CLAIM_AT) { code = next_task(mb, lane); if (code >= 0) cur = pull_piece(p, code, dsel); }
                if (f > WIN - 2 && fast) WAIT_V(TAILWIN - 1); else if (f != WIN - 2 && wide) WAIT_V(WIN); else WAIT_V(WIN - 1);
                if (f == WIN - 2 && pend && lane == 0)
                    asm volatile("ds_write_b128 %0, %1" :: "v"(slot + ((k - 1) & 1) * 4), "v"(desc) : "memory");
            }
            desc = ndesc; pend = true; ++k;
        }
        lds_wait_zero(slot + (k & 1) * 4);
        if (lane == 0) asm volatile("ds_write_b32 %0, %1" :: "v"(slot + (k & 1) * 4), "v"(15u) : "memory");
        return;
    }
    if (dsel != 4) {
        char* dws = p.ws[p.rank ^ dsel];
        const __amdgpu_buffer_rsrc_t rs = __builtin_amdgcn_make_buffer_rsrc(dws, 0, STAGE_B, 0x00020000);
        WAIT_V(0);
        u32x4 v[16];
        if (first.kind) { piece_read_tr(v, buf + HT_B, lane); WAIT_L(0);
#pragma unroll
            for (int f = 0; f < 16; ++f) asm volatile("" : "+v"(v[f]));
        } else piece_read(v, true, buf + HT_B, lane);
        if (lane == 0) asm volatile("ds_write_b32 %0, %1" :: "v"(slot + 4), "v"(0u) : "memory");
#pragma unroll
        for (int f = 0; f < 16; ++f) { __builtin_amdgcn_raw_buffer_store_b128(v[f], rs, first.dst + lane * 16 + f * 1024, 0, AUX_SYS); WAIT_V(WIN - 1); }
        WAIT_V(0);
        if (lane == 0) st8_sys(dws + first.pub, call_tag(call));
    }
    const __amdgpu_buffer_rsrc_t rsw = __builtin_amdgcn_make_buffer_rsrc(ws, 0, POUT_OFF, 0x00020000);
    const __amdgpu_buffer_rsrc_t rso = __builtin_amdgcn_make_buffer_rsrc(p.out[p.rank], 0, OUT_B, 0x00020000);
    const unsigned ol = (unsigned)(lane >> 5) * (DIM * 2) + (unsigned)(lane & 31) * 16;
    char* fwd = ws + HV_OFF + dsel * 4096;
    for (int k = 0;;) {
        u32x4 ds4;
        asm volatile("ds_read_b128 %0, %1\n\ts_waitcnt lgkmcnt(0)" : "=&v"(ds4) : "v"(slot + (k & 1) * 4) : "memory");
        const int kind = __builtin_amdgcn_readfirstlane(ds4.x) - 1;
        if (kind > 3) break;
        if (kind < 0) { __builtin_amdgcn_s_sleep(1); continue; }
        const unsigned dst = __builtin_amdgcn_readfirstlane(ds4.y), pub = __builtin_amdgcn_readfirstlane(ds4.z);
        u32x4 v[16];
        piece_read(v, false, buf + (k & 1) * HT_B, lane);
        if (lane == 0) asm volatile("ds_write_b32 %0, %1" :: "v"(slot + (k & 1) * 4), "v"(0u) : "memory");
        const bool o = kind == 3;
        const __amdgpu_buffer_rsrc_t rs = o ? rso : rsw;
        const unsigned ds = o ? 2u * DIM * 2 : 1024u;
        unsigned vo = dst + (o ? ol : (unsigned)lane * 16);
#pragma unroll
        for (int f = 0; f < 16; ++f) {
            __builtin_amdgcn_raw_buffer_store_b128(v[f], rs, vo, 0, AUX_SYS);
            vo += ds;
        }
        if (pub) {
            WAIT_V(0);
            if (lane == 0) {
                if (o) st_sys(fwd + ((pub - 1u) >> 10) * 4, (call & 0xfffffu) << 12 | ((pub - 1u) & 1023u));
                else if (pub >= PF_OFF) st_sys(ws + pub, call);
                else st8_sys(ws + pub, call_tag(call));
            }
        }
        ++k;
    }
}

__device__ __forceinline__ int ready_prefix(unsigned v, unsigned target) {
    const unsigned long long m = __ballot(v == target);
    const unsigned both = (unsigned)m & (unsigned)(m >> 32);
    return both == 0xffffffffu ? 32 : __builtin_ctz(~both);
}
__device__ __forceinline__ void kn_put(LAS unsigned* kn, unsigned v) { asm volatile("ds_write_b32 %0, %1" :: "v"(kn), "v"(v) : "memory"); }
__device__ __noinline__ int wait_slot(const char* cptr, char* tmo, unsigned call, int need, LAS unsigned* kn, int job, int duty) {
    int known;
    if (__builtin_amdgcn_readfirstlane(duty)) {
        const unsigned long long t0 = __builtin_amdgcn_s_memrealtime();
        for (unsigned polls = 1;; ++polls) {
            known = ready_prefix(ld_sys(cptr), call_tag(call) * 0x01010101u);
            if (known > need) break;
            if ((polls & 63u) == 0u && give_up(tmo, call, t0)) { known = 32; break; }
            __builtin_amdgcn_s_sleep(32);
        }
        kn_put(kn, (unsigned)(job + known));
    } else {
        for (;;) {
            unsigned f;
            asm volatile("ds_read_b32 %0, %1\n\ts_waitcnt lgkmcnt(0)" : "=&v"(f) : "v"(kn) : "memory");
            known = (int)f - job;
            if (known > need) break;
            __builtin_amdgcn_s_sleep(4);
        }
    }
    return known;
}

__device__ __forceinline__ void gemm_wg(const Params& p, LAS char* lds, int gb) {
    const int tid = threadIdx.x, wid = __builtin_amdgcn_readfirstlane(tid >> 6), lane = tid & 63, wr = wid >> 2, wc = wid & 3;
    char* ws = p.myws;
    unsigned cv;
    asm volatile("global_load_dword %0, %1, %2 sc0 sc1" : "=v"(cv) : "v"(CALL_OFF), "s"(ws) : "memory");
    const int late = (int)(signed char)((p.late[gb >> 2] >> ((gb & 3) * 8)) & 255);
    const bool bulk = wid >= 4;
    LAS unsigned* ucnt = (LAS unsigned*)(lds + 131072);
    if (tid < 2) ucnt[tid] = 0u;
    touch_args(p, gb >> 4);
    WAIT_L(0); BAR; asm volatile("" ::: "memory");
    for (int i = bulk ? (late >= 0 ? 4 + wid : 0) : (gb < URG0 ? 0 : 2 + ((gb - URG0) >> 5)); i > 0; --i) __builtin_amdgcn_s_sleep(32);
    unsigned nbulk = 0;
    for (int r = !bulk ? gb * 4 + wid : late >= 0 ? 4 * NGEMM + (wid - 4) * NLATE + late : 2048; r < 2048; r = bulk ? r + 4 * NLATE : 2048) {
        const int other = (r >> 3) & 1, half = (r >> 2) & 1, pcs = r & 3;
        const int u = unit_at(p, r >> 4), panel = u & 31, cc = u >> 5;
        const int d0 = (((p.rank >> 2) ^ other) << 2) + (panel >= 16 ? half : half * 2), d1 = d0 + (panel >= 16 ? 2 : 1);
        const bool own0 = ((d0 ^ p.rank) & 3) == 0, own1 = ((d1 ^ p.rank) & 3) == 0, own = own0 || own1;
        const Piece pc = shard_piece(p, d0, panel, cc, pcs, panel < 16 && other ? panel ^ 8 : panel, other * 22 + cc);
        LAS char* buf = lds + wid * HT_B;
        piece_dma(pc, buf, lane);
        asm volatile("s_waitcnt vmcnt(0)" : "+v"(cv) :: "memory");
        const unsigned call = __builtin_amdgcn_readfirstlane(cv) + 1u;
        u32x4 v[16];
        if (pc.kind) piece_read_tr(v, buf, lane); else piece_read(v, true, buf, lane);
        const __amdgpu_buffer_rsrc_t rs = __builtin_amdgcn_make_buffer_rsrc(ws, 0, DATA_B, 0x00020000);
        const unsigned so = send_off(d0, panel, cc, pcs);
#pragma unroll
        for (int f = 0; f < 16; ++f) __builtin_amdgcn_raw_buffer_store_b128(v[f], rs, so + lane * 16 + f * 1024, 0, AUX_SYS);
        if (own) {
#pragma unroll
            for (int f = 0; f < 16; ++f) __builtin_amdgcn_raw_buffer_store_b128(v[f], rs, pc.dst + lane * 16 + f * 1024, 0, AUX_SYS);
        }
        WAIT_V(0);
        ++nbulk;
        if (lane == 0) {
            if (own) st8_sys(ws + pc.pub, call_tag(call));
            if (!bulk && __hip_atomic_fetch_add(ucnt, 1u, __ATOMIC_RELAXED, __HIP_MEMORY_SCOPE_WORKGROUP) == 3u) {
                if (!own0) st_sys(p.ws[d0] + IR_OFF + (p.rank ^ d0) * 1024 + (r >> 4) * 4, (call & 0xfffffu) << 12);
                if (!own1) st_sys(p.ws[d1] + IR_OFF + (p.rank ^ d1) * 1024 + (r >> 4) * 4, (call & 0xfffffu) << 12);
            }
        }
    }
    asm volatile("s_waitcnt vmcnt(0)" : "+v"(cv) :: "memory");
    const unsigned call = __builtin_amdgcn_readfirstlane(cv) + 1u;
    if (bulk && nbulk && lane == 0 &&
        __hip_atomic_fetch_add((GAS unsigned*)(ws + TAIL_OFF + 256), nbulk, __ATOMIC_RELAXED, __HIP_MEMORY_SCOPE_AGENT) + nbulk == call * (2048u - 4u * NGEMM)) {
#pragma unroll
        for (int q = 1; q < NR; ++q) if (q != 4) st_sys(p.ws[p.rank ^ q] + IR_OFF + q * 1024 + NURG * 4, (call & 0xfffffu) << 12);
    }
    BAR; asm volatile("" ::: "memory");
    if (tid == 0) add_sys(ws + DONE_OFF, 1u);
    const unsigned target = call_tag(call) * 0x01010101u;
    const unsigned voff = (unsigned)tid * 16;
    const unsigned ldsw = (unsigned)wid * 1024u;
    const int aoff = wr * 8192 + lane * 16, boff = wc * 4096 + lane * 16;
#define SA(b, h) (((b) * 2 + (h)) * HT_B)
#define SB(b, h) ((4 + (b) * 2 + (h)) * HT_B)
#define STAGE(bufoff, gbase) do { _Pragma("unroll") for (int _i = 0; _i < 2; ++_i) \
        __builtin_amdgcn_global_load_lds((const GAS unsigned*)((const char*)(gbase) + voff + _i * 8192), (LAS unsigned*)(lds + (bufoff) + ldsw + _i * 8192), 16, 0, AUX_DEV); } while (0)
#define LDA(dst, b, h) do { _Pragma("unroll") for (int m = 0; m < 4; ++m) _Pragma("unroll") for (int k = 0; k < 2; ++k) dst[m][k] = *(const LAS bf16x8*)(lds + SA(b, h) + aoff + m * 2048 + k * 1024); } while (0)
#define LDB(dst, b, h) do { _Pragma("unroll") for (int n = 0; n < 2; ++n) _Pragma("unroll") for (int k = 0; k < 2; ++k) dst[n][k] = *(const LAS bf16x8*)(lds + SB(b, h) + boff + n * 2048 + k * 1024); } while (0)
#define MMA(ai, bj, At, Bt) do { __builtin_amdgcn_s_setprio(1); _Pragma("unroll") for (int m = 0; m < 4; ++m) _Pragma("unroll") for (int n = 0; n < 2; ++n) _Pragma("unroll") for (int k = 0; k < 2; ++k) \
        acc[ai][bj][m][n] = __builtin_amdgcn_mfma_f32_16x16x32_bf16(Bt[n][k], At[m][k], acc[ai][bj][m][n], 0, 0, 0); __builtin_amdgcn_s_setprio(0); } while (0)
    for (int rnd = 0; rnd < 2; ++rnd) {
        const int code = (p.tiles[gb] >> (16 * rnd)) & 0xffff;
        if (code == 0xffff) break;
        const int pm = (code >> 4) & 15, pn = code & 15, part = (code >> 8) & 3;
        const char* Ab = ws + (size_t)pm * PANEL_B;
        const char* Bb = ws + (size_t)(16 + pn) * PANEL_B;
        const unsigned coff = CNT_OFF + (unsigned)(lane < 32 ? pm : 16 + pn) * 128 + (unsigned)(lane & 31) * 4;
        f32x4 acc[2][2][4][2];
#pragma unroll
        for (int a = 0; a < 2; ++a)
#pragma unroll
            for (int b = 0; b < 2; ++b)
#pragma unroll
                for (int m = 0; m < 4; ++m)
#pragma unroll
                    for (int n = 0; n < 2; ++n) acc[a][b][m][n] = (f32x4){0.f, 0.f, 0.f, 0.f};
        bf16x8 At[4][2], B0[2][2], B1[2][2];
        if (code >> 10) {
            const char* Aq = Ab + (part >> 1) * HT_B; const char* Bq = Bb + (part & 1) * HT_B;
            int kn = __builtin_amdgcn_readfirstlane(wait_slot(ws + coff, ws + TMO_OFF, call, 1, ucnt + 1, rnd * 64, wid == 0));
            STAGE(0, Aq); STAGE(HT_B, Bq); STAGE(KT_B, Aq + KT_B); STAGE(KT_B + HT_B, Bq + KT_B); STAGE(2 * KT_B, Aq + 2 * KT_B); STAGE(2 * KT_B + HT_B, Bq + 2 * KT_B);
            for (int kt = 0; kt < 64; ++kt) {
                const int nx = kt + 3 < 64 ? kt + 3 : 63;
                if (kn <= nx >> 1) kn = __builtin_amdgcn_readfirstlane(wait_slot(ws + coff, ws + TMO_OFF, call, nx >> 1, ucnt + 1, rnd * 64, wid == 0));
                WAIT_V(8); BAR; asm volatile("" ::: "memory");
                const unsigned sn = (unsigned)((kt + 3) & 3) * KT_B, sc = (unsigned)(kt & 3) * KT_B;
                STAGE(sn, Aq + (unsigned)nx * KT_B); STAGE(sn + HT_B, Bq + (unsigned)nx * KT_B);
#pragma unroll
                for (int n = 0; n < 2; ++n)
#pragma unroll
                    for (int k = 0; k < 2; ++k) asm volatile("ds_read_b128 %0, %1 offset:%2" : "=v"(B0[n][k]) : "v"(lds + sc + boff), "i"(HT_B + n * 2048 + k * 1024) : "memory");
#pragma unroll
                for (int m = 0; m < 4; ++m)
#pragma unroll
                    for (int k = 0; k < 2; ++k) asm volatile("ds_read_b128 %0, %1 offset:%2" : "=v"(At[m][k]) : "v"(lds + sc + aoff), "i"(m * 2048 + k * 1024) : "memory");
                asm volatile("s_waitcnt lgkmcnt(0)" : "+v"(B0[0][0]), "+v"(B0[0][1]), "+v"(B0[1][0]), "+v"(B0[1][1]), "+v"(At[0][0]), "+v"(At[0][1]), "+v"(At[1][0]), "+v"(At[1][1]),
                             "+v"(At[2][0]), "+v"(At[2][1]), "+v"(At[3][0]), "+v"(At[3][1]) :: "memory");
                MMA(0, 0, At, B0);
            }
            const bool bad = __hip_atomic_load((const GAS unsigned*)(ws + TMO_OFF), __ATOMIC_RELAXED, __HIP_MEMORY_SCOPE_AGENT) == call;
            const int id0 = (pm * 16 + pn) * 8;
            const __amdgpu_buffer_rsrc_t rp = __builtin_amdgcn_make_buffer_rsrc(ws + POUT_OFF, 0, 128 * TILE_B, 0x00020000);
            const unsigned base = (unsigned)(id0 + wid) * HT_B + (unsigned)lane * 16 + (unsigned)((part >> 1) * 8 + (part & 1)) * 1024;
            asm volatile("s_nop 15" ::: "memory");
#pragma unroll
            for (int m = 0; m < 4; ++m) {
                const f32x4 v0 = acc[0][0][m][0], v1 = acc[0][0][m][1];
                u32x4 w; w.x = cvt_pk_bf16(v0[0], v0[1]); w.y = cvt_pk_bf16(v0[2], v0[3]); w.z = cvt_pk_bf16(v1[0], v1[1]); w.w = cvt_pk_bf16(v1[2], v1[3]);
                if (bad) w = (u32x4){0x7fc07fc0u, 0x7fc07fc0u, 0x7fc07fc0u, 0x7fc07fc0u};
                __builtin_amdgcn_raw_buffer_store_b128(w, rp, base + (unsigned)m * 2048, 0, AUX_SYS);
            }
            WAIT_V(0); BAR; asm volatile("" ::: "memory");
            if (wid == 0) {
                unsigned c = 0;
                if (lane == 0) c = __hip_atomic_fetch_add((GAS unsigned*)(ws + QC_OFF + id0 * 4), 1u, __ATOMIC_RELAXED, __HIP_MEMORY_SCOPE_AGENT);
                if ((__builtin_amdgcn_readfirstlane(c) & 3u) == 3u) {
                    if (lane == 0) c = __hip_atomic_fetch_add((GAS unsigned*)(ws + TAIL_OFF), 8u, __ATOMIC_RELAXED, __HIP_MEMORY_SCOPE_AGENT);
                    c = __builtin_amdgcn_readfirstlane(c) & 1023u;
                    if (lane < 8) st_sys(p.ws[p.rank ^ 4] + PL_OFF + (c + lane) * 4, (call & 0xfffffu) << 12 | (unsigned)(id0 + lane));
                }
            }
            continue;
        }
        int known = __builtin_amdgcn_readfirstlane(wait_slot(ws + coff, ws + TMO_OFF, call, 0, ucnt + 1, rnd * 64, wid == 0));
        STAGE(SB(0, 0), Bb); STAGE(SA(0, 0), Ab); STAGE(SB(0, 1), Bb + HT_B); STAGE(SA(0, 1), Ab + HT_B);
        if (wr == 1) BAR;
        WAIT_V(4); BAR;
        STAGE(SB(1, 0), Bb + KT_B); STAGE(SA(1, 0), Ab + KT_B); STAGE(SB(1, 1), Bb + KT_B + HT_B);
        WAIT_V(6); BAR;
#pragma nounroll
        for (int T = 0; T < 32; ++T) {
            if (T < 31 && known <= T + 1) known = __builtin_amdgcn_readfirstlane(wait_slot(ws + coff, ws + TMO_OFF, call, T + 1, ucnt + 1, rnd * 64, wid == 0));
            const unsigned o1 = (unsigned)T * SLOT_B, o2 = (unsigned)(T < 31 ? T + 1 : T) * SLOT_B;
            const char* a1 = Ab + o1 + KT_B; const char* a2 = Ab + o2; const char* b2 = Bb + o2;
            const char* a3 = a2 + KT_B; const char* b3 = b2 + KT_B;
            unsigned pv = 0;
            const bool ask = known < 32;
            LDB(B0, 0, 0); SCHED; LDA(At, 0, 0);
            if (ask) asm volatile("global_load_dword %0, %1, %2 sc0 sc1" : "+v"(pv) : "v"(coff), "s"(ws) : "memory");
            STAGE(SA(1, 1), a1 + HT_B);
            WAIT_L(8); BAR; WAIT_L(0); MMA(0, 0, At, B0); BAR; SCHED;
            LDB(B1, 0, 1); STAGE(SB(0, 0), b2);
            BAR; WAIT_L(0); MMA(0, 1, At, B1); BAR;
            LDA(At, 0, 1); STAGE(SA(0, 0), a2);
            BAR; WAIT_L(0); MMA(1, 0, At, B0); BAR; SCHED;
            STAGE(SB(0, 1), b2 + HT_B);
            WAIT_V(6);
            asm volatile("" : "+v"(pv));
            { const int k2 = ready_prefix(pv, target); if (k2 > known) { known = k2; if (wid == 0) kn_put(ucnt + 1, (unsigned)(rnd * 64 + k2)); } }
            BAR; MMA(1, 1, At, B1); BAR;
            LDB(B0, 1, 0); SCHED; LDA(At, 1, 0); STAGE(SA(0, 1), a2 + HT_B);
            WAIT_L(8); BAR; WAIT_L(0); MMA(0, 0, At, B0); BAR; SCHED;
            LDB(B1, 1, 1); STAGE(SB(1, 0), b3);
            BAR; WAIT_L(0); MMA(0, 1, At, B1); BAR;
            LDA(At, 1, 1); STAGE(SA(1, 0), a3);
            BAR; WAIT_L(0); MMA(1, 0, At, B0); BAR; SCHED;
            STAGE(SB(1, 1), b3 + HT_B);
            WAIT_V(6); BAR; MMA(1, 1, At, B1); BAR;
        }
        WAIT_V(0);
        if (wr == 0) BAR;
        BAR; asm volatile("" ::: "memory");
        int tz = threadIdx.x; asm volatile("" : "+v"(tz));
        const int ln = tz & 63;
        const int id = ((pm & 7) * 16 + pn) * 8 + wid;
        if (pm < 8) {
            unsigned pos = 0;
            if (tz == 0) draw8(pos, ws, TAIL_OFF);
            const bool bad = __hip_atomic_load((const GAS unsigned*)(ws + TMO_OFF), __ATOMIC_RELAXED, __HIP_MEMORY_SCOPE_AGENT) == call;
            const __amdgpu_buffer_rsrc_t rp = __builtin_amdgcn_make_buffer_rsrc(ws + POUT_OFF, 0, 128 * TILE_B, 0x00020000);
            const unsigned base = (unsigned)id * HT_B + (unsigned)ln * 16;
#pragma unroll
            for (int ai = 0; ai < 2; ++ai)
#pragma unroll
                for (int m = 0; m < 4; ++m)
#pragma unroll
                    for (int bj = 0; bj < 2; ++bj) {
                        const f32x4 v0 = acc[ai][bj][m][0], v1 = acc[ai][bj][m][1];
                        u32x4 w; w.x = cvt_pk_bf16(v0[0], v0[1]); w.y = cvt_pk_bf16(v0[2], v0[3]); w.z = cvt_pk_bf16(v1[0], v1[1]); w.w = cvt_pk_bf16(v1[2], v1[3]);
                        if (bad) w = (u32x4){0x7fc07fc0u, 0x7fc07fc0u, 0x7fc07fc0u, 0x7fc07fc0u};
                        __builtin_amdgcn_raw_buffer_store_b128(w, rp, base + (unsigned)((ai * 4 + m) * 2 + bj) * 1024, 0, AUX_SYS);
                    }
            asm volatile("s_waitcnt vmcnt(0)" : "+v"(pos) :: "memory");
            BAR; asm volatile("" ::: "memory");
            pos = __builtin_amdgcn_readfirstlane(pos) & 1023u;
            if (tz < 8) st_sys(p.ws[p.rank ^ 4] + PL_OFF + (pos + tz) * 4, (call & 0xfffffu) << 12 | (unsigned)(id + tz));
            continue;
        }
        {
            const unsigned long long t0 = __builtin_amdgcn_s_memrealtime();
            for (unsigned polls = 1;; ++polls) {
                if ((int)(__builtin_amdgcn_readfirstlane(ld_sys(ws + PF_OFF + id * 4)) - call) >= 0) break;
                if ((polls & 63u) == 0u && give_up(ws + TMO_OFF, call, t0)) break;
                __builtin_amdgcn_s_sleep(127);
            }
            asm volatile("" ::: "memory");
            const bool bad = __builtin_amdgcn_readfirstlane(__hip_atomic_load((const GAS unsigned*)(ws + TMO_OFF), __ATOMIC_RELAXED, __HIP_MEMORY_SCOPE_AGENT)) == call;
            const __amdgpu_buffer_rsrc_t rp = __builtin_amdgcn_make_buffer_rsrc(ws + PIN_OFF, 0, 128 * TILE_B, 0x00020000);
            const unsigned base = (unsigned)id * HT_B + (unsigned)ln * 16;
            u32x4 pr[16];
#pragma unroll
            for (int f = 0; f < 16; ++f) pr[f] = __builtin_amdgcn_raw_buffer_load_b128(rp, base + f * 1024, 0, AUX_SYS);
            const int fr = ln & 15, fq = ln >> 4;
#pragma unroll
            for (int ai = 0; ai < 2; ++ai)
#pragma unroll
                for (int m = 0; m < 4; ++m) {
                    const int row = ai * 128 + wr * 64 + m * 16 + fr;
#pragma unroll
                    for (int bj = 0; bj < 2; ++bj) {
                        const u32x4 q = pr[(ai * 4 + m) * 2 + bj];
                        const f32x4 v0 = acc[ai][bj][m][0], v1 = acc[ai][bj][m][1];
                        u32x4 w;
                        w.x = cvt_pk_bf16(v0[0] + __uint_as_float(q.x << 16), v0[1] + __uint_as_float(q.x & 0xffff0000u));
                        w.y = cvt_pk_bf16(v0[2] + __uint_as_float(q.y << 16), v0[3] + __uint_as_float(q.y & 0xffff0000u));
                        w.z = cvt_pk_bf16(v1[0] + __uint_as_float(q.z << 16), v1[1] + __uint_as_float(q.z & 0xffff0000u));
                        w.w = cvt_pk_bf16(v1[2] + __uint_as_float(q.w << 16), v1[3] + __uint_as_float(q.w & 0xffff0000u));
                        if (bad) w = (u32x4){0x7fc07fc0u, 0x7fc07fc0u, 0x7fc07fc0u, 0x7fc07fc0u};
                        const int ch = (bj * 16 + wc * 4 + fq) ^ (fr & 7);
                        *(LAS u32x4*)(lds + row * 512 + ch * 16) = w;
                    }
                }
        }
        WAIT_L(0); BAR; asm volatile("" ::: "memory");
        {
            unsigned pos = 0;
            if (tz == 0) draw8(pos, ws, TAIL_OFF + 128);
            const __amdgpu_buffer_rsrc_t ro = __builtin_amdgcn_make_buffer_rsrc(p.out[p.rank], 0, OUT_B, 0x00020000);
            const unsigned vo = (unsigned)(panel_row(p.rank, pm) + wid * 32 + (ln >> 5)) * (DIM * 2) + (unsigned)panel_col(p.rank, pn) * 2 + (unsigned)(ln & 31) * 16;
            const int hi = ln >> 5, chn = ln & 31;
#pragma unroll
            for (int f = 0; f < 16; ++f) {
                const u32x4 v = *(const LAS u32x4*)(lds + (wid * 32 + 2 * f + hi) * 512 + ((chn ^ ((2 * f + hi) & 7)) * 16));
                __builtin_amdgcn_raw_buffer_store_b128(v, ro, vo + (unsigned)(2 * f) * (DIM * 2), 0, AUX_SYS);
            }
            asm volatile("s_waitcnt vmcnt(0)" : "+v"(pos) :: "memory");
            BAR; asm volatile("" ::: "memory");
            pos = __builtin_amdgcn_readfirstlane(pos) & 1023u;
            if (tz < 8) {
#pragma unroll
                for (int q = 1; q < NR; ++q) st_sys(p.ws[p.rank ^ q] + FL_OFF + (q * 1024 + pos + tz) * 4, (call & 0xfffffu) << 12 | (unsigned)(id + tz));
            }
        }
        break;
    }
#undef SA
#undef SB
#undef STAGE
#undef LDA
#undef LDB
#undef MMA
}

__global__ __launch_bounds__(512, 2) void gemm_ar_kernel(const Params p) {
    extern __shared__ __attribute__((aligned(16))) char smem[];
    LAS char* lds = (LAS char*)smem;
    const int b = blockIdx.x;
    if (b >= NLINK) { gemm_wg(p, lds, b - NLINK); return; }
    if ((b + (b >> 3)) & 7) { link_wg(p, lds, (b + (b >> 3)) & 7, b >> 3); return; }
    if (b == 0 && threadIdx.x < 64) {
        char* ws = p.myws;
        const unsigned call = __builtin_amdgcn_readfirstlane(ld_sys(ws + CALL_OFF)) + 1u;
        const unsigned long long t0 = __builtin_amdgcn_s_memrealtime();
        for (unsigned polls = 1;; ++polls) {
            if ((int)(__builtin_amdgcn_readfirstlane(ld_sys(ws + DONE_OFF)) - call * (unsigned)(7 * NE + NGEMM)) >= 0) break;
            if ((polls & 63u) == 0u && give_up(ws + TMO_OFF, call, t0)) break;
            __builtin_amdgcn_s_sleep(32);
        }
        if (threadIdx.x == 0) st_sys(ws + CALL_OFF, call);
    }
}

void build_schedule(Params& p) {
    static const int bt[] = {0, 16, 1, 17, 2, 18, 19, -1, 3, 4, 20, 21, -1, 8, 9, -1, 22, 23, -1, 5, 24, 25, -1, 10, 11, -1,
                             6, 26, 27, -1, 7, 28, 29, -1, 12, 13, -1, 30, 31, -1, 14, 15, -1};
    int pos[32], nu = 0, nb = 0;
    unsigned char* un = (unsigned char*)p.units;
    unsigned short* tl = (unsigned short*)p.tiles;
    for (int i = 0; i < (int)(sizeof(bt) / sizeof(bt[0])); ++nb) {
        int e = i; while (bt[e] >= 0) pos[bt[e++]] = nb;
        for (int c = 0; c < 4; ++c) for (int k = i; k < e; ++k) un[nu++] = (unsigned char)(bt[k] | c << 5);
        i = e + 1;
    }
    for (int i = 0; i < NGEMM; ++i) p.tiles[i] = 0xffffffffu;
    int next = 0, nf = 0;
    const int LF = 128 + 3 * NQUAD - NGEMM;
    auto second = [&](int j) { return j < LF ? j + EARLYO : j < LF + EARLYO ? j - LF : j; };
    for (int own = 0; own < 2; ++own)
        for (int b = 0; b < nb; ++b) {
            int list[512], n = 0;
            for (int r = 0; r < 8; ++r) for (int c = 0; c < 16; ++c) {
                const int a = pos[own * 8 + r], bb = pos[16 + c], t = (own * 8 + r) << 4 | c;
                if ((a > bb ? a : bb) != b) continue;
                if (!own && nf++ < NQUAD) { for (int k = 0; k < 4; ++k) list[n++] = t | (4 + k) << 8; } else list[n++] = t;
            }
            int k = 0;
            for (int x = 0; x < 8; ++x) for (int s = next; s < next + n; ++s) if ((s & 7) == x)
                tl[s < NGEMM ? 2 * s : 2 * second(s - NGEMM) + 1] = (unsigned short)list[k++];
            next += n;
        }
    signed char* lt = (signed char*)p.late;
    int nl = 0;
    for (int i = 0; i < NGEMM; ++i) lt[i] = -1;
    for (int b = nb - 1; b >= 0; --b) for (int i = NGEMM - 1; i >= 0 && nl < NLATE; --i) {
        const int a = pos[(tl[2 * i] >> 4) & 15], bb = pos[16 + (tl[2 * i] & 15)];
        if ((a > bb ? a : bb) == b) lt[i] = (signed char)nl++;
    }
}
static_assert(256 + 3 * NQUAD <= 2 * NGEMM && 128 + 3 * NQUAD >= NGEMM && NGEMM % 8 == 0, "two jobs per workgroup, an owned tile last");
}

extern "C" void kernel_launch(int rank, int nranks, ncclComm_t comm,
                              hipStream_t stream,
                              void* const* d_in, const int* in_sizes, int n_in,
                              void* d_out, int out_size,
                              const int* in_dtypes, int out_dtype,
                              void* const* d_out_all,
                              void* const* d_ws_all, size_t ws_bytes) {
    (void)comm; (void)in_dtypes; (void)out_dtype; (void)d_out;
    if (nranks != NR || (unsigned)rank >= (unsigned)NR || n_in < 2 || in_sizes[0] != DIM * KL || in_sizes[1] != KL * DIM ||
        out_size != DIM * DIM || ws_bytes < WS_NEED) {
        fprintf(stderr, "kernel_launch: unexpected problem shape\n");
        return;
    }
    Params p;
    p.x = (const char*)d_in[0]; p.w = (const char*)d_in[1]; p.rank = rank;
    for (int r = 0; r < NR; ++r) { p.out[r] = (char*)d_out_all[r]; p.ws[r] = (char*)d_ws_all[r]; }
    p.myws = p.ws[rank];
    build_schedule(p);
    HIP_CHECK(hipFuncSetAttribute((const void*)gemm_ar_kernel, hipFuncAttributeMaxDynamicSharedMemorySize, LDS_B));
    hipLaunchKernelGGL(gemm_ar_kernel, dim3(NLINK + NGEMM), dim3(512), LDS_B, stream, p);
    HIP_CHECK(hipGetLastError());
}
```
